```python
import numpy as np

kernel_name = "gpu_gemm_rs_m8192_k8192_n8192_bf16_relu"


ABSMAX_RTOL = 0.008

COMPUTE_DTYPE = "bfloat16"
EPILOGUE = "relu"
IN_QUANT = None
OUT_QUANT = None
IN_DTYPES = {"x": "bfloat16", "w_mat": "bfloat16"}
WORLD_SIZE = 8

def _to_bf16(x):
    u = np.ascontiguousarray(x, dtype=np.float32).view(np.uint32)
    return ((u + 0x7FFF + ((u >> 16) & 1)) & 0xFFFF0000).view(np.float32)

def setup_inputs():

    x = _to_bf16(np.random.randn(8192, 8192).astype(np.float32))
    w_mat = _to_bf16((np.random.randn(8192, 8192) / np.sqrt(8192)).astype(np.float32))
    return {"x": x, "w_mat": w_mat}

def _epilogue(y):
    return np.maximum(y, 0.0).astype(np.float32)

def reference(x, w_mat):
    y = (x @ w_mat).astype(np.float32)
    return _epilogue(y)
```

```cpp
#include <hip/hip_runtime.h>
#include <rccl/rccl.h>
#include <cstdio>

#define HIP_CHECK(expr) do { hipError_t _e = (expr); if (_e != hipSuccess) { \
    fprintf(stderr, "HIP error %s at %s:%d\n", hipGetErrorString(_e), __FILE__, __LINE__); \
    return; } } while (0)

namespace {

#define LAS __attribute__((address_space(3)))
typedef short bf16x8 __attribute__((ext_vector_type(8)));
typedef short s16x4 __attribute__((ext_vector_type(4)));
typedef float f32x4 __attribute__((ext_vector_type(4)));
typedef unsigned u32x4 __attribute__((ext_vector_type(4)));
typedef unsigned u32x2 __attribute__((ext_vector_type(2)));
typedef unsigned short u16x2 __attribute__((ext_vector_type(2)));
typedef __amdgpu_buffer_rsrc_t rsrc_t;

constexpr int NR = 8;
constexpr unsigned MiB = 1u << 20;
constexpr unsigned OFF_CALL = 0;
constexpr unsigned OFF_TMO = 32768;
constexpr unsigned OFF_RDY = 36864;
constexpr unsigned OFF_RECI = 65536;
constexpr unsigned OFF_RECO = 327680;
constexpr unsigned OFF_DBG = 524288;
constexpr unsigned CTL_BYTES = MiB;
constexpr unsigned OFF_PKI = 1 * MiB, PKI_MAIN = 24 * MiB, PKI_PAR = 32 * MiB;
constexpr unsigned OFF_PKO = 65 * MiB, PKO_MAIN = 36 * MiB, PKO_PAR = 48 * MiB;
constexpr unsigned OFF_XS = 161 * MiB, XS_BYTES = 24 * MiB;
constexpr unsigned OFF_WS = 185 * MiB, WS_BYTES = 24 * MiB;
constexpr unsigned WS_NEED = 209 * MiB;
constexpr unsigned IN_BYTES = 16 * MiB;
constexpr unsigned OOB = 0x80000000u;
constexpr unsigned long long DEADLINE = 30000000ull;

constexpr int LDS_STAGE = 131072, LDS_BYTES = LDS_STAGE + 1024;
constexpr int NGEMM = 224;
#define NPW 5
#define RWG 10
#define RWV 4
constexpr int AUX_SYS = 17, AUX_AGENT = 16;

struct Params {
    const char* x; const char* w; char* out; char* ws[NR]; int rank;
};

__device__ __forceinline__ rsrc_t mk(const void* p, unsigned bytes) { return __builtin_amdgcn_make_buffer_rsrc((void*)p, 0, bytes, 0x00020000); }
__device__ __forceinline__ unsigned rfl(unsigned v) { return __builtin_amdgcn_readfirstlane(v); }
__device__ __forceinline__ unsigned rdl(unsigned v, unsigned l) { return __builtin_amdgcn_readlane(v, l); }
#define WAIT_V(n) asm volatile("s_waitcnt vmcnt(" #n ")" ::: "memory")
#define WAIT_L(n) asm volatile("s_waitcnt lgkmcnt(" #n ")" ::: "memory")
#define BAR __builtin_amdgcn_s_barrier()
#define SCHED __builtin_amdgcn_sched_barrier(0)

#define STAMP(k) do { } while (0)

__device__ __forceinline__ unsigned pkmax(unsigned a, unsigned b) {
    return __builtin_bit_cast(unsigned, __builtin_elementwise_max(__builtin_bit_cast(u16x2, a), __builtin_bit_cast(u16x2, b)));
}
__device__ __forceinline__ unsigned umax_(unsigned a, unsigned b) { return a > b ? a : b; }
__device__ __forceinline__ unsigned wave_umax(unsigned v) {
    v = umax_(v, (unsigned)__builtin_amdgcn_update_dpp(0, (int)v, 0xB1, 0xf, 0xf, true));
    v = umax_(v, (unsigned)__builtin_amdgcn_update_dpp(0, (int)v, 0x4E, 0xf, 0xf, true));
    v = umax_(v, (unsigned)__builtin_amdgcn_update_dpp(0, (int)v, 0x141, 0xf, 0xf, true));
    v = umax_(v, (unsigned)__builtin_amdgcn_update_dpp(0, (int)v, 0x140, 0xf, 0xf, true));
    return umax_(umax_(rdl(v, 0), rdl(v, 16)), umax_(rdl(v, 32), rdl(v, 48)));
}
__device__ __forceinline__ unsigned cvt_pk_bf16(float lo, float hi) {
    unsigned r; asm("v_cvt_pk_bf16_f32 %0, %1, %2" : "=v"(r) : "v"(lo), "v"(hi)); return r;
}

__device__ __forceinline__ void encode(const unsigned (&D)[16], int lane, u32x4 (&P)[4], unsigned& lo7, unsigned& m32, bool& esc) {
    unsigned mx = D[0] & 0x7FFF7FFFu;
#pragma unroll
    for (int i = 1; i < 16; ++i) mx = pkmax(mx, D[i] & 0x7FFF7FFFu);
    const unsigned base7 = wave_umax(umax_(mx & 0xFFFFu, mx >> 16)) >> 8;
    lo7 = base7 > 7 ? base7 - 7 : 0;
    const unsigned lo4 = lo7 * 0x01010101u;
    unsigned H[8], T[8], bad = 0;
#pragma unroll
    for (int i = 0; i < 8; ++i) {
        P[i >> 2][i & 3] = __builtin_amdgcn_perm(D[2 * i + 1], D[2 * i], 0x06040200u);
        H[i] = __builtin_amdgcn_perm(D[2 * i + 1], D[2 * i], 0x07050301u);
        T[i] = H[i] - lo4; bad |= T[i];
    }
    const unsigned long long bal = __ballot((bad & 0x78787878u) != 0);
    m32 = (unsigned)bal | (unsigned)(bal >> 32);
    esc = (m32 >> (lane & 31)) & 1;
#pragma unroll
    for (int i = 0; i < 4; ++i) {
        const unsigned n = T[i] | __builtin_amdgcn_alignbit(T[i + 4], T[i + 4], 4);
        P[2][i] = esc ? H[i] : n; P[3][i] = H[i + 4];
    }
}
__device__ __forceinline__ void decode_h(const u32x4 (&P)[4], unsigned lo7, bool esc, unsigned (&H)[8]) {
    const unsigned lo4 = lo7 * 0x01010101u;
#pragma unroll
    for (int i = 0; i < 4; ++i) {
        const unsigned a = (P[2][i] & 0x87878787u) + lo4, b = (__builtin_amdgcn_alignbit(P[2][i], P[2][i], 28) & 0x87878787u) + lo4;
        H[i] = esc ? P[2][i] : a; H[i + 4] = esc ? P[3][i] : b;
    }
}
__device__ __forceinline__ void store_chunk(rsrc_t rs, unsigned main_off, unsigned esc_off, int lane, const u32x4 (&P)[4], bool esc) {
#pragma unroll
    for (int i = 0; i < 3; ++i) __builtin_amdgcn_raw_buffer_store_b128(P[i], rs, main_off + lane * 16 + i * 1024, 0, AUX_SYS);
    __builtin_amdgcn_raw_buffer_store_b128(P[3], rs, esc ? esc_off + lane * 16 : OOB, 0, AUX_SYS);
}
__device__ __forceinline__ void load_chunk(rsrc_t rs, unsigned main_off, unsigned esc_off, int lane, u32x4 (&P)[4], bool esc) {
#pragma unroll
    for (int i = 0; i < 3; ++i) P[i] = __builtin_amdgcn_raw_buffer_load_b128(rs, lane * 16 + i * 1024, main_off, AUX_SYS);
    P[3] = __builtin_amdgcn_raw_buffer_load_b128(rs, esc ? lane * 16 : OOB, esc_off, AUX_SYS);
}

__device__ __forceinline__ bool spin_fail(char* myws, unsigned call, unsigned long long t0) {
    unsigned* tmo = (unsigned*)(myws + OFF_TMO);
    if (rfl(__hip_atomic_load(tmo, __ATOMIC_RELAXED, __HIP_MEMORY_SCOPE_AGENT)) == call || __builtin_amdgcn_s_memrealtime() - t0 > DEADLINE) {
        __hip_atomic_store(tmo, call, __ATOMIC_RELAXED, __HIP_MEMORY_SCOPE_AGENT);
        return true;
    }
    __builtin_amdgcn_s_sleep(4);
    return false;
}
__device__ __forceinline__ bool timed_out(char* myws, unsigned call) {
    return rfl(__hip_atomic_load((unsigned*)(myws + OFF_TMO), __ATOMIC_RELAXED, __HIP_MEMORY_SCOPE_AGENT)) == call;
}

struct PackJob { unsigned src_off, stride, gc, rec_idx; int mtx, kk, hi; };
__device__ __forceinline__ PackJob pack_job(unsigned G, int wid) {
    PackJob j; const unsigned q = G >> 8, e = G & 255, tile = (e >> 1) & 31, ch = (e & 1) * 8 + wid;
    j.mtx = e >> 7; j.kk = (e >> 6) & 1; const unsigned ks = j.kk * 4 + q;
    unsigned t16;
    if (j.mtx) { j.src_off = (tile * 256 + ch * 16) * 2048 + ks * 256; j.stride = 2048; j.hi = (tile >> 1) & 1; t16 = (tile >> 2) * 2 + (tile & 1); }
    else { j.src_off = (ks * 128 + (ch >> 1) * 16) * 16384 + tile * 512 + (ch & 1) * 256; j.stride = 16384; j.hi = tile >> 4; t16 = tile & 15; }
    j.gc = (j.mtx * 256 + tile * 8 + ks) * 16 + ch;
    j.rec_idx = (j.mtx * 64 + t16 * 4 + q) * 16 + (e & 1) * 8;
    return j;
}
__device__ __forceinline__ void pack_load(const Params& p, unsigned G, int wid, int lane, u32x4 (&R)[4]) {
    const PackJob j = pack_job(G < 1024 ? G : 0, wid);
    const rsrc_t rs = mk(j.mtx ? p.x : p.w, G < 1024 ? IN_BYTES : 0);
    const unsigned vo = (lane >> 4) * j.stride + (lane & 15) * 16;
#pragma unroll
    for (int i = 0; i < 4; ++i) R[i] = __builtin_amdgcn_raw_buffer_load_b128(rs, vo, j.src_off + i * 4 * j.stride, 0);
}

template <int NS>
__device__ __forceinline__ void reduce_jobs(const Params& p, char* myws, unsigned call, unsigned tag, int lane, int quad,
                                            unsigned n0, unsigned nstep, unsigned njobs) {
    constexpr int LS = NS == 2 ? 32 : 8;
    const unsigned par = call & 1;
    const unsigned pr = ((unsigned)p.rank >> 1) ^ (unsigned)quad;
    const unsigned ls = lane / LS, lk = lane % LS;
    const unsigned sidx = NS == 2 ? 2 * quad + ls : (ls == 0 ? 1 : 7 + ls);
    const unsigned rec_off = (lk < njobs && ls < NS) ? OFF_RECO + ((par * 12 + sidx) * 1024 + n0 + nstep * lk) * 8 : OOB;
    const rsrc_t rctl = mk(myws, CTL_BYTES), rout = mk(p.out, IN_BYTES);
    rsrc_t rsrc[2];
    rsrc[0] = mk((NS == 2 ? p.ws[(p.rank ^ (2 * quad)) & 7] : myws) + OFF_PKO + par * PKO_PAR, PKO_PAR);
    rsrc[1] = mk((NS == 2 ? p.ws[(p.rank ^ (2 * quad + 1)) & 7] : myws) + OFF_PKO + par * PKO_PAR, PKO_PAR);
    u32x2 rec = {0, 0};
    const int fr = lane & 15, fq = lane >> 4;
    const unsigned vo = fr * 16384 + ((fq & 1) * 16 + (fq >> 1) * 8) * 2;
    bool dead = false;
    for (unsigned k = 0; k < njobs; ++k) {
        unsigned long long need = 0;
#pragma unroll
        for (int s = 0; s < NS; ++s) need |= 1ull << (s * LS + k);
        if ((__ballot((rec[1] >> 8) == tag) & need) != need && !dead) {
            const unsigned long long t0 = __builtin_amdgcn_s_memrealtime();
            for (;;) {
                rec = __builtin_amdgcn_raw_buffer_load_b64(rctl, rec_off, 0, AUX_SYS);
                if ((__ballot((rec[1] >> 8) == tag) & need) == need) break;
                if (spin_fail(myws, call, t0)) { dead = true; break; }
            }
        }
        asm volatile("" ::: "memory");
        const unsigned n = n0 + nstep * k, tile = n >> 5, t = tile >> 4, ct = tile & 15, ch = n & 31;
        u32x4 P[NS][4]; unsigned lo7[NS]; bool esc[NS];
#pragma unroll
        for (int s = 0; s < NS; ++s) {
            const unsigned m32 = rdl(rec[0], s * LS + k); lo7[s] = rdl(rec[1], s * LS + k) & 255; esc[s] = (m32 >> (lane & 31)) & 1;
            const unsigned slot = NS == 2 ? (t * 8 + 2 * quad + s) * 16 + ct : (s == 0 ? (t * 8 + 1) * 16 + ct : 256 + (s - 1) * 32 + tile);
            const unsigned c = slot * 32 + ch;
            load_chunk(rsrc[s & 1], dead ? OOB : c * 3072, dead ? OOB : PKO_MAIN + c * 1024, lane, P[s], esc[s]);
        }
        float v[32];
#pragma unroll
        for (int s = 0; s < NS; ++s) {
            unsigned H[8]; decode_h(P[s], lo7[s], esc[s], H);
#pragma unroll
            for (int i = 0; i < 8; ++i)
#pragma unroll
                for (int b = 0; b < 4; ++b) {
                    const float f = __builtin_bit_cast(float, __builtin_amdgcn_perm(H[i], P[s][i >> 2][i & 3], ((4u + b) << 24) | ((unsigned)b << 16) | 0x0c0cu));
                    v[4 * i + b] = s == 0 ? f : v[4 * i + b] + f;
                }
        }
        if (k + 1 == njobs && !dead) dead = timed_out(myws, call);
        const unsigned w = ch >> 2, u = ch & 3;
        const unsigned so = ((pr >> 1) * 512 + t * 256 + (u >> 1) * 128 + (w >> 2) * 64) * 16384 + ((pr & 1) * 4096 + ct * 256 + (u & 1) * 128 + (w & 3) * 32) * 2;
#pragma unroll
        for (int m = 0; m < 4; ++m) {
            unsigned d[4];
#pragma unroll
            for (int h = 0; h < 4; ++h) {
                const float a = v[8 * m + 2 * h], b = v[8 * m + 2 * h + 1];
                d[h] = cvt_pk_bf16(a < 0.f ? 0.f : a, b < 0.f ? 0.f : b);
            }
            const auto s0 = __builtin_amdgcn_permlane16_swap(d[0], d[2], false, false);
            const auto s1 = __builtin_amdgcn_permlane16_swap(d[1], d[3], false, false);
            u32x4 o = {s0[0], s1[0], s0[1], s1[1]};
            if (dead) o = (u32x4){0x7FC07FC0u, 0x7FC07FC0u, 0x7FC07FC0u, 0x7FC07FC0u};
            __builtin_amdgcn_raw_buffer_store_b128(o, rout, vo + so + m * 16 * 16384, 0, AUX_AGENT);
        }
    }
}

__global__ __launch_bounds__(512, 2) void gemm_rs_kernel(const Params p) {
    extern __shared__ __attribute__((aligned(16))) char shm[];
    LAS char* lds = (LAS char*)shm;
    const int tid = threadIdx.x, wid = rfl(tid >> 6), lane = tid & 63;
    const unsigned b = blockIdx.x, rank = p.rank & 7;
    char* const myws = p.ws[rank];
    STAMP(0);
    u32x4 R[4];
    pack_load(p, b, wid, lane, R);
    unsigned* callp = (unsigned*)(myws + OFF_CALL + b * 128);
    const unsigned call = rfl(__hip_atomic_load(callp, __ATOMIC_RELAXED, __HIP_MEMORY_SCOPE_AGENT)) + 1;
    const unsigned par = call & 1, tag = call % 0xFFFFFFu + 1;
    {
        const rsrc_t rpk = mk(myws + OFF_PKI + par * PKI_PAR, PKI_PAR);
        unsigned G = b, it = 0;
        while (G < 1024) {
            const unsigned Gn = b < NGEMM ? 256 + b + NGEMM * it : 1024;
            unsigned D[16];
#pragma unroll
            for (int i = 0; i < 16; ++i) D[i] = R[i >> 2][i & 3];
            u32x4 P[4]; unsigned lo7, m32; bool esc;
            encode(D, lane, P, lo7, m32, esc);
            const PackJob j = pack_job(G, wid);
            store_chunk(rpk, j.gc * 3072, PKI_MAIN + j.gc * 1024, lane, P, esc);
            SCHED;
            pack_load(p, Gn, wid, lane, R);
            SCHED;
            WAIT_V(4);
            LAS u32x2* lrec = (LAS u32x2*)(lds + LDS_STAGE + (it & 1) * 64);
            if (lane == 0) lrec[wid] = (u32x2){m32, (tag << 8) | lo7};
            WAIT_L(0); BAR;
            if (wid < 2) {
                const unsigned h = j.mtx ? j.hi * 4 + wid * 2 + j.kk : wid * 4 + j.hi * 2 + j.kk;
                if ((h >> 1) != (rank >> 1) && lane < 8) {
                    const u32x2 r = lrec[lane];
                    __builtin_amdgcn_raw_buffer_store_b64(r, mk(p.ws[h & 7], CTL_BYTES), lane * 8, OFF_RECI + ((par * 8 + rank) * 2048 + j.rec_idx) * 8, AUX_SYS);
                }
            }
            if (it == 0 && tid == 128) __hip_atomic_store(callp, call, __ATOMIC_RELAXED, __HIP_MEMORY_SCOPE_AGENT);
            G = Gn; ++it;
        }
    }
    STAMP(1);
    const unsigned gi = rank >> 2, gj = (rank >> 1) & 1, gk = rank & 1;
    if (b >= NGEMM) {
        const unsigned c = b - NGEMM;
        if (c < 6 * NPW) {
            constexpr unsigned NWL = NPW * 8;
            const unsigned si = c % 6, u = (c / 6) * 8 + wid, g = (rank ^ (si + 2)) & 7;
            const unsigned njobs = (2048 - u + NWL - 1) / NWL;
            unsigned v_rec, v_gc, v_dst, v_cnt;
            {
                const unsigned cidx = u + NWL * lane, pp = (cidx >> 4) & 127, ch = cidx & 15;
                unsigned q, mtx, t = 0, d = 0, ct = 0;
                if (pp < 120) { q = pp / 30; const unsigned e = pp % 30; mtx = e >= 16; if (mtx) { t = (e - 16) / 7; d = 1 + (e - 16) % 7; } else ct = e; }
                else { q = (pp - 120) >> 1; t = pp & 1; mtx = 1; }
                const unsigned sh = rank ^ d, ks = gk * 4 + q;
                const unsigned tile32 = mtx ? sh * 4 + gi * 2 + t : gj * 16 + ct, t16 = mtx ? sh * 2 + t : ct;
                v_gc = (mtx * 256 + tile32 * 8 + ks) * 16 + ch;
                v_rec = cidx < 2048 ? OFF_RECI + ((par * 8 + g) * 2048 + (mtx * 64 + t16 * 4 + q) * 16 + ch) * 8 : OOB;
                v_dst = mtx ? (si >> 1) * 8 * MiB + ((t * 8 + d) * 256 + ch * 16) * 2048 + (si & 1) * 1024 + q * 256
                            : XS_BYTES + ((si >> 1) * 512 + q * 128 + (ch >> 1) * 16) * 16384 + (si & 1) * 8192 + ct * 512 + (ch & 1) * 256;
                v_cnt = OFF_RDY + (mtx ? t * 8 + d : 16 + ct) * 128 + (q * 6 + si) * 4;
            }
            const rsrc_t rctl = mk(myws, CTL_BYTES), rsrc = mk(p.ws[g] + OFF_PKI + par * PKI_PAR, PKI_PAR), rdst = mk(myws + OFF_XS, XS_BYTES + WS_BYTES);
            const unsigned vox = (lane >> 4) * 2048 + (lane & 15) * 16, vow = (lane >> 4) * 16384 + (lane & 15) * 16;
            u32x2 rec = {0, 0};
            unsigned prev_cnt = 0; bool dead = false;
            for (unsigned k = 0; k < njobs; ++k) {
                if ((rdl(rec[1], k) >> 8) != tag) {
                    const unsigned long long t0 = __builtin_amdgcn_s_memrealtime();
                    for (;;) {
                        rec = __builtin_amdgcn_raw_buffer_load_b64(rctl, v_rec, 0, AUX_SYS);
                        if ((rdl(rec[1], k) >> 8) == tag) break;
                        if (spin_fail(myws, call, t0)) { dead = true; break; }
                    }
                    if (dead) break;
                }
                asm volatile("" ::: "memory");
                const unsigned m32 = rdl(rec[0], k), lo7 = rdl(rec[1], k) & 255, gc = rdl(v_gc, k), dst = rdl(v_dst, k);
                const bool esc = (m32 >> (lane & 31)) & 1;
                u32x4 P[4];
                load_chunk(rsrc, gc * 3072, PKI_MAIN + gc * 1024, lane, P, esc);
                WAIT_V(0);
                if (k && lane == 0) __hip_atomic_fetch_add((unsigned*)(myws + prev_cnt), 1u, __ATOMIC_RELAXED, __HIP_MEMORY_SCOPE_AGENT);
                prev_cnt = rdl(v_cnt, k);
                unsigned H[8]; decode_h(P, lo7, esc, H);
                const bool isx = dst < XS_BYTES;
                const unsigned vo = isx ? vox : vow, st4 = isx ? 4 * 2048 : 4 * 16384;
#pragma unroll
                for (int i = 0; i < 4; ++i) {
                    u32x4 o;
                    o[0] = __builtin_amdgcn_perm(H[2 * i], P[i >> 1][(2 * i) & 3], 0x05010400u);
                    o[1] = __builtin_amdgcn_perm(H[2 * i], P[i >> 1][(2 * i) & 3], 0x07030602u);
                    o[2] = __builtin_amdgcn_perm(H[2 * i + 1], P[i >> 1][(2 * i + 1) & 3], 0x05010400u);
                    o[3] = __builtin_amdgcn_perm(H[2 * i + 1], P[i >> 1][(2 * i + 1) & 3], 0x07030602u);
                    __builtin_amdgcn_raw_buffer_store_b128(o, rdst, vo + dst + i * st4, 0, AUX_AGENT);
                }
            }
            WAIT_V(0);
            if (!dead && lane == 0) __hip_atomic_fetch_add((unsigned*)(myws + prev_cnt), 1u, __ATOMIC_RELAXED, __HIP_MEMORY_SCOPE_AGENT);
        }
        STAMP(2);
        if (c < 3 * RWG && wid < RWV) {
            constexpr unsigned NWR = RWG * RWV;
            static_assert(NWR >= 32, "at most 32 jobs per wave");
            const unsigned u = (c / 3) * RWV + wid;
            reduce_jobs<2>(p, myws, call, tag, lane, 1 + c % 3, u, NWR, (1024 - u + NWR - 1) / NWR);
        }
        STAMP(3);
        return;
    }
    const unsigned z = b & 7, slot = b >> 3, ut = z & 1, uct = (z >> 1) * 4 + slot / 7, ud = 1 + slot % 7;
    const int wr = wid >> 2, wc = wid & 3, fr = lane & 15, fq = lane >> 4;
    const int nunits = (ud >= 2 && ud <= 5) ? 2 : 1;
    const unsigned voffA = (wid * 8 + (lane >> 3)) * 2048 + (((lane & 7) ^ ((wid * 4 + (lane >> 4)) & 7)) * 16);
    const unsigned voffB = (wid * 4 + (lane >> 4)) * 16384 + (((lane & 15) ^ (((lane >> 4) << 2) | (wid & 2))) * 16);
    const unsigned ldsw = wid * 1024;
    int aoff[2], boff[2];
#pragma unroll
    for (int k = 0; k < 2; ++k) aoff[k] = (wr * 64 + fr) * 128 + (((4 * k + fq) ^ ((fr >> 1) & 7)) * 16);
#pragma unroll
    for (int n = 0; n < 2; ++n) { const int q = (lane & 15) >> 2, pp = lane & 3;
        boff[n] = (int)(unsigned)(size_t)lds + 65536 + (8 * fq + q) * 256 + (((4 * wc + 2 * n + (pp >> 1)) ^ ((q << 2) | ((fq & 1) << 1))) * 16) + 8 * (pp & 1); }
    const rsrc_t rctl = mk(myws, CTL_BYTES);
#define HT 16384
#define SA(bb, h) (((bb) * 2 + (h)) * HT)
#define SB(bb, h) ((4 + (bb) * 2 + (h)) * HT)
#define STAGE_A(bb, h, rs, so) do { _Pragma("unroll") for (int _i = 0; _i < 2; ++_i) \
        __builtin_amdgcn_raw_ptr_buffer_load_lds(rs, (LAS void*)(lds + SA(bb, h) + ldsw + _i * 8192), 16, voffA, (so) + (h) * 262144 + _i * 131072, 0, AUX_AGENT); } while (0)
#define STAGE_B(bb, h, rs, so) do { _Pragma("unroll") for (int _i = 0; _i < 2; ++_i) \
        __builtin_amdgcn_raw_ptr_buffer_load_lds(rs, (LAS void*)(lds + SB(bb, h) + ldsw + _i * 8192), 16, voffB, (so) + (h) * 256 + _i * 524288, 0, AUX_AGENT); } while (0)
#define LDA(dst, bb, h) do { _Pragma("unroll") for (int m = 0; m < 4; ++m) _Pragma("unroll") for (int k = 0; k < 2; ++k) \
        dst[m][k] = *(const LAS bf16x8*)(lds + SA(bb, h) + aoff[k] + m * 2048); } while (0)
#define LDB(T, bb, h) asm volatile( \
        "ds_read_b64_tr_b16 %0, %8 offset:%10\n\tds_read_b64_tr_b16 %1, %8 offset:%11\n\tds_read_b64_tr_b16 %2, %8 offset:%12\n\tds_read_b64_tr_b16 %3, %8 offset:%13\n\t" \
        "ds_read_b64_tr_b16 %4, %9 offset:%10\n\tds_read_b64_tr_b16 %5, %9 offset:%11\n\tds_read_b64_tr_b16 %6, %9 offset:%12\n\tds_read_b64_tr_b16 %7, %9 offset:%13" \
        : "=&v"(T[0]), "=&v"(T[1]), "=&v"(T[2]), "=&v"(T[3]), "=&v"(T[4]), "=&v"(T[5]), "=&v"(T[6]), "=&v"(T[7]) \
        : "v"(boff[0]), "v"(boff[1]), "i"(SB(bb, h) - 65536), "i"(SB(bb, h) - 65536 + 1024), "i"(SB(bb, h) - 65536 + 8192), "i"(SB(bb, h) - 65536 + 9216) : "memory")
#define LDB_USE(dst, T) do { asm volatile("s_waitcnt lgkmcnt(0)" : "+v"(T[0]), "+v"(T[1]), "+v"(T[2]), "+v"(T[3]), "+v"(T[4]), "+v"(T[5]), "+v"(T[6]), "+v"(T[7]) :: "memory"); \
        _Pragma("unroll") for (int _j = 0; _j < 4; ++_j) dst[_j >> 1][_j & 1] = __builtin_shufflevector(T[2 * _j], T[2 * _j + 1], 0, 1, 2, 3, 4, 5, 6, 7); } while (0)
#define MMA(ai, bj, At, Bt) do { __builtin_amdgcn_s_setprio(1); _Pragma("unroll") for (int m = 0; m < 4; ++m) _Pragma("unroll") for (int n = 0; n < 2; ++n) _Pragma("unroll") for (int k = 0; k < 2; ++k) \
        acc[ai][bj][m][n] = __builtin_amdgcn_mfma_f32_16x16x32_bf16(Bt[n][k], At[m][k], acc[ai][bj][m][n], 0, 0, 0); __builtin_amdgcn_s_setprio(0); } while (0)
    constexpr unsigned AK = 128, BK = 1048576;
    for (int un = 0; un < nunits; ++un) {
        const unsigned d = un ? 0 : ud, s0 = un ? (ud - 2) * 8 : 0, s1 = un ? s0 + 8 : 32;
        const unsigned rt = ut * 8 + d;
        const unsigned aown = ((rank ^ d) * 1024 + gi * 512 + ut * 256) * 2048, bown = (gj * 4096 + uct * 256) * 2;
        const unsigned arem = rt * 256 * 2048, brem = uct * 512;
        const unsigned rdy_off = OFF_RDY + (lane < 32 ? rt : 16 + uct) * 128 + (lane & 31) * 4;
        unsigned ready = 0xFFu;
        auto step_a = [&](unsigned s) -> unsigned { const unsigned r = s - 8, q = (r * 43) >> 8, si = r - q * 6;
            return s < 8 ? aown + s * 256 : arem + (si >> 1) * 8 * MiB + (si & 1) * 1024 + q * 256; };
        auto step_b = [&](unsigned s) -> unsigned { const unsigned r = s - 8, q = (r * 43) >> 8, si = r - q * 6;
            return s < 8 ? bown + s * 128 * 16384 : brem + ((si >> 1) * 512 + q * 128) * 16384 + (si & 1) * 8192; };
        auto wait_step = [&](unsigned s) {
            if (!((ready >> s) & 1)) {
                const unsigned long long t0 = __builtin_amdgcn_s_memrealtime();
                for (;;) {
                    const unsigned v = __builtin_amdgcn_raw_buffer_load_b32(rctl, rdy_off, 0, AUX_AGENT);
                    const unsigned long long m = __ballot((int)(v - call * 16u) >= 0);
                    ready = (((unsigned)m & (unsigned)(m >> 32)) << 8) | 0xFFu;
                    if ((ready >> s) & 1) break;
                    if (spin_fail(myws, call, t0)) { ready = 0xFFFFFFFFu; break; }
                }
                asm volatile("" ::: "memory");
            }
        };
        f32x4 acc[2][2][4][2];
#pragma unroll
        for (int a = 0; a < 2; ++a)
#pragma unroll
            for (int bb = 0; bb < 2; ++bb)
#pragma unroll
                for (int m = 0; m < 4; ++m)
#pragma unroll
                    for (int n = 0; n < 2; ++n) acc[a][bb][m][n] = (f32x4){0.f, 0.f, 0.f, 0.f};
        bf16x8 At[4][2], B0[2][2], B1[2][2]; s16x4 TB[8];
        wait_step(s0);
        rsrc_t ra = mk(s0 < 8 ? p.x : myws + OFF_XS, s0 < 8 ? IN_BYTES : XS_BYTES);
        unsigned sa = step_a(s0);
        {
            const rsrc_t rb = mk(s0 < 8 ? p.w : myws + OFF_WS, s0 < 8 ? IN_BYTES : WS_BYTES);
            const unsigned sb = step_b(s0);
            STAGE_B(0, 0, rb, sb); STAGE_A(0, 0, ra, sa); STAGE_B(0, 1, rb, sb); STAGE_A(0, 1, ra, sa);
            if (wr == 1) BAR;
            WAIT_V(4); BAR;
            STAGE_B(1, 0, rb, sb + BK); STAGE_A(1, 0, ra, sa + AK); STAGE_B(1, 1, rb, sb + BK);
            WAIT_V(6); BAR;
        }
        for (unsigned s = s0; s < s1; ++s) {
            const unsigned sn = s + 1 < s1 ? s + 1 : s;
            wait_step(sn);
            const rsrc_t rna = mk(sn < 8 ? p.x : myws + OFF_XS, sn < 8 ? IN_BYTES : XS_BYTES);
            const rsrc_t rnb = mk(sn < 8 ? p.w : myws + OFF_WS, sn < 8 ? IN_BYTES : WS_BYTES);
            const unsigned na = step_a(sn), nb = step_b(sn);
            LDB(TB, 0, 0); SCHED; LDA(At, 0, 0); STAGE_A(1, 1, ra, sa + AK);
            WAIT_L(8); BAR; LDB_USE(B0, TB); MMA(0, 0, At, B0); BAR; SCHED;
            LDB(TB, 0, 1); STAGE_B(0, 0, rnb, nb);
            BAR; LDB_USE(B1, TB); MMA(0, 1, At, B1); BAR;
            LDA(At, 0, 1); STAGE_A(0, 0, rna, na);
            BAR; WAIT_L(0); MMA(1, 0, At, B0); BAR; SCHED;
            STAGE_B(0, 1, rnb, nb);
            WAIT_V(6); BAR; MMA(1, 1, At, B1); BAR;
            LDB(TB, 1, 0); SCHED; LDA(At, 1, 0); STAGE_A(0, 1, rna, na);
            WAIT_L(8); BAR; LDB_USE(B0, TB); MMA(0, 0, At, B0); BAR; SCHED;
            LDB(TB, 1, 1); STAGE_B(1, 0, rnb, nb + BK);
            BAR; LDB_USE(B1, TB); MMA(0, 1, At, B1); BAR;
            LDA(At, 1, 1); STAGE_A(1, 0, rna, na + AK);
            BAR; WAIT_L(0); MMA(1, 0, At, B0); BAR; SCHED;
            STAGE_B(1, 1, rnb, nb + BK);
            WAIT_V(6); BAR; MMA(1, 1, At, B1); BAR;
            ra = rna; sa = na;
        }
        STAMP(4 + 2 * un);
        {
            const unsigned dst = (rank ^ d) & 7;
            const unsigned pslot = un ? 256 + (ud - 2) * 32 + ut * 16 + uct : rt * 16 + uct;
            const rsrc_t rpk = mk((d == 1 ? p.ws[dst] : myws) + OFF_PKO + par * PKO_PAR, PKO_PAR);
            u32x2 rec = {0, 0};
#pragma unroll
            for (int u = 0; u < 4; ++u) {
                unsigned D[16];
#pragma unroll
                for (int m = 0; m < 4; ++m)
#pragma unroll
                    for (int n = 0; n < 2; ++n) {
                        const f32x4 a = acc[u >> 1][u & 1][m][n];
                        D[m * 4 + n * 2] = cvt_pk_bf16(a[0], a[1]); D[m * 4 + n * 2 + 1] = cvt_pk_bf16(a[2], a[3]);
                    }
                u32x4 P[4]; unsigned lo7, m32; bool esc;
                encode(D, lane, P, lo7, m32, esc);
                const unsigned c = pslot * 32 + wid * 4 + u;
                store_chunk(rpk, c * 3072, PKO_MAIN + c * 1024, lane, P, esc);
                if (lane == u) rec = (u32x2){m32, (tag << 8) | lo7};
            }
            WAIT_V(0);
            const unsigned sidx = un ? 8 + (ud - 2) : d;
            if (!timed_out(myws, call) && lane < 4)
                __builtin_amdgcn_raw_buffer_store_b64(rec, mk(p.ws[dst], CTL_BYTES), lane * 8, OFF_RECO + ((par * 12 + sidx) * 1024 + (ut * 16 + uct) * 32 + wid * 4) * 8, AUX_SYS);
        }
        if (wr == 0) BAR;
        BAR;
        STAMP(5 + 2 * un);
    }
    if (ud >= 6) reduce_jobs<5>(p, myws, call, tag, lane, 0, (ut * 16 + uct) * 32 + (ud - 6) * 16 + wid * 2, 1, 2);
    STAMP(3);
}

}

extern "C" void kernel_launch(int rank, int nranks, ncclComm_t comm, hipStream_t stream,
                              void* const* d_in, const int* in_sizes, int n_in,
                              void* d_out, int out_size,
                              const int* in_dtypes, int out_dtype,
                              void* const* d_out_all,
                              void* const* d_ws_all, size_t ws_bytes) {
    (void)comm; (void)in_dtypes; (void)out_dtype; (void)d_out_all;
    if (nranks != NR || (unsigned)rank >= (unsigned)NR || n_in < 2 || ws_bytes < WS_NEED || out_size != 1024 * 8192 ||
        in_sizes[0] != 8192 * 1024 || in_sizes[1] != 1024 * 8192) {
        fprintf(stderr, "kernel_launch: unexpected problem geometry\n");
        return;
    }
    HIP_CHECK(hipFuncSetAttribute((const void*)gemm_rs_kernel, hipFuncAttributeMaxDynamicSharedMemorySize, LDS_BYTES));
    Params p;
    p.x = (const char*)d_in[0]; p.w = (const char*)d_in[1]; p.out = (char*)d_out; p.rank = rank;
    for (int r = 0; r < NR; ++r) p.ws[r] = (char*)d_ws_all[r];
    hipLaunchKernelGGL(gemm_rs_kernel, dim3(256), dim3(512), LDS_BYTES, stream, p);
    HIP_CHECK(hipGetLastError());
}
```

```cpp
#include <hip/hip_runtime.h>
#include <rccl/rccl.h>
#include <cstdio>

#define HIP_CHECK(expr) do { hipError_t _e = (expr); if (_e != hipSuccess) { \
    fprintf(stderr, "HIP error %s at %s:%d\n", hipGetErrorString(_e), __FILE__, __LINE__); \
    return; } } while (0)

namespace {

#define LAS __attribute__((address_space(3)))
typedef short bf16x8 __attribute__((ext_vector_type(8)));
typedef short s16x4 __attribute__((ext_vector_type(4)));
typedef float f32x4 __attribute__((ext_vector_type(4)));
typedef unsigned u32x4 __attribute__((ext_vector_type(4)));
typedef unsigned u32x2 __attribute__((ext_vector_type(2)));
typedef unsigned short u16x2 __attribute__((ext_vector_type(2)));
typedef __amdgpu_buffer_rsrc_t rsrc_t;

constexpr int NR = 8;
constexpr unsigned MiB = 1u << 20;
constexpr unsigned OFF_CALL = 0;
constexpr unsigned OFF_TMO = 32768;
constexpr unsigned OFF_RDY = 36864;
constexpr unsigned OFF_TKT = 40960;
constexpr unsigned OFF_RECI = 65536;
constexpr unsigned OFF_RECO = 327680;
constexpr unsigned OFF_DBG = 524288;
constexpr unsigned CTL_BYTES = MiB;
constexpr unsigned OFF_PKI = 1 * MiB, PKI_MAIN = 24 * MiB, PKI_PAR = 32 * MiB;
constexpr unsigned OFF_PKO = 65 * MiB, PKO_MAIN = 36 * MiB, PKO_PAR = 48 * MiB;
constexpr unsigned OFF_XS = 161 * MiB, XS_BYTES = 24 * MiB;
constexpr unsigned OFF_WS = 185 * MiB, WS_BYTES = 24 * MiB;
constexpr unsigned WS_NEED = 209 * MiB;
constexpr unsigned IN_BYTES = 16 * MiB;
constexpr unsigned OOB = 0x80000000u;
constexpr unsigned long long DEADLINE = 30000000ull;

constexpr int LDS_STAGE = 131072, LDS_BYTES = LDS_STAGE + 1024 + 8 * 3104;
constexpr int NGEMM = 224;
#define NPW 5
#define PWV 6
#define PDEPTH 1
#define RDEPTH 1
#define RWG 9
#define RWV 3
#define CWG 5
#define CWV 6
static_assert(3 * RWG + CWG <= 32 && 6 * NPW <= 32, "32 communication workgroups");
#define SLEEP_G 8
#define SLEEP_S 4
#define SLEEP_O 32
constexpr int AUX_SYS = 17, AUX_AGENT = 16;

#define SLOPE 28
#define GROUPT 156
struct Order { unsigned char e[128]; };
constexpr Order make_order() {
    Order o{}; int key[128] = {}; int n = 0;
    for (int q = 0; q < 4; ++q) {
        for (int d = 1; d < 8; ++d) for (int t = 0; t < 2; ++t) { o.e[n] = (unsigned char)(64 | q << 4 | (t * 8 + d)); key[n++] = q * GROUPT - 1; }
        for (int c = 0; c < 16; ++c) { o.e[n] = (unsigned char)(q << 4 | c); key[n++] = q * GROUPT + c * SLOPE; }
        for (int t = 0; t < 2; ++t) { o.e[n] = (unsigned char)(64 | q << 4 | (t * 8)); key[n++] = 100000 + q; }
    }
    for (int i = 1; i < 128; ++i)
        for (int j = i; j > 0 && key[j - 1] > key[j]; --j) {
            const int k = key[j]; key[j] = key[j - 1]; key[j - 1] = k; const unsigned char e = o.e[j]; o.e[j] = o.e[j - 1]; o.e[j - 1] = e; }
    return o;
}

struct Params {
    const char* x; const char* w; char* out; char* ws[NR]; int rank; Order order;
};

__device__ __forceinline__ rsrc_t mk(const void* p, unsigned bytes) { return __builtin_amdgcn_make_buffer_rsrc((void*)p, 0, bytes, 0x00020000); }
__device__ __forceinline__ unsigned rfl(unsigned v) { return __builtin_amdgcn_readfirstlane(v); }
__device__ __forceinline__ unsigned rdl(unsigned v, unsigned l) { return __builtin_amdgcn_readlane(v, l); }
#define WAIT_V(n) asm volatile("s_waitcnt vmcnt(" #n ")" ::: "memory")
#define WAIT_L(n) asm volatile("s_waitcnt lgkmcnt(" #n ")" ::: "memory")
#define BAR __builtin_amdgcn_s_barrier()
#define SCHED __builtin_amdgcn_sched_barrier(0)

#define STAMP(k) do { } while (0)

__device__ __forceinline__ unsigned cvt_pk_bf16(float lo, float hi) {
    unsigned r; asm("v_cvt_pk_bf16_f32 %0, %1, %2" : "=v"(r) : "v"(lo), "v"(hi)); return r;
}
__device__ __forceinline__ unsigned pk_add(unsigned a, unsigned b) { return __builtin_bit_cast(unsigned, __builtin_bit_cast(u16x2, a) + __builtin_bit_cast(u16x2, b)); }
__device__ __forceinline__ unsigned pk_sub(unsigned a, unsigned b) { return __builtin_bit_cast(unsigned, __builtin_bit_cast(u16x2, a) - __builtin_bit_cast(u16x2, b)); }
__device__ __forceinline__ unsigned bfi(unsigned m, unsigned a, unsigned b) { return (a & m) | (b & ~m); }
__device__ __forceinline__ unsigned rotr(unsigned a, unsigned n) { return __builtin_amdgcn_alignbit(a, a, n); }
template <int CTRL> __device__ __forceinline__ float dpp_f(float v) { return __builtin_bit_cast(float, __builtin_amdgcn_update_dpp(0, __builtin_bit_cast(int, v), CTRL, 0xf, 0xf, true)); }
__device__ __forceinline__ float rdl_f(float v, unsigned l) { return __builtin_bit_cast(float, rdl(__builtin_bit_cast(unsigned, v), l)); }

constexpr unsigned CAP = 128, RAW = 255, CW_BYTES = 3104;
__device__ __forceinline__ unsigned plane2_bytes(unsigned r0) { return 768 + 2 * (r0 >> 16); }
__device__ __forceinline__ unsigned encode(const unsigned (&D)[16], LAS char* cw, int lane, u32x4 (&P)[4]) {
    float sm = 0.f;
#pragma unroll
    for (int i = 0; i < 16; i += 4) sm += __builtin_fabsf(__builtin_bit_cast(float, D[i] << 16)) + __builtin_fabsf(__builtin_bit_cast(float, D[i] & 0xFFFF0000u));
    sm += dpp_f<0xB1>(sm); sm += dpp_f<0x4E>(sm); sm += dpp_f<0x141>(sm); sm += dpp_f<0x140>(sm);
    const float top = ((rdl_f(sm, 0) + rdl_f(sm, 16)) + (rdl_f(sm, 32) + rdl_f(sm, 48))) * (3.54f / 512.f);
    const unsigned t15 = rfl(__builtin_bit_cast(unsigned, top) >> 16) & 0x7FFFu, lo = t15 > 1024 ? t15 - 1024 : 0, lo2 = lo * 0x10001u;
    unsigned H[8], M = 0;
#pragma unroll
    for (int i = 0; i < 8; ++i) {
        const unsigned a = pk_sub(D[2 * i], lo2), b = pk_sub(D[2 * i + 1], lo2);
        P[i >> 2][i & 3] = __builtin_amdgcn_perm(b, a, 0x06040200u);
        H[i] = __builtin_amdgcn_perm(b, a, 0x07050301u);
        M = bfi(0x80808080u >> i, ((H[i] & 0x7C7C7C7Cu) + 0x7C7C7C7Cu) >> i, M);
    }
    *(LAS u32x4*)(cw + lane * 16) = (u32x4){H[0], H[1], H[2], H[3]};
    *(LAS u32x4*)(cw + 1024 + lane * 16) = (u32x4){H[4], H[5], H[6], H[7]};
    LAS unsigned* w = (LAS unsigned*)(cw + 2048) + lane;
    w[0] = bfi(0xC0C0C0C0u, H[6] << 6, bfi(0x38383838u, rotr(H[1], 28), rotr(H[0], 31)));
    w[64] = bfi(0xC0C0C0C0u, H[7] << 6, bfi(0x38383838u, rotr(H[3], 28), rotr(H[2], 31)));
    w[128] = bfi(0x80808080u, H[6], bfi(0x40404040u, H[7] >> 1, bfi(0x38383838u, rotr(H[5], 28), rotr(H[4], 31))));
    LAS unsigned short* list = (LAS unsigned short*)(cw + 2816);
    unsigned n = 0;
    for (;;) {
        const unsigned long long act = __ballot(M != 0);
        if (!act || n > CAP) break;
        if (M) {
            const unsigned off = n + __builtin_amdgcn_mbcnt_hi((unsigned)(act >> 32), __builtin_amdgcn_mbcnt_lo((unsigned)act, 0));
            const unsigned bt = __builtin_ctz(M), i = 7 - (bt & 7);
            list[off < CAP ? off : CAP] = (unsigned short)((i >> 2) * 1024 + lane * 16 + (i & 3) * 4 + (bt >> 3));
            M &= M - 1;
        }
        n += __builtin_popcountll(act);
    }
    P[3] = (u32x4){H[4], H[5], H[6], H[7]};
    if (n > CAP) { P[2] = (u32x4){H[0], H[1], H[2], H[3]}; return lo | RAW << 16; }
    asm volatile("" ::: "memory");
    const unsigned e = w[192] & 0x07FF07FFu;
    const unsigned h0 = *(LAS unsigned char*)(cw + (e & 0xFFFFu)), h1 = *(LAS unsigned char*)(cw + (e >> 16));
    w[192] = e | (h0 & 0x7Cu) << 9 | (h1 & 0x7Cu) << 25;
    asm volatile("" ::: "memory");
    P[2] = *(LAS u32x4*)(cw + 2048 + lane * 16);
    asm volatile("" ::: "memory");
    return lo | n << 16;
}
__device__ __forceinline__ void decode(const u32x4 (&P)[4], unsigned r0, LAS char* cw, int lane, unsigned (&D)[16]) {
    const unsigned n = r0 >> 16, lo2 = (r0 & 0xFFFFu) * 0x10001u;
    unsigned H[8];
    if (n == RAW) {
#pragma unroll
        for (int i = 0; i < 4; ++i) { H[i] = P[2][i]; H[i + 4] = P[3][i]; }
    } else {
        *(LAS u32x4*)(cw + 2048 + lane * 16) = P[2];
        *(LAS u32x4*)(cw + lane * 16) = (u32x4){0, 0, 0, 0};
        *(LAS u32x4*)(cw + 1024 + lane * 16) = (u32x4){0, 0, 0, 0};
        asm volatile("" ::: "memory");
        const LAS unsigned* w = (const LAS unsigned*)(cw + 2048) + lane;
        const unsigned w0 = w[0], w1 = w[64], w2 = w[128], e = w[192];
        *(LAS unsigned char*)(cw + (2u * lane < n ? e & 0x7FFu : 3072u)) = (unsigned char)((e >> 9) & 0x7Cu);
        *(LAS unsigned char*)(cw + (2u * lane + 1 < n ? (e >> 16) & 0x7FFu : 3072u)) = (unsigned char)((e >> 25) & 0x7Cu);
        asm volatile("" ::: "memory");
        const u32x4 x0 = *(LAS u32x4*)(cw + lane * 16), x1 = *(LAS u32x4*)(cw + 1024 + lane * 16);
        asm volatile("" ::: "memory");
        H[0] = (rotr(w0, 1) & 0x83838383u) | x0[0]; H[1] = (rotr(w0, 4) & 0x83838383u) | x0[1];
        H[2] = (rotr(w1, 1) & 0x83838383u) | x0[2]; H[3] = (rotr(w1, 4) & 0x83838383u) | x0[3];
        H[4] = (rotr(w2, 1) & 0x83838383u) | x1[0]; H[5] = (rotr(w2, 4) & 0x83838383u) | x1[1];
        H[6] = ((w0 >> 6) & 0x03030303u) | (w2 & 0x80808080u) | x1[2];
        H[7] = ((w1 >> 6) & 0x03030303u) | ((w2 << 1) & 0x80808080u) | x1[3];
    }
#pragma unroll
    for (int i = 0; i < 8; ++i) {
        D[2 * i] = pk_add(__builtin_amdgcn_perm(H[i], P[i >> 2][i & 3], 0x05010400u), lo2);
        D[2 * i + 1] = pk_add(__builtin_amdgcn_perm(H[i], P[i >> 2][i & 3], 0x07030602u), lo2);
    }
}
__device__ __forceinline__ void store_chunk(rsrc_t rs, unsigned main_off, unsigned esc_off, int lane, const u32x4 (&P)[4], unsigned r0) {
#pragma unroll
    for (int i = 0; i < 2; ++i) __builtin_amdgcn_raw_buffer_store_b128(P[i], rs, main_off + lane * 16 + i * 1024, 0, AUX_SYS);
    __builtin_amdgcn_raw_buffer_store_b128(P[2], rs, lane * 16u < plane2_bytes(r0) ? main_off + lane * 16 + 2048 : OOB, 0, AUX_SYS);
    __builtin_amdgcn_raw_buffer_store_b128(P[3], rs, (r0 >> 16) == RAW ? esc_off + lane * 16 : OOB, 0, AUX_SYS);
}
__device__ __forceinline__ void load_chunk(rsrc_t rs, unsigned main_off, unsigned esc_off, int lane, u32x4 (&P)[4], unsigned r0) {
#pragma unroll
    for (int i = 0; i < 2; ++i) P[i] = __builtin_amdgcn_raw_buffer_load_b128(rs, lane * 16 + i * 1024, main_off, AUX_SYS);
    P[2] = __builtin_amdgcn_raw_buffer_load_b128(rs, lane * 16u < plane2_bytes(r0) ? lane * 16 + 2048 : OOB, main_off, AUX_SYS);
    P[3] = __builtin_amdgcn_raw_buffer_load_b128(rs, lane * 16, (r0 >> 16) == RAW ? esc_off : OOB, AUX_SYS);
}

template <int SLEEP>
__device__ __forceinline__ bool spin_fail(char* myws, unsigned call, unsigned long long t0, unsigned& n) {
    unsigned* tmo = (unsigned*)(myws + OFF_TMO);
    if ((++n & 63) == 0 &&
        (rfl(__hip_atomic_load(tmo, __ATOMIC_RELAXED, __HIP_MEMORY_SCOPE_AGENT)) == call || __builtin_amdgcn_s_memrealtime() - t0 > DEADLINE)) {
        __hip_atomic_store(tmo, call, __ATOMIC_RELAXED, __HIP_MEMORY_SCOPE_AGENT);
        return true;
    }
    __builtin_amdgcn_s_sleep(SLEEP);
    return false;
}
__device__ __forceinline__ bool timed_out(char* myws, unsigned call) {
    return rfl(__hip_atomic_load((unsigned*)(myws + OFF_TMO), __ATOMIC_RELAXED, __HIP_MEMORY_SCOPE_AGENT)) == call;
}

struct PackJob { unsigned src_off, stride, gc, rec_idx; int mtx, kk, hi; };
__device__ __forceinline__ PackJob pack_job(unsigned G, int wid) {
    PackJob j; const unsigned q = G >> 8, e = G & 255, tile = (e >> 1) & 31, ch = (e & 1) * 8 + wid;
    j.mtx = e >> 7; j.kk = (e >> 6) & 1; const unsigned ks = j.kk * 4 + q;
    unsigned t16;
    if (j.mtx) { j.src_off = (tile * 256 + ch * 16) * 2048 + ks * 256; j.stride = 2048; j.hi = (tile >> 1) & 1; t16 = (tile >> 2) * 2 + (tile & 1); }
    else { j.src_off = (ks * 128 + (ch >> 1) * 16) * 16384 + tile * 512 + (ch & 1) * 256; j.stride = 16384; j.hi = tile >> 4; t16 = tile & 15; }
    j.gc = (j.mtx * 256 + tile * 8 + ks) * 16 + ch;
    j.rec_idx = (j.mtx * 64 + t16 * 4 + q) * 16 + (e & 1) * 8;
    return j;
}
__device__ __forceinline__ void pack_load(const Params& p, unsigned G, int wid, int lane, u32x4 (&R)[4]) {
    const PackJob j = pack_job(G < 1024 ? G : 0, wid);
    const rsrc_t rs = mk(j.mtx ? p.x : p.w, G < 1024 ? IN_BYTES : 0);
    const unsigned vo = (lane >> 4) * j.stride + (lane & 15) * 16;
#pragma unroll
    for (int i = 0; i < 4; ++i) R[i] = __builtin_amdgcn_raw_buffer_load_b128(rs, vo, j.src_off + i * 4 * j.stride, 0);
}

template <int NS> struct RBuf { u32x4 P[NS][4]; unsigned r0[NS]; };
template <int NS, int DEPTH, class J>
__device__ __forceinline__ void run_stream(J& job, char* myws, unsigned call, int lane, unsigned u, unsigned NW, unsigned N, unsigned ctr) {
    const rsrc_t rctl = mk(myws, CTL_BYTES);
    const unsigned tk_off = lane == 0 ? OFF_TKT + ctr * 128 : OOB, tk_base = (call - 1) * (N + NW) - (DEPTH + 1) * NW;
    bool dead = false;
    auto rec_off = [&](unsigned idx) -> unsigned { return (idx < N && lane < NS) ? job.rec(idx, lane) : OOB; };
    auto check = [&](unsigned idx, u32x2 rec, RBuf<NS>& B) {
        const bool live = idx < N;
        constexpr unsigned need = (1u << NS) - 1;
        if (live && !dead && ((unsigned)__ballot(rec[1] == call) & need) != need) {
            const unsigned long long t0 = __builtin_amdgcn_s_memrealtime();
            const unsigned ro = rec_off(idx);
            for (unsigned sp = 0;;) {
                rec = __builtin_amdgcn_raw_buffer_load_b64(rctl, ro, 0, AUX_SYS);
                if (((unsigned)__ballot(rec[1] == call) & need) == need) break;
                if (spin_fail<SLEEP_S>(myws, call, t0, sp)) { dead = true; break; }
            }
        }
        if (J::WRITES_OUT && !live && !dead) dead = timed_out(myws, call);
        asm volatile("" ::: "memory");
#pragma unroll
        for (int s = 0; s < NS; ++s) B.r0[s] = rdl(rec[0], s);
    };
    auto issue = [&](unsigned idx, RBuf<NS>& B) {
        const bool off = dead || idx >= N;
#pragma unroll
        for (int s = 0; s < NS; ++s) {
            const unsigned c = job.src_chunk(idx, s);
            load_chunk(job.src(s), off ? OOB : c * 3072, off ? OOB : J::ESC_BASE + c * 1024, lane, B.P[s], B.r0[s]);
        }
    };
    auto pad = [&]() {
#pragma unroll
        for (int i = 0; i < 4; ++i) __builtin_amdgcn_raw_buffer_store_b32(0, rctl, OOB + i * 4, 0, 0);
    };
    RBuf<NS> A, B, C;
    unsigned ip = N, p0 = 0, iq = N, q0 = 0;
    unsigned i0 = u, i1 = u + NW, i2 = u + 2 * NW;
    check(i0, (u32x2){0, 0}, A);
    if (DEPTH == 2) { issue(i0, A); SCHED; pad(); check(i1, (u32x2){0, 0}, B); }
    u32x2 rec = __builtin_amdgcn_raw_buffer_load_b64(rctl, rec_off(DEPTH == 2 ? i2 : i1), 0, AUX_SYS);
    unsigned tk = __builtin_amdgcn_raw_ptr_buffer_atomic_add_i32(1, rctl, tk_off, 0, 0);
    SCHED;
    if (DEPTH == 2) issue(i1, B); else issue(i0, A);
    SCHED; pad(); SCHED;
#define PROG() do { } while (0)
#define STREAM_STEP(X, Y, inx) \
        _Pragma("unroll") for (int s = 0; s < NS; ++s) asm volatile("" : "+v"(X.P[s][0]), "+v"(X.P[s][1]), "+v"(X.P[s][2]), "+v"(X.P[s][3])); \
        if (!J::WRITES_OUT) { if (DEPTH == 2) WAIT_V(8); else WAIT_V(4); } \
        PROG(); \
        check(inx, rec, Y); \
        { const unsigned in2 = rfl(tk) - tk_base; \
          if (iq < N && !dead) job.signal(iq, q0, call, lane); \
          rec = __builtin_amdgcn_raw_buffer_load_b64(rctl, rec_off(in2), 0, AUX_SYS); \
          tk = __builtin_amdgcn_raw_ptr_buffer_atomic_add_i32(1, rctl, tk_off, 0, 0); SCHED; \
          issue(inx, Y); SCHED; \
          job.finish(i0, X, lane, dead); SCHED; \
          iq = ip; q0 = p0; ip = i0; p0 = X.r0[0]; \
          if (i1 >= N) break; \
          i0 = i1; if (DEPTH == 2) { i1 = i2; i2 = in2; } else i1 = in2; }
    if (DEPTH == 2) for (;;) { STREAM_STEP(A, C, i2) STREAM_STEP(B, A, i2) STREAM_STEP(C, B, i2) }
    else for (;;) { STREAM_STEP(A, B, i1) STREAM_STEP(B, A, i1) }
#undef STREAM_STEP
    WAIT_V(0);
    if (!dead) { if (iq < N) job.signal(iq, q0, call, lane); job.signal(ip, p0, call, lane); }
}

template <int NS>
__device__ __forceinline__ void reduce_store(rsrc_t rout, unsigned pr, unsigned n, const RBuf<NS>& B, LAS char* cw, int lane, bool dead) {
    const unsigned t = (n >> 5) & 1, ct = (n >> 6) & 15, w = (n >> 2) & 7, u = n & 3;
    float v[32];
#pragma unroll
    for (int s = 0; s < NS; ++s) {
        unsigned D[16]; decode(B.P[s], B.r0[s], cw, lane, D);
#pragma unroll
        for (int i = 0; i < 16; ++i) {
            const float a = __builtin_bit_cast(float, D[i] << 16), b = __builtin_bit_cast(float, D[i] & 0xFFFF0000u);
            v[2 * i] = s == 0 ? a : v[2 * i] + a; v[2 * i + 1] = s == 0 ? b : v[2 * i + 1] + b;
        }
    }
    const int fr = lane & 15, fq = lane >> 4;
    const unsigned vo = fr * 16384 + ((fq & 1) * 16 + (fq >> 1) * 8) * 2;
    const unsigned so = ((pr >> 1) * 512 + t * 256 + (u >> 1) * 128 + (w >> 2) * 64) * 16384 + ((pr & 1) * 4096 + ct * 256 + (u & 1) * 128 + (w & 3) * 32) * 2;
#pragma unroll
    for (int m = 0; m < 4; ++m) {
        unsigned d[4];
#pragma unroll
        for (int h = 0; h < 4; ++h) {
            const float a = v[8 * m + 2 * h], b = v[8 * m + 2 * h + 1];
            d[h] = cvt_pk_bf16(a < 0.f ? 0.f : a, b < 0.f ? 0.f : b);
        }
        const auto s0 = __builtin_amdgcn_permlane16_swap(d[0], d[2], false, false);
        const auto s1 = __builtin_amdgcn_permlane16_swap(d[1], d[3], false, false);
        u32x4 o = {s0[0], s1[0], s0[1], s1[1]};
        if (dead) o = (u32x4){0x7FC07FC0u, 0x7FC07FC0u, 0x7FC07FC0u, 0x7FC07FC0u};
        __builtin_amdgcn_raw_buffer_store_b128(o, rout, vo + so + m * 16 * 16384, 0, AUX_AGENT);
    }
}

struct UnpackJob {
    static constexpr bool WRITES_OUT = false; static constexpr unsigned ESC_BASE = PKI_MAIN;
    rsrc_t rs, rd; char* myws; LAS char* cw; unsigned tb[2][4];
    __device__ __forceinline__ unsigned tab(int f, unsigned idx) const { const unsigned l = (idx >> 4) & 63, a = rdl(tb[0][f], l), b = rdl(tb[1][f], l); return (idx & 1024) ? b : a; }
    __device__ __forceinline__ unsigned rec(unsigned idx, int) const { return tab(0, idx) + (idx & 15) * 8; }
    __device__ __forceinline__ unsigned src_chunk(unsigned idx, int) const { return tab(1, idx) + (idx & 15); }
    __device__ __forceinline__ rsrc_t src(int) const { return rs; }
    __device__ __forceinline__ void signal(unsigned idx, unsigned, unsigned, int lane) const {
        __builtin_amdgcn_raw_ptr_buffer_atomic_add_i32(1, mk(myws, CTL_BYTES), lane == 0 ? tab(3, idx) : OOB, 0, 0);
    }
    __device__ __forceinline__ void finish(unsigned idx, const RBuf<1>& B, int lane, bool) const {
        const unsigned d0 = tab(2, idx), ch = idx & 15;
        const bool isx = d0 < XS_BYTES;
        const unsigned st = isx ? 2048 : 16384, dst = d0 + (isx ? ch * 32768 : (ch >> 1) * 262144 + (ch & 1) * 256);
        const unsigned vo = (lane >> 4) * st + (lane & 15) * 16 + dst;
        unsigned D[16]; decode(B.P[0], B.r0[0], cw, lane, D);
#pragma unroll
        for (int i = 0; i < 4; ++i)
            __builtin_amdgcn_raw_buffer_store_b128((u32x4){D[4 * i], D[4 * i + 1], D[4 * i + 2], D[4 * i + 3]}, rd, vo + i * 4 * st, 0, AUX_AGENT);
    }
};
struct CopyJob {
    static constexpr bool WRITES_OUT = false; static constexpr unsigned ESC_BASE = PKO_MAIN;
    rsrc_t rs, rd; char* myws; unsigned rbase;
    __device__ __forceinline__ unsigned rec(unsigned idx, int) const { return rbase + (1024 + idx) * 8; }
    __device__ __forceinline__ unsigned src_chunk(unsigned idx, int) const { return 1024 + (idx & 1023); }
    __device__ __forceinline__ rsrc_t src(int) const { return rs; }
    __device__ __forceinline__ void signal(unsigned idx, unsigned r0, unsigned r1, int lane) const {
        __builtin_amdgcn_raw_buffer_store_b64((u32x2){r0, r1}, mk(myws, CTL_BYTES), lane == 0 ? rbase + idx * 8 : OOB, 0, AUX_SYS);
    }
    __device__ __forceinline__ void finish(unsigned idx, const RBuf<1>& B, int lane, bool) const {
        const unsigned c = idx & 1023;
        store_chunk(rd, c * 3072, PKO_MAIN + c * 1024, lane, B.P[0], B.r0[0]);
    }
};
struct ReduceJob {
    static constexpr bool WRITES_OUT = true; static constexpr unsigned ESC_BASE = PKO_MAIN;
    rsrc_t rs[2], ro; LAS char* cw; unsigned rbase, quad, pr;
    __device__ __forceinline__ unsigned rec(unsigned idx, int lane) const { return rbase + ((2 * quad + lane) * 1024 + idx) * 8; }
    __device__ __forceinline__ unsigned src_chunk(unsigned idx, int s) const { return (2 * quad + s) * 1024 + (idx & 1023); }
    __device__ __forceinline__ rsrc_t src(int s) const { return rs[s]; }
    __device__ __forceinline__ void signal(unsigned, unsigned, unsigned, int) const {}
    __device__ __forceinline__ void finish(unsigned idx, const RBuf<2>& B, int lane, bool dead) const { reduce_store<2>(ro, pr, idx, B, cw, lane, dead); }
};

__device__ __forceinline__ void reduce_own(const Params& p, char* myws, unsigned call, LAS char* cw, int lane, unsigned n0) {
    const unsigned par = call & 1;
    const rsrc_t rctl = mk(myws, CTL_BYTES), rout = mk(p.out, IN_BYTES), rs = mk(myws + OFF_PKO + par * PKO_PAR, PKO_PAR);
    bool dead = false;
    for (unsigned k = 0; k < 2; ++k) {
        const unsigned n = n0 + k;
        const unsigned ro = lane < 5 ? OFF_RECO + ((par * 12 + (lane ? 7 + lane : 0)) * 1024 + n) * 8 : OOB;
        u32x2 rec = {0, 0};
        const unsigned long long t0 = __builtin_amdgcn_s_memrealtime();
        unsigned sp = 0;
        while (!dead) {
            rec = __builtin_amdgcn_raw_buffer_load_b64(rctl, ro, 0, AUX_SYS);
            if (((unsigned)__ballot(rec[1] == call) & 31u) == 31u) break;
            dead = spin_fail<SLEEP_O>(myws, call, t0, sp);
        }
        asm volatile("" ::: "memory");
        RBuf<5> B;
#pragma unroll
        for (int s = 0; s < 5; ++s) {
            B.r0[s] = rdl(rec[0], s);
            const unsigned c = (s ? 7 + s : 0) * 1024 + n;
            load_chunk(rs, dead ? OOB : c * 3072, dead ? OOB : PKO_MAIN + c * 1024, lane, B.P[s], B.r0[s]);
        }
        if (k == 1 && !dead) dead = timed_out(myws, call);
        reduce_store<5>(rout, (unsigned)p.rank >> 1, n, B, cw, lane, dead);
    }
}

__global__ __launch_bounds__(512, 2) void gemm_rs_kernel(const Params p) {
    extern __shared__ __attribute__((aligned(16))) char shm[];
    LAS char* lds = (LAS char*)shm;
    const int tid = threadIdx.x, wid = rfl(tid >> 6), lane = tid & 63;
    const unsigned b = blockIdx.x, rank = p.rank & 7;
    char* const myws = p.ws[rank];
    STAMP(0);
    u32x4 R[4];
    pack_load(p, b, wid, lane, R);
    if (tid < 64) *(LAS unsigned*)(lds + LDS_STAGE + 256 + tid * 4) = 0;
    unsigned* callp = (unsigned*)(myws + OFF_CALL + b * 128);
    const unsigned call = rfl(__hip_atomic_load(callp, __ATOMIC_RELAXED, __HIP_MEMORY_SCOPE_AGENT)) + 1;
    const unsigned par = call & 1;
    LAS char* const cw = lds + LDS_STAGE + 1024 + wid * CW_BYTES;
    {
        const rsrc_t rpk = mk(myws + OFF_PKI + par * PKI_PAR, PKI_PAR);
        unsigned G = b, it = 0;
        while (G < 1024) {
            const unsigned Gn = b < NGEMM ? 256 + b + NGEMM * it : 1024;
            unsigned D[16];
#pragma unroll
            for (int i = 0; i < 16; ++i) D[i] = R[i >> 2][i & 3];
            u32x4 P[4];
            const unsigned r0 = encode(D, cw, lane, P);
            const PackJob j = pack_job(G, wid);
            store_chunk(rpk, j.gc * 3072, PKI_MAIN + j.gc * 1024, lane, P, r0);
            SCHED;
            pack_load(p, Gn, wid, lane, R);
            SCHED;
            WAIT_V(4);
            LAS u32x2* lrec = (LAS u32x2*)(lds + LDS_STAGE + (it & 1) * 64);
            if (lane == 0) lrec[wid] = (u32x2){r0, call};
            WAIT_L(0); BAR;
            if (wid < 2) {
                const unsigned h = j.mtx ? j.hi * 4 + wid * 2 + j.kk : wid * 4 + j.hi * 2 + j.kk;
                if ((h >> 1) != (rank >> 1) && lane < 8) {
                    const u32x2 r = lrec[lane];
                    __builtin_amdgcn_raw_buffer_store_b64(r, mk(p.ws[h & 7], CTL_BYTES), lane * 8, OFF_RECI + ((par * 8 + rank) * 2048 + j.rec_idx) * 8, AUX_SYS);
                }
            }
            if (it == 0 && tid == 128) __hip_atomic_store(callp, call, __ATOMIC_RELAXED, __HIP_MEMORY_SCOPE_AGENT);
            G = Gn; ++it;
        }
    }
    STAMP(1);
    const unsigned gi = rank >> 2, gj = (rank >> 1) & 1, gk = rank & 1;
    if (b >= NGEMM) {
        const unsigned c = b - NGEMM;
        if (c < 6 * NPW && wid < PWV) {
            const unsigned si = c % 6, g = (rank ^ (si + 2)) & 7;
            UnpackJob job;
            job.rs = mk(p.ws[g] + OFF_PKI + par * PKI_PAR, PKI_PAR); job.rd = mk(myws + OFF_XS, XS_BYTES + WS_BYTES); job.myws = myws; job.cw = cw;
#pragma unroll
            for (int hf = 0; hf < 2; ++hf) {
                const unsigned en = p.order.e[hf * 64 + lane], mtx = (en >> 6) & 1, q = (en >> 4) & 3, t = (en >> 3) & 1, d = en & 7, ct = en & 15;
                const unsigned sh = rank ^ d, ks = gk * 4 + q;
                const unsigned tile32 = mtx ? sh * 4 + gi * 2 + t : gj * 16 + ct, t16 = mtx ? sh * 2 + t : ct;
                job.tb[hf][0] = OFF_RECI + ((par * 8 + g) * 2048 + (mtx * 64 + t16 * 4 + q) * 16) * 8;
                job.tb[hf][1] = (mtx * 256 + tile32 * 8 + ks) * 16;
                job.tb[hf][2] = mtx ? (si >> 1) * 8 * MiB + (t * 8 + d) * 256 * 2048 + (si & 1) * 1024 + q * 256
                                    : XS_BYTES + ((si >> 1) * 512 + q * 128) * 16384 + (si & 1) * 8192 + ct * 512;
                job.tb[hf][3] = OFF_RDY + (mtx ? t * 8 + d : 16 + ct) * 128 + (q * 6 + si) * 4;
            }
            run_stream<1, PDEPTH>(job, myws, call, lane, (c / 6) * PWV + wid, NPW * PWV, 2048, si);
        }
        STAMP(2);
        if (c < 3 * RWG && wid < RWV) {
            ReduceJob job; job.quad = 1 + c % 3; job.pr = (rank >> 1) ^ job.quad; job.rbase = OFF_RECO + par * 12 * 1024 * 8;
            job.rs[0] = mk(p.ws[(rank ^ (2 * job.quad)) & 7] + OFF_PKO + par * PKO_PAR, PKO_PAR);
            job.rs[1] = mk(p.ws[(rank ^ (2 * job.quad + 1)) & 7] + OFF_PKO + par * PKO_PAR, PKO_PAR);
            job.ro = mk(p.out, IN_BYTES); job.cw = cw;
            run_stream<2, RDEPTH>(job, myws, call, lane, (c / 3) * RWV + wid, RWG * RWV, 1024, 6 + job.quad);
        }
        if (c >= 32 - CWG && wid < CWV) {
            CopyJob job; job.myws = myws; job.rbase = OFF_RECO + par * 12 * 1024 * 8;
            job.rs = mk(p.ws[rank ^ 1] + OFF_PKO + par * PKO_PAR, PKO_PAR); job.rd = mk(myws + OFF_PKO + par * PKO_PAR, PKO_PAR);
            run_stream<1, 1>(job, myws, call, lane, (31 - c) * CWV + wid, CWG * CWV, 1024, 6);
        }
        STAMP(3); STAMP(60 + par);
        return;
    }
    const unsigned z = b & 7, slot = b >> 3;
    unsigned ut = z & 1, uct = (z >> 1) * 4 + slot / 7, role = 0;
    const int wr = wid >> 2, wc = wid & 3, fr = lane & 15, fq = lane >> 4;
    const unsigned voffA = (wid * 8 + (lane >> 3)) * 2048 + (((lane & 7) ^ ((wid * 4 + (lane >> 4)) & 7)) * 16);
    const unsigned voffB = (wid * 4 + (lane >> 4)) * 16384 + (((lane & 15) ^ (((lane >> 4) << 2) | (wid & 2))) * 16);
    const unsigned ldsw = wid * 1024;
    int aoff[2], boff[2];
#pragma unroll
    for (int k = 0; k < 2; ++k) aoff[k] = (wr * 64 + fr) * 128 + (((4 * k + fq) ^ ((fr >> 1) & 7)) * 16);
#pragma unroll
    for (int n = 0; n < 2; ++n) { const int q = (lane & 15) >> 2, pp = lane & 3;
        boff[n] = (int)(unsigned)(size_t)lds + 65536 + (8 * fq + q) * 256 + (((4 * wc + 2 * n + (pp >> 1)) ^ ((q << 2) | ((fq & 1) << 1))) * 16) + 8 * (pp & 1); }
    const rsrc_t rctl = mk(myws, CTL_BYTES);
#define HT 16384
#define SA(bb, h) (((bb) * 2 + (h)) * HT)
#define SB(bb, h) ((4 + (bb) * 2 + (h)) * HT)
#define STAGE_A(bb, h, rs, so) do { _Pragma("unroll") for (int _i = 0; _i < 2; ++_i) \
        __builtin_amdgcn_raw_ptr_buffer_load_lds(rs, (LAS void*)(lds + SA(bb, h) + ldsw + _i * 8192), 16, voffA, (so) + (h) * 262144 + _i * 131072, 0, AUX_AGENT); } while (0)
#define STAGE_B(bb, h, rs, so) do { _Pragma("unroll") for (int _i = 0; _i < 2; ++_i) \
        __builtin_amdgcn_raw_ptr_buffer_load_lds(rs, (LAS void*)(lds + SB(bb, h) + ldsw + _i * 8192), 16, voffB, (so) + (h) * 256 + _i * 524288, 0, AUX_AGENT); } while (0)
#define LDA(dst, bb, h) do { _Pragma("unroll") for (int m = 0; m < 4; ++m) _Pragma("unroll") for (int k = 0; k < 2; ++k) \
        dst[m][k] = *(const LAS bf16x8*)(lds + SA(bb, h) + aoff[k] + m * 2048); } while (0)
#define LDB(T, bb, h) asm volatile( \
        "ds_read_b64_tr_b16 %0, %8 offset:%10\n\tds_read_b64_tr_b16 %1, %8 offset:%11\n\tds_read_b64_tr_b16 %2, %8 offset:%12\n\tds_read_b64_tr_b16 %3, %8 offset:%13\n\t" \
        "ds_read_b64_tr_b16 %4, %9 offset:%10\n\tds_read_b64_tr_b16 %5, %9 offset:%11\n\tds_read_b64_tr_b16 %6, %9 offset:%12\n\tds_read_b64_tr_b16 %7, %9 offset:%13" \
        : "=&v"(T[0]), "=&v"(T[1]), "=&v"(T[2]), "=&v"(T[3]), "=&v"(T[4]), "=&v"(T[5]), "=&v"(T[6]), "=&v"(T[7]) \
        : "v"(boff[0]), "v"(boff[1]), "i"(SB(bb, h) - 65536), "i"(SB(bb, h) - 65536 + 1024), "i"(SB(bb, h) - 65536 + 8192), "i"(SB(bb, h) - 65536 + 9216) : "memory")
#define LDB_USE(dst, T) do { asm volatile("s_waitcnt lgkmcnt(0)" : "+v"(T[0]), "+v"(T[1]), "+v"(T[2]), "+v"(T[3]), "+v"(T[4]), "+v"(T[5]), "+v"(T[6]), "+v"(T[7]) :: "memory"); \
        _Pragma("unroll") for (int _j = 0; _j < 4; ++_j) dst[_j >> 1][_j & 1] = __builtin_shufflevector(T[2 * _j], T[2 * _j + 1], 0, 1, 2, 3, 4, 5, 6, 7); } while (0)
#define MMA(ai, bj, At, Bt) do { __builtin_amdgcn_s_setprio(1); _Pragma("unroll") for (int m = 0; m < 4; ++m) _Pragma("unroll") for (int n = 0; n < 2; ++n) _Pragma("unroll") for (int k = 0; k < 2; ++k) \
        acc[ai][bj][m][n] = __builtin_amdgcn_mfma_f32_16x16x32_bf16(Bt[n][k], At[m][k], acc[ai][bj][m][n], 0, 0, 0); __builtin_amdgcn_s_setprio(0); } while (0)
    constexpr unsigned AK = 128, BK = 1048576;
    for (int un = 0; un < 2; ++un) {
        const unsigned d = un ? 0 : 1 + slot % 7, s0 = un ? (role & 3) * 8 : 0, nst = un ? 8 : 32;
        const unsigned rt = ut * 8 + d;
        const unsigned aown = ((rank ^ d) * 1024 + gi * 512 + ut * 256) * 2048, bown = (gj * 4096 + uct * 256) * 2;
        const unsigned arem = rt * 256 * 2048, brem = uct * 512;
        const unsigned rdy_off = OFF_RDY + (lane < 32 ? rt : 16 + uct) * 128 + (lane & 31) * 4;
        auto step_a = [&](unsigned s) -> unsigned { const unsigned r = s - 8, q = (r * 43) >> 8, si = r - q * 6;
            return s < 8 ? aown + s * 256 : arem + (si >> 1) * 8 * MiB + (si & 1) * 1024 + q * 256; };
        auto step_b = [&](unsigned s) -> unsigned { const unsigned r = s - 8, q = (r * 43) >> 8, si = r - q * 6;
            return s < 8 ? bown + s * 128 * 16384 : brem + ((si >> 1) * 512 + q * 128) * 16384 + (si & 1) * 8192; };
        LAS unsigned* lseq = (LAS unsigned*)(lds + LDS_STAGE + 256 + un * 128);
        const unsigned want = un ? 0xFFu << s0 : 0xFFFFFFFFu;
        unsigned seen = want & 0xFFu, seqv = (lane < 8 && s0 == 0) ? lane + 1 : 0;
        auto get_step = [&](unsigned k) -> unsigned {
            unsigned e = rdl(seqv, k);
            if (e == 0) {
                if (wid == 0 && (unsigned)__builtin_popcount(seen) <= k) {
                    const unsigned long long t0 = __builtin_amdgcn_s_memrealtime();
                    unsigned fresh;
                    for (unsigned sp = 0;;) {
                        const unsigned v = __builtin_amdgcn_raw_buffer_load_b32(rctl, rdy_off, 0, AUX_AGENT);
                        const unsigned long long m = __ballot((int)(v - call * 16u) >= 0);
                        fresh = (((unsigned)m & (unsigned)(m >> 32)) << 8) & want & ~seen;
                        if (fresh) break;
                        if (spin_fail<SLEEP_G>(myws, call, t0, sp)) { fresh = want & ~seen; break; }
                    }
                    if (lane < 32 && ((fresh >> lane) & 1))
                        __hip_atomic_store(lseq + __builtin_popcount(seen) + __builtin_popcount(fresh & ((1u << lane) - 1)), (unsigned)lane + 1, __ATOMIC_RELEASE, __HIP_MEMORY_SCOPE_WORKGROUP);
                    seen |= fresh;
                }
                for (;;) {
                    seqv = __hip_atomic_load(lseq + (lane & 31), __ATOMIC_ACQUIRE, __HIP_MEMORY_SCOPE_WORKGROUP);
                    e = rdl(seqv, k);
                    if (e) break;
                    __builtin_amdgcn_s_sleep(1);
                }
                asm volatile("" ::: "memory");
            }
            return (e - 1) & 31;
        };
        f32x4 acc[2][2][4][2];
#pragma unroll
        for (int a = 0; a < 2; ++a)
#pragma unroll
            for (int bb = 0; bb < 2; ++bb)
#pragma unroll
                for (int m = 0; m < 4; ++m)
#pragma unroll
                    for (int n = 0; n < 2; ++n) acc[a][bb][m][n] = (f32x4){0.f, 0.f, 0.f, 0.f};
        bf16x8 At[4][2], B0[2][2], B1[2][2]; s16x4 TB[8];
        unsigned sn = get_step(0);
        rsrc_t ra = mk(sn < 8 ? p.x : myws + OFF_XS, sn < 8 ? IN_BYTES : XS_BYTES);
        unsigned sa = step_a(sn);
        {
            const rsrc_t rb = mk(sn < 8 ? p.w : myws + OFF_WS, sn < 8 ? IN_BYTES : WS_BYTES);
            const unsigned sb = step_b(sn);
            STAGE_B(0, 0, rb, sb); STAGE_A(0, 0, ra, sa); STAGE_B(0, 1, rb, sb); STAGE_A(0, 1, ra, sa);
            if (wr == 1) BAR;
            WAIT_V(4); BAR;
            STAGE_B(1, 0, rb, sb + BK); STAGE_A(1, 0, ra, sa + AK); STAGE_B(1, 1, rb, sb + BK);
            WAIT_V(6); BAR;
        }
        for (unsigned s = 0; s < nst; ++s) {
            if (s + 1 < nst) sn = get_step(s + 1);
            const rsrc_t rna = mk(sn < 8 ? p.x : myws + OFF_XS, sn < 8 ? IN_BYTES : XS_BYTES);
            const rsrc_t rnb = mk(sn < 8 ? p.w : myws + OFF_WS, sn < 8 ? IN_BYTES : WS_BYTES);
            const unsigned na = step_a(sn), nb = step_b(sn);
            LDB(TB, 0, 0); SCHED; LDA(At, 0, 0); STAGE_A(1, 1, ra, sa + AK);
            WAIT_L(8); BAR; LDB_USE(B0, TB); MMA(0, 0, At, B0); BAR; SCHED;
            LDB(TB, 0, 1); STAGE_B(0, 0, rnb, nb);
            BAR; LDB_USE(B1, TB); MMA(0, 1, At, B1); BAR;
            LDA(At, 0, 1); STAGE_A(0, 0, rna, na);
            BAR; WAIT_L(0); MMA(1, 0, At, B0); BAR; SCHED;
            STAGE_B(0, 1, rnb, nb);
            WAIT_V(6); BAR; MMA(1, 1, At, B1); BAR;
            LDB(TB, 1, 0); SCHED; LDA(At, 1, 0); STAGE_A(0, 1, rna, na);
            WAIT_L(8); BAR; LDB_USE(B0, TB); MMA(0, 0, At, B0); BAR; SCHED;
            LDB(TB, 1, 1); STAGE_B(1, 0, rnb, nb + BK);
            BAR; LDB_USE(B1, TB); MMA(0, 1, At, B1); BAR;
            LDA(At, 1, 1); STAGE_A(1, 0, rna, na + AK);
            BAR; WAIT_L(0); MMA(1, 0, At, B0); BAR; SCHED;
            STAGE_B(1, 1, rnb, nb + BK);
            WAIT_V(6); BAR; MMA(1, 1, At, B1); BAR;
            ra = rna; sa = na;
        }
        if (wr == 0) BAR;
        STAMP(4 + 2 * un);
        {
            const unsigned dst = (rank ^ d) & 7;
            const unsigned c0 = (un ? 8 + (role & 3) : d) * 1024 + uct * 64 + ut * 32 + wid * 4;
            unsigned tk = 0;
            if (un == 0 && wid == 0) tk = __builtin_amdgcn_raw_ptr_buffer_atomic_add_i32(1, rctl, lane == 0 ? OFF_TKT + 10 * 128 : OOB, 0, 0);
            const rsrc_t rpk = mk(myws + OFF_PKO + par * PKO_PAR, PKO_PAR);
            u32x2 rec = {0, 0};
            const unsigned tmo = __hip_atomic_load((unsigned*)(myws + OFF_TMO), __ATOMIC_RELAXED, __HIP_MEMORY_SCOPE_AGENT);
#pragma unroll
            for (int u = 0; u < 4; ++u) {
                unsigned D[16];
#pragma unroll
                for (int m = 0; m < 4; ++m)
#pragma unroll
                    for (int n = 0; n < 2; ++n) {
                        const f32x4 a = acc[u >> 1][u & 1][m][n];
                        D[m * 4 + n * 2] = cvt_pk_bf16(a[0], a[1]); D[m * 4 + n * 2 + 1] = cvt_pk_bf16(a[2], a[3]);
                    }
                u32x4 P[4];
                const unsigned r0 = encode(D, cw, lane, P), c = c0 + u;
                store_chunk(rpk, c * 3072, PKO_MAIN + c * 1024, lane, P, r0);
                if (lane == u) rec = (u32x2){r0, call};
            }
            WAIT_V(0);
            if (tmo != call && lane < 4)
                __builtin_amdgcn_raw_buffer_store_b64(rec, mk(p.ws[dst], CTL_BYTES), lane * 8, OFF_RECO + (par * 12 * 1024 + c0) * 8, AUX_SYS);
            if (un == 0 && tid == 0) *(LAS unsigned*)(lds + LDS_STAGE + 128) = tk - (call - 1) * NGEMM;
            WAIT_L(0);
        }
        BAR;
        STAMP(5 + 2 * un);
        if (un) break;
        role = rfl(*(volatile LAS unsigned*)(lds + LDS_STAGE + 128));
        if (role >= 128) break;
        uct = (role >> 3) & 15; ut = (role >> 2) & 1;
    }
    if (role >= 128 && role < 192) reduce_own(p, myws, call, cw, lane, (role - 128) * 16 + wid * 2);
    STAMP(3); STAMP(60 + par);
}

}

extern "C" void kernel_launch(int rank, int nranks, ncclComm_t comm, hipStream_t stream,
                              void* const* d_in, const int* in_sizes, int n_in,
                              void* d_out, int out_size,
                              const int* in_dtypes, int out_dtype,
                              void* const* d_out_all,
                              void* const* d_ws_all, size_t ws_bytes) {
    (void)comm; (void)in_dtypes; (void)out_dtype; (void)d_out_all;
    if (nranks != NR || (unsigned)rank >= (unsigned)NR || n_in < 2 || ws_bytes < WS_NEED || out_size != 1024 * 8192 ||
        in_sizes[0] != 8192 * 1024 || in_sizes[1] != 1024 * 8192) {
        fprintf(stderr, "kernel_launch: unexpected problem geometry\n");
        return;
    }
    HIP_CHECK(hipFuncSetAttribute((const void*)gemm_rs_kernel, hipFuncAttributeMaxDynamicSharedMemorySize, LDS_BYTES));
    Params p;
    p.x = (const char*)d_in[0]; p.w = (const char*)d_in[1]; p.out = (char*)d_out; p.rank = rank;
    constexpr Order order = make_order();
    p.order = order;
    for (int r = 0; r < NR; ++r) p.ws[r] = (char*)d_ws_all[r];
    hipLaunchKernelGGL(gemm_rs_kernel, dim3(256), dim3(512), LDS_BYTES, stream, p);
    HIP_CHECK(hipGetLastError());
}
```

```cpp
#include <hip/hip_runtime.h>
#include <rccl/rccl.h>
#include <cstdio>

#define HIP_CHECK(expr) do { hipError_t _e = (expr); if (_e != hipSuccess) { \
    fprintf(stderr, "HIP error %s at %s:%d\n", hipGetErrorString(_e), __FILE__, __LINE__); \
    return; } } while (0)

namespace {

#define LAS __attribute__((address_space(3)))
typedef short bf16x8 __attribute__((ext_vector_type(8)));
typedef short s16x4 __attribute__((ext_vector_type(4)));
typedef float f32x4 __attribute__((ext_vector_type(4)));
typedef unsigned u32x4 __attribute__((ext_vector_type(4)));
typedef unsigned u32x2 __attribute__((ext_vector_type(2)));
typedef unsigned short u16x2 __attribute__((ext_vector_type(2)));
typedef __amdgpu_buffer_rsrc_t rsrc_t;

constexpr int NR = 8;
constexpr unsigned MiB = 1u << 20;
constexpr unsigned OFF_CALL = 0;
constexpr unsigned OFF_TMO = 32768;
constexpr unsigned OFF_RDY = 36864;
constexpr unsigned OFF_TKT = 40960;
constexpr unsigned OFF_RECI = 65536;
constexpr unsigned OFF_RECO = 327680;
constexpr unsigned OFF_DBG = 524288;
constexpr unsigned CTL_BYTES = MiB;
constexpr unsigned OFF_PKI = 1 * MiB, PKI_MAIN = 27 * MiB, PKI_PAR = 36 * MiB;
constexpr unsigned OFF_PKO = 73 * MiB, PKO_MAIN = 36 * MiB, PKO_PAR = 48 * MiB;
constexpr unsigned OFF_XS = 169 * MiB, XS_BYTES = 24 * MiB;
constexpr unsigned OFF_WS = 193 * MiB, WS_BYTES = 24 * MiB;
constexpr unsigned WS_NEED = 217 * MiB;
constexpr unsigned IN_BYTES = 16 * MiB;
constexpr unsigned OOB = 0x80000000u;
constexpr unsigned long long DEADLINE = 30000000ull;

constexpr int LDS_STAGE = 131072, LDS_LUT = LDS_STAGE + 1024 + 8 * 2128, LDS_RING = LDS_LUT + 2048, LDS_BYTES = LDS_RING + 1024;
constexpr int NGEMM = 224;
constexpr int NPW = 5;
#define WIN 12
#define FINE 0
constexpr int DEPTH = (WIN + 2) / 3;
#define AHEAD 4
#define TAILG 20
#define CPY 3
#define NPUSH 4
#define INBOX_IT 1
#define INBOX_G0 0
#define DELAY0 0
#define URGENT 64
static_assert(NPUSH % 2 == 0 && NPUSH >= 2 && NPUSH <= 4 && TAILG % 2 == 0, "");
#define SLEEP_G 8
#define SLEEP_S 4
#define SLEEP_O 32
constexpr int AUX_SYS = 17, AUX_AGENT = 16;

#define SLOPE 28
#define XSTEP 26
#define XLEAD 1
#define PDELAY 0
#define GROUPT 156
__device__ __forceinline__ unsigned xpiece(unsigned x) { return (unsigned)(0x80A2FD75EC64B931ull >> (4 * x)) & 15; }
struct Order { unsigned char e[128]; };
constexpr Order make_order() {
    Order o{}; int key[128] = {}; int n = 0;
    for (int q = 0; q < 4; ++q) {
        for (int x = 0; x < 16; ++x) {
            o.e[n] = (unsigned char)(64 | q << 4 | x);
            key[n++] = (q == 0 && x < NPUSH) ? 200000 : x >= 14 ? 100000 + q : !q ? -1 : x < 2 ? q * GROUPT - XLEAD : q * GROUPT + 1 + (x - 2) / 2 * XSTEP; }
        for (int c = 0; c < 16; ++c) { o.e[n] = (unsigned char)(q << 4 | c); key[n++] = q * GROUPT + c * SLOPE; }
    }
    for (int i = 1; i < 128; ++i)
        for (int j = i; j > 0 && key[j - 1] > key[j]; --j) {
            const int k = key[j]; key[j] = key[j - 1]; key[j - 1] = k; const unsigned char e = o.e[j]; o.e[j] = o.e[j - 1]; o.e[j - 1] = e; }
    return o;
}

struct Params {
    const char* x; const char* w; char* out; char* ws[NR]; int rank; Order order;
};

__device__ __forceinline__ rsrc_t mk(const void* p, unsigned bytes) { return __builtin_amdgcn_make_buffer_rsrc((void*)p, 0, bytes, 0x00020000); }
__device__ __forceinline__ unsigned rfl(unsigned v) { return __builtin_amdgcn_readfirstlane(v); }
__device__ __forceinline__ unsigned rdl(unsigned v, unsigned l) { return __builtin_amdgcn_readlane(v, l); }
#define WAIT_V(n) asm volatile("s_waitcnt vmcnt(" #n ")" ::: "memory")
#define WAIT_L(n) asm volatile("s_waitcnt lgkmcnt(" #n ")" ::: "memory")
#define BAR __builtin_amdgcn_s_barrier()
#define SCHED __builtin_amdgcn_sched_barrier(0)

#define STAMP(k) do { } while (0)
#define TIC do { } while (0)
#define TOC(k) do { } while (0)
#define TOUT(k) do { } while (0)

__device__ __forceinline__ unsigned cvt_pk_bf16(float lo, float hi) {
    unsigned r; asm("v_cvt_pk_bf16_f32 %0, %1, %2" : "=v"(r) : "v"(lo), "v"(hi)); return r;
}
__device__ __forceinline__ unsigned pk_sub(unsigned a, unsigned b) { return __builtin_bit_cast(unsigned, __builtin_bit_cast(u16x2, a) - __builtin_bit_cast(u16x2, b)); }
__device__ __forceinline__ unsigned bfi(unsigned m, unsigned a, unsigned b) { return (a & m) | (b & ~m); }
__device__ __forceinline__ unsigned rotr(unsigned a, unsigned n) { return __builtin_amdgcn_alignbit(a, a, n); }
template <int CTRL> __device__ __forceinline__ float dpp_f(float v) { return __builtin_bit_cast(float, __builtin_amdgcn_update_dpp(0, __builtin_bit_cast(int, v), CTRL, 0xf, 0xf, true)); }
__device__ __forceinline__ float rdl_f(float v, unsigned l) { return __builtin_bit_cast(float, rdl(__builtin_bit_cast(unsigned, v), l)); }

constexpr unsigned RAW = 255, EB = 1040, CW_BYTES = EB + 1024 + 64;
struct Codec { LAS char* cw; LAS const char* lut; };
__device__ __forceinline__ bool is_raw(unsigned r0) { return ((r0 >> 16) & 255) == RAW; }
__device__ __forceinline__ unsigned plane2_bytes(unsigned r0) { return 704 + 4 * ((r0 >> 16) & 255); }
__device__ __forceinline__ unsigned lut_entry(unsigned t) {
    unsigned k = 0, e = t < 256 ? 0 : 0x0C0C0C0Cu;
#pragma unroll
    for (unsigned b = 0; b < 4; ++b) {
        const unsigned c = (t >> (2 * b)) & 3;
        if (t < 256) e |= (c ? 4 + c : k) << (8 * b); else if (!c) e ^= (12 ^ b) << (8 * k);
        k += !c;
    }
    return e;
}
__device__ __forceinline__ unsigned scan_add(unsigned v) {
    int s = v;
    s += __builtin_amdgcn_update_dpp(0, s, 0x111, 0xf, 0xf, false); s += __builtin_amdgcn_update_dpp(0, s, 0x112, 0xf, 0xf, false);
    s += __builtin_amdgcn_update_dpp(0, s, 0x114, 0xf, 0xf, false); s += __builtin_amdgcn_update_dpp(0, s, 0x118, 0xf, 0xf, false);
    s += __builtin_amdgcn_update_dpp(0, s, 0x142, 0xa, 0xf, false); s += __builtin_amdgcn_update_dpp(0, s, 0x143, 0xc, 0xf, false);
    return s;
}
__device__ __forceinline__ unsigned pick_top(const unsigned (&D)[16]) {
    float sm = 0.f;
#pragma unroll
    for (int i = 0; i < 16; i += 4) sm += __builtin_fabsf(__builtin_bit_cast(float, D[i] << 16)) + __builtin_fabsf(__builtin_bit_cast(float, D[i] & 0xFFFF0000u));
    sm += dpp_f<0xB1>(sm); sm += dpp_f<0x4E>(sm); sm += dpp_f<0x141>(sm); sm += dpp_f<0x140>(sm);
    const float top = ((rdl_f(sm, 0) + rdl_f(sm, 16)) + (rdl_f(sm, 32) + rdl_f(sm, 48))) * (20.6f / 512.f);
    return (rfl(__builtin_bit_cast(unsigned, top) >> 15) & 0xFFFFu) | 1u;
}
#define FENCE asm volatile("" ::: "memory")
__device__ __forceinline__ unsigned encode(const unsigned (&D)[16], unsigned top, const Codec& k, int lane, u32x4 (&P)[4]) {
    LAS char* const cw = k.cw;
    const unsigned top2 = top * 0x10001u;
    unsigned F[8], sel[8], C[2] = {0, 0};
#pragma unroll
    for (int i = 0; i < 8; ++i) {
        const unsigned a = pk_sub(top2, rotr(D[2 * i], 31)), b = pk_sub(top2, rotr(D[2 * i + 1], 31));
        P[i >> 2][i & 3] = __builtin_amdgcn_perm(b, a, 0x06040200u);
        F[i] = __builtin_amdgcn_perm(b, a, 0x07050301u);
        const unsigned c = __builtin_amdgcn_perm(0x00000302u, 0x01000000u, F[i]);
        const unsigned cb = __builtin_amdgcn_udot4((c ^ (c >> 2)) & 0x03030303u, 0x40100401u, 0, false);
        sel[i] = *(LAS const unsigned*)(k.lut + 1024 + cb * 4);
        C[i >> 2] |= cb << (8 * (i & 3));
    }
    const unsigned m[2] = {~(C[0] | C[0] >> 1) & 0x55555555u, ~(C[1] | C[1] >> 1) & 0x55555555u};
    const unsigned k0 = __builtin_popcount(m[0]), kk = k0 + __builtin_popcount(m[1]), incl = scan_add(kk), cnt = rdl(incl, 63);
    P[2] = (u32x4){F[0], F[1], F[2], F[3]}; P[3] = (u32x4){F[4], F[5], F[6], F[7]};
    if (cnt > 1024) return top | RAW << 16;
    const unsigned fill = lane * 16u >= cnt ? 0x02020202u : 0;
    *(LAS u32x4*)(cw + EB + lane * 16) = (u32x4){fill, fill, fill, fill};
    FENCE;
    *(LAS unsigned char*)(cw + EB + cnt + lane) = 2;
    *(LAS u32x2*)(cw + lane * 8) = (u32x2){C[0], C[1]};
    FENCE;
    const unsigned e1 = (unsigned)(size_t)cw + EB - 1 + incl;
    unsigned carry = 0;
#pragma unroll
    for (int i = 0; i < 8; ++i) {
        const unsigned p = e1 - kk + (i >= 4 ? k0 : 0) + __builtin_popcount(m[i >> 2] & ((1u << (8 * (i & 3))) - 1)), pk = __builtin_amdgcn_perm(0u, F[i], sel[i]);
        __hip_atomic_fetch_or((LAS unsigned*)(size_t)(p & ~3u), __builtin_amdgcn_alignbyte(pk, 0u, ~p) | carry, __ATOMIC_RELAXED, __HIP_MEMORY_SCOPE_WAVEFRONT);
        carry = __builtin_amdgcn_alignbyte(0u, pk, ~p);
    }
    __hip_atomic_fetch_or((LAS unsigned*)(size_t)(e1 & ~3u), carry, __ATOMIC_RELAXED, __HIP_MEMORY_SCOPE_WAVEFRONT);
    FENCE;
    unsigned M = 0, ovf = 0;
    auto dense = [&](int r) -> unsigned {
        const u32x2 f = *(LAS u32x2*)(cw + EB + r * 512 + lane * 8);
        const unsigned t0 = __builtin_amdgcn_perm(0u, 0x00074040u, f[0]), t1 = __builtin_amdgcn_perm(0u, 0x00074040u, f[1]);
        M |= bfi(0x40404040u, t0, t1 << 1) >> (6 - 2 * r) & 0x03030303u << (2 * r);
        return ((f[0] ^ t0) & 0x07070707u) | ((f[1] ^ t1) & 0x07070707u) << 3;
    };
    const unsigned x = dense(0), y = bfi(0xC0C0C0u, __umul24(x >> 24, 0x41040u), x);
    *(LAS unsigned short*)(cw + 512 + lane * 2) = (unsigned short)y;
    *(LAS unsigned char*)(cw + 640 + lane) = (unsigned char)(y >> 16);
    if (cnt > 512) { *(LAS unsigned*)(cw + 704 + lane * 4) = dense(1); ovf = (cnt - 505) >> 3; }
    FENCE;
    unsigned n = 0;
    for (;;) {
        const unsigned long long act = __ballot(M != 0);
        if (!act) break;
        if (M) {
            const unsigned off = ovf + n + __builtin_amdgcn_mbcnt_hi((unsigned)(act >> 32), __builtin_amdgcn_mbcnt_lo((unsigned)act, 0));
            const unsigned bt = __builtin_ctz(M), pos = (bt & 2) * 256 + lane * 8 + (bt & 1) * 4 + (bt >> 3);
            *(LAS unsigned*)(cw + 704 + (off < 80 ? off : 80) * 4) = pos | (unsigned)*(LAS unsigned char*)(cw + EB + pos) << 16;
            M &= M - 1;
        }
        n += __builtin_popcountll(act);
        if (n > 64 || ovf + n > 80) return top | RAW << 16;
    }
    FENCE;
    P[2] = *(LAS u32x4*)(cw + lane * 16);
    FENCE;
    return top | (ovf + n) << 16 | n << 24;
}
template <class Fn>
__device__ __forceinline__ void decode(LAS const char* in, unsigned r0, const Codec& k, int lane, unsigned (&D)[16], Fn done) {
    LAS char* const cw = k.cw;
    LAS const char* const p2 = in + 2048;
    const unsigned n = r0 >> 24, ovf = ((r0 >> 16) & 255) - n, top2 = (r0 & 0xFFFFu) * 0x10001u;
    unsigned F[8];
    u32x4 L[2];
    FENCE;
    if (is_raw(r0)) {
        const u32x4 f0 = *(LAS const u32x4*)(p2 + lane * 16), f1 = *(LAS const u32x4*)(p2 + 1024 + lane * 16);
        L[0] = *(LAS const u32x4*)(in + lane * 16); L[1] = *(LAS const u32x4*)(in + 1024 + lane * 16);
        FENCE; done(); FENCE;
#pragma unroll
        for (int i = 0; i < 4; ++i) { F[i] = f0[i]; F[i + 4] = f1[i]; }
    } else {
        const u32x2 C = *(LAS const u32x2*)(p2 + lane * 8);
        const unsigned y = *(LAS const unsigned short*)(p2 + 512 + lane * 2) | (unsigned)*(LAS const unsigned char*)(p2 + 640 + lane) << 16;
        const unsigned xo = *(LAS const unsigned*)(p2 + 704 + lane * 4), e = *(LAS const unsigned*)(p2 + 704 + ((ovf + lane) & 127) * 4);
        L[0] = *(LAS const u32x4*)(in + lane * 16); L[1] = *(LAS const u32x4*)(in + 1024 + lane * 16);
        FENCE; done(); FENCE;
        auto spread = [&](int r, unsigned x) {
            *(LAS u32x2*)(cw + EB + r * 512 + lane * 8) = (u32x2){__builtin_amdgcn_perm(0x0706020Cu, 0x0B0A0908u, x & 0x07070707u),
                                                                  __builtin_amdgcn_perm(0x0706020Cu, 0x0B0A0908u, (x >> 3) & 0x07070707u)};
        };
        unsigned sel[8];
#pragma unroll
        for (int i = 0; i < 8; ++i) sel[i] = *(LAS const unsigned*)(k.lut + ((C[i >> 2] >> (8 * (i & 3))) & 255) * 4);
        spread(0, bfi(0x3F000000u, __umul24(y & 0xC0C0C0u, 0x41040u), y));
        if (ovf) spread(1, xo);
        FENCE;
        *(LAS unsigned char*)(cw + ((unsigned)lane < n ? EB + (e & 1023) : 1024)) = (unsigned char)(e >> 16);
        FENCE;
        const unsigned m[2] = {~(C[0] | C[0] >> 1) & 0x55555555u, ~(C[1] | C[1] >> 1) & 0x55555555u};
        const unsigned k0 = __builtin_popcount(m[0]), kk = k0 + __builtin_popcount(m[1]), p0 = (unsigned)(size_t)cw + EB + scan_add(kk) - kk;
#pragma unroll
        for (int i = 0; i < 8; ++i) {
            const unsigned p = p0 + (i >= 4 ? k0 : 0) + __builtin_popcount(m[i >> 2] & ((1u << (8 * (i & 3))) - 1));
            const LAS unsigned* w = (const LAS unsigned*)(size_t)(p & ~3u);
            F[i] = __builtin_amdgcn_perm(0x05040300u, __builtin_amdgcn_alignbyte(w[1], w[0], p), sel[i]);
        }
        FENCE;
    }
#pragma unroll
    for (int i = 0; i < 8; ++i) {
        D[2 * i] = rotr(pk_sub(top2, __builtin_amdgcn_perm(F[i], L[i >> 2][i & 3], 0x05010400u)), 1);
        D[2 * i + 1] = rotr(pk_sub(top2, __builtin_amdgcn_perm(F[i], L[i >> 2][i & 3], 0x07030602u)), 1);
    }
}
__device__ __forceinline__ void store_chunk(rsrc_t rs, unsigned main_off, unsigned esc_off, int lane, const u32x4 (&P)[4], unsigned r0) {
#pragma unroll
    for (int i = 0; i < 2; ++i) __builtin_amdgcn_raw_buffer_store_b128(P[i], rs, main_off + lane * 16 + i * 1024, 0, AUX_SYS);
    __builtin_amdgcn_raw_buffer_store_b128(P[2], rs, lane * 16u < plane2_bytes(r0) ? main_off + lane * 16 + 2048 : OOB, 0, AUX_SYS);
    __builtin_amdgcn_raw_buffer_store_b128(P[3], rs, is_raw(r0) ? esc_off + lane * 16 : OOB, 0, AUX_SYS);
}
__device__ __forceinline__ void load_chunk(rsrc_t rs, unsigned main_off, unsigned esc_off, unsigned r0, LAS char* to, int lane) {
    if (esc_off != OOB) __builtin_amdgcn_raw_ptr_buffer_load_lds(rs, (LAS void*)(to + 3072), 16, lane * 16, esc_off, 0, AUX_SYS);
    __builtin_amdgcn_raw_ptr_buffer_load_lds(rs, (LAS void*)to, 16, lane * 16, main_off, 0, AUX_SYS);
    __builtin_amdgcn_raw_ptr_buffer_load_lds(rs, (LAS void*)(to + 1024), 16, lane * 16 + 1024, main_off, 0, AUX_SYS);
    __builtin_amdgcn_raw_ptr_buffer_load_lds(rs, (LAS void*)(to + 2048), 16, lane * 16u < plane2_bytes(r0) ? lane * 16 + 2048 : OOB, main_off, 0, AUX_SYS);
}

template <int SLEEP>
__device__ __forceinline__ bool spin_fail(char* myws, unsigned call, unsigned long long t0, unsigned& n) {
    unsigned* tmo = (unsigned*)(myws + OFF_TMO);
    if ((++n & 63) == 0 &&
        (rfl(__hip_atomic_load(tmo, __ATOMIC_RELAXED, __HIP_MEMORY_SCOPE_AGENT)) == call || __builtin_amdgcn_s_memrealtime() - t0 > DEADLINE)) {
        __hip_atomic_store(tmo, call, __ATOMIC_RELAXED, __HIP_MEMORY_SCOPE_AGENT);
        return true;
    }
    __builtin_amdgcn_s_sleep(SLEEP);
    return false;
}
__device__ __forceinline__ bool timed_out(char* myws, unsigned call) {
    return rfl(__hip_atomic_load((unsigned*)(myws + OFF_TMO), __ATOMIC_RELAXED, __HIP_MEMORY_SCOPE_AGENT)) == call;
}

struct PackJob { unsigned src_off, stride, gc, rec_idx, x; int mtx, kk, hi; };
__device__ __forceinline__ PackJob pack_job(unsigned G, int wid) {
    PackJob j; const unsigned q = G >> 8, ch = (G & 1) * 8 + wid;
    j.mtx = ((G >> 7) & 1) ^ 1; j.x = (G >> 3) & 15; j.hi = (G >> 2) & 1; j.kk = (G >> 1) & 1; const unsigned ks = j.kk * 4 + q;
    unsigned t16, tile;
    if (j.mtx) { const unsigned e = xpiece(j.x), sh = (j.hi * 4 + j.kk) ^ (e & 7), t = e >> 3;
        tile = sh * 4 + j.hi * 2 + t; t16 = sh * 2 + t; j.src_off = (tile * 256 + ch * 16) * 2048 + ks * 256; j.stride = 2048; }
    else { tile = j.hi * 16 + j.x; t16 = j.x; j.src_off = (ks * 128 + (ch >> 1) * 16) * 16384 + tile * 512 + (ch & 1) * 256; j.stride = 16384; }
    j.gc = (j.mtx * 256 + tile * 8 + ks) * 16 + ch;
    j.rec_idx = (j.mtx * 64 + t16 * 4 + q) * 16 + (G & 1) * 8;
    return j;
}
__device__ __forceinline__ void pack_load(const Params& p, unsigned G, int wid, int lane, u32x4 (&R)[4]) {
    const PackJob j = pack_job(G < 1024 ? G : 0, wid);
    const rsrc_t rs = mk(j.mtx ? p.x : p.w, G < 1024 ? IN_BYTES : 0);
    const unsigned vo = (lane >> 4) * j.stride + (lane & 15) * 16;
#pragma unroll
    for (int i = 0; i < 4; ++i) R[i] = __builtin_amdgcn_raw_buffer_load_b128(rs, vo, j.src_off + i * 4 * j.stride, 0);
}

__device__ __forceinline__ void add_chunk(float (&v)[32], const unsigned (&D)[16], bool first) {
#pragma unroll
    for (int i = 0; i < 16; ++i) {
        const float a = __builtin_bit_cast(float, D[i] << 16), b = __builtin_bit_cast(float, D[i] & 0xFFFF0000u);
        v[2 * i] = first ? a : v[2 * i] + a; v[2 * i + 1] = first ? b : v[2 * i + 1] + b;
    }
}
__device__ __forceinline__ void store_out(rsrc_t rout, unsigned pr, unsigned n, const float (&v)[32], int lane, bool dead) {
    const unsigned t = (n >> 3) & 1, ct = (n >> 6) & 15, w = n & 7, u = (n >> 4) & 3;
    const int fr = lane & 15, fq = lane >> 4;
    const unsigned vo = fr * 16384 + ((fq & 1) * 16 + (fq >> 1) * 8) * 2;
    const unsigned so = ((pr >> 1) * 512 + t * 256 + (u >> 1) * 128 + (w >> 2) * 64) * 16384 + ((pr & 1) * 4096 + ct * 256 + (u & 1) * 128 + (w & 3) * 32) * 2;
#pragma unroll
    for (int m = 0; m < 4; ++m) {
        unsigned d[4];
#pragma unroll
        for (int h = 0; h < 4; ++h) {
            const float a = v[8 * m + 2 * h], b = v[8 * m + 2 * h + 1];
            d[h] = cvt_pk_bf16(a < 0.f ? 0.f : a, b < 0.f ? 0.f : b);
        }
        const auto s0 = __builtin_amdgcn_permlane16_swap(d[0], d[2], false, false);
        const auto s1 = __builtin_amdgcn_permlane16_swap(d[1], d[3], false, false);
        u32x4 o = {s0[0], s1[0], s0[1], s1[1]};
        if (dead) o = (u32x4){0x7FC07FC0u, 0x7FC07FC0u, 0x7FC07FC0u, 0x7FC07FC0u};
        __builtin_amdgcn_raw_buffer_store_b128(o, rout, vo + so + m * 16 * 16384, 0, AUX_AGENT);
    }
}

enum { K_UNPACK = 0, K_SUM = 1, K_COPY = 2, K_END = 3 };
enum { R_FULL = 0, R_ISSUED = 4, R_AVAIL = 8, R_NEXT = 12, R_FREE = 128, R_DESC = 256 };
__device__ __forceinline__ unsigned lds_ld(unsigned a) { unsigned v; asm volatile("ds_read_b32 %0, %1\n\ts_waitcnt lgkmcnt(0)" : "=v"(v) : "v"(a) : "memory"); return v; }
__device__ __forceinline__ unsigned lds_ld2(unsigned a, unsigned b, unsigned& vb) {
    unsigned v; asm volatile("ds_read_b32 %0, %2\n\tds_read_b32 %1, %3\n\ts_waitcnt lgkmcnt(0)" : "=&v"(v), "=&v"(vb) : "v"(a), "v"(b) : "memory"); return v; }
__device__ __forceinline__ void lds_st(unsigned a, unsigned v) { asm volatile("ds_write_b32 %0, %1" :: "v"(a), "v"(v) : "memory"); }
struct Patience {
    unsigned n = 0; unsigned long long t0 = 0;
    __device__ __forceinline__ bool over(char* myws, unsigned call) {
        if ((n++ & 1023) == 0) {
            const unsigned long long t = __builtin_amdgcn_s_memrealtime();
            if (n == 1) t0 = t;
            if (t - t0 > 2 * DEADLINE) { __hip_atomic_store((unsigned*)(myws + OFF_TMO), call, __ATOMIC_RELAXED, __HIP_MEMORY_SCOPE_AGENT); return true; }
        }
        __builtin_amdgcn_s_sleep(1);
        return false;
    }
};
struct Piece { unsigned f[3]; };
__device__ __forceinline__ Piece describe(unsigned en, unsigned si, unsigned rank, unsigned par) {
    const unsigned mtx = (en >> 6) & 1, q = (en >> 4) & 3, ct = en & 15, g = (rank ^ (si + 2)) & 7;
    const unsigned y = (ct + 10 + 2 * si) % 12, x = !q || ct >= 14 ? ct : ct < 2 ? 2 * ct : y ? y + 2 : 1;
    const unsigned e = xpiece(x), t = e >> 3, d = e & 7, sh = rank ^ d, ks = (rank & 1) * 4 + q;
    const unsigned tile32 = mtx ? sh * 4 + (rank >> 2) * 2 + t : (rank & 2) * 8 + ct, t16 = mtx ? sh * 2 + t : ct;
    const unsigned dst = mtx ? (si >> 1) * 8 * MiB + (t * 8 + d) * 256 * 2048 + (si & 1) * 1024 + q * 256
                             : XS_BYTES + ((si >> 1) * 512 + q * 128) * 16384 + (si & 1) * 8192 + ct * 512;
    return {{OFF_RECI + ((par * 8 + g) * 2048 + (mtx * 64 + t16 * 4 + q) * 16) * 8, (mtx * 256 + tile32 * 8 + ks) * 16,
             dst << 5 | ((mtx ? t * 8 + d : 16 + ct) * 32 + q * 6 + si) << 3}};
}
__device__ __forceinline__ unsigned in_word(unsigned w, unsigned ch) { return w + ((w < XS_BYTES << 5 ? ch * 32768 : (ch >> 1) * 262144 + (ch & 1) * 256) << 5); }
__device__ __forceinline__ unsigned records(char* myws, unsigned call, unsigned ro, unsigned need, bool& dead) {
    const unsigned long long t0 = __builtin_amdgcn_s_memrealtime();
    for (unsigned sp = 0; !dead;) {
        const u32x2 rec = __builtin_amdgcn_raw_buffer_load_b64(mk(myws, CTL_BYTES), ro, 0, AUX_SYS);
        if (((unsigned)__ballot(rec[1] == call) & need) == need) return rec[0];
        dead = spin_fail<SLEEP_S>(myws, call, t0, sp);
    }
    return 0;
}
__device__ __forceinline__ void store_in(char* myws, const unsigned (&D)[16], unsigned w, int lane) {
    const unsigned dst = w >> 13 << 8, st = dst < XS_BYTES ? 2048 : 16384, vo = (lane >> 4) * st + (lane & 15) * 16 + dst;
#pragma unroll
    for (int i = 0; i < 4; ++i)
        __builtin_amdgcn_raw_buffer_store_b128((u32x4){D[4 * i], D[4 * i + 1], D[4 * i + 2], D[4 * i + 3]}, mk(myws + OFF_XS, XS_BYTES + WS_BYTES), vo + i * 4 * st, 0, AUX_AGENT);
}
__device__ __forceinline__ void ring(const Params& p, char* myws, unsigned call, LAS char* lds, const Codec& cw, unsigned c, unsigned en0, unsigned en1, int wid, int lane) {
    const unsigned rank = p.rank & 7, par = call & 1;
    const bool cp = c >= 30;
    const unsigned si = c % 6, g = (rank ^ (cp ? 1 : si + 2)) & 7, quad = 1 + (si >> 1), ctl = (unsigned)(size_t)lds + LDS_RING;
    const unsigned pki = OFF_PKI + par * PKI_PAR, pko = OFF_PKO + par * PKO_PAR, reco = OFF_RECO + par * 12 * 1024 * 8;
    const rsrc_t rctl = mk(myws, CTL_BYTES);
    if (wid == 0) {
        __builtin_amdgcn_s_setprio(3);
        unsigned long long src0 = (unsigned long long)p.ws[g], src1 = (unsigned long long)p.ws[g ^ 1];
        asm volatile("" : "+s"(src0), "+s"(src1));
        unsigned av = 0, k0 = 0, d = 0, lim = 0;
        unsigned long long raw = 0;
        int full = 0;
        auto post = [&](unsigned issued) { asm volatile("ds_write_b64 %0, %1" :: "v"(ctl + R_FULL), "v"((u32x2){(unsigned)full, issued}) : "memory"); };
        for (unsigned k = 0;; k += 4) {
            if (av <= k) {
                k0 = k;
                for (Patience pt;;) {
                    const unsigned q = rfl(lds_ld2(ctl + R_AVAIL, ctl + R_DESC + ((k * 16 + lane * 4) & 511), d));
                    av = q < k + 16 ? q : k + 16;
                    if (av > k) break;
                    if (full < (int)k) { WAIT_V(0); full = k; post(k); } else if (pt.over(myws, call)) return;
                }
                lim = plane2_bytes(d); raw = __ballot((lane & 3) == 1 && d != OOB);
            }
            const unsigned l = (k - k0) * 4, w = rdl(d, l + 3);
            if ((w & 3) == K_END) { WAIT_V(0); full = 0x7FFFFFFF; post(k); return; }
            const rsrc_t rs[2] = {mk((void*)(w & si & 1 ? src1 : src0), WS_NEED), mk((void*)(w & ~si & 1 ? src1 : src0), WS_NEED)};
            LAS char* const to = lds + (k & 31) * 4096;
            if ((raw >> l) & 0xFFFF)
#pragma unroll
                for (int i = 0; i < 4; ++i) __builtin_amdgcn_raw_ptr_buffer_load_lds(rs[i & 1], (LAS void*)(to + i * 4096 + 3072), 16, lane * 16, rdl(d, l + 4 * i + 1), 0, AUX_SYS);
#pragma unroll
            for (int i = 0; i < 4; ++i) {
                const unsigned o0 = rdl(d, l + 4 * i), v2 = lane * 16u < rdl(lim, l + 4 * i + 2) ? lane * 16 : OOB;
                SCHED;
                asm volatile("s_waitcnt vmcnt(%0)" :: "i"(FINE ? WIN - 1 : WIN - 3) : "memory");
                __builtin_amdgcn_raw_ptr_buffer_load_lds(rs[i & 1], (LAS void*)(to + i * 4096), 16, lane * 16, o0, 0, AUX_SYS);
                if (FINE) asm volatile("s_waitcnt vmcnt(%0)" :: "i"(WIN - 1) : "memory");
                __builtin_amdgcn_raw_ptr_buffer_load_lds(rs[i & 1], (LAS void*)(to + i * 4096), 16, lane * 16, o0, 1024, AUX_SYS);
                if (FINE) asm volatile("s_waitcnt vmcnt(%0)" :: "i"(WIN - 1) : "memory");
                __builtin_amdgcn_raw_ptr_buffer_load_lds(rs[i & 1], (LAS void*)(to + i * 4096), 16, v2, o0, 2048, AUX_SYS);
                SCHED;
                full = max(full, (int)k + i + 1 - DEPTH);
                post(k + i + 1);
            }
        }
    }
    if (wid == 1) {
        const Piece tb[2] = {describe(en0, si, rank, par), describe(en1, si, rank, par)};
        auto piece = [&](int f, unsigned pc) -> unsigned { const unsigned a = rdl(tb[0].f[f], pc & 63), b = rdl(tb[1].f[f], pc & 63); return pc & 64 ? b : a; };
        constexpr unsigned NG_IN = 256 - 2 * NPUSH;
        const unsigned big = (cp ? 128 : 256) - TAILG / 2, ng_out = big + TAILG, l = lane & 7;
        unsigned kind = cp ? K_COPY : K_UNPACK, gr = cp ? c - 30 : c / 6, tk = 0, ro = OOB, ch = 0, w = 0;
        u32x2 rec = {0, 0};
        bool dead = false;
        auto ask = [&]() {
            if (kind == K_UNPACK && gr >= NG_IN) { kind = K_SUM; gr = (c / 6) * 2 + (si & 1); }
            if (kind != K_UNPACK && gr >= ng_out) { kind = K_END; return; }
            if (kind == K_UNPACK) { const unsigned h = (gr & 1) * 8 + l; ro = piece(0, gr >> 1) + h * 8; ch = piece(1, gr >> 1) + h; w = in_word(piece(2, gr >> 1), h); }
            else {
                const unsigned sl = (gr < big ? gr * 8 : big * 4 + gr * 4) + l, n = (cp ? sl : sl >> 1) & 1023;
                ch = (cp ? 1 : 2 * quad + (l & 1)) * 1024 + n; ro = reco + ch * 8; w = kind | n << 3;
            }
            rec = __builtin_amdgcn_raw_buffer_load_b64(rctl, lane < 8 ? ro : lane == 8 ? OFF_TMO - 4 : OOB, 0, AUX_SYS);
            tk = __builtin_amdgcn_raw_ptr_buffer_atomic_add_i32(1, rctl, lane == 0 ? OFF_TKT + (kind == K_UNPACK ? si : cp ? 6 : 6 + quad) * 128 : OOB, 0, 0);
        };
        ask();
        for (unsigned k = 0;;) {
            const bool in = kind == K_UNPACK, end = kind == K_END;
            const unsigned cnt = in || gr < big ? 8 : 4, need = (1u << cnt) - 1;
            u32x4 d = {OOB, OOB, rec[0], K_END};
            if (!end) {
                const unsigned m = (unsigned)__ballot(rec[1] == call);
                dead |= (m >> 8) & 1;
                if ((m & need) != need) d[2] = records(myws, call, lane < 8 ? ro : OOB, need, dead);
                d[3] = dead ? w | 4 : w;
                if (!dead) { d[0] = (in ? pki : pko) + ch * 3072; if (is_raw(d[2])) d[1] = (in ? pki + PKI_MAIN : pko + PKO_MAIN) + ch * 1024; }
                gr = rfl(tk) - (call - 1) * (in ? NG_IN : ng_out) + (in ? NPW : cp ? 2 + CPY : 2 * NPW);
                ask();
            }
            for (Patience pt;;) {
                unsigned fr;
                if (rfl(lds_ld2(ctl + R_ISSUED, ctl + R_FREE + ((k + l) & 31) * 4, fr)) + AHEAD >= k && !__ballot(fr * 32 + 32 <= k + l)) break;
                if (pt.over(myws, call)) return;
            }
            if (lane < 8) *(LAS u32x4*)(lds + LDS_RING + R_DESC + ((k + l) & 31) * 16) = d;
            k += cnt;
            lds_st(ctl + R_AVAIL, k);
            if (end) { TOUT(13); return; }
        }
    }
    const rsrc_t rd = mk(myws, WS_NEED), rout = mk(p.out, IN_BYTES);
    unsigned owe = ~0u, owe_r0 = 0;
    auto announce = [&]() {
        if (owe != ~0u) {
            WAIT_V(0);
            const unsigned n = (owe >> 3) & 1023;
            if (cp) __builtin_amdgcn_raw_buffer_store_b64((u32x2){owe_r0, call}, rctl, lane == 0 ? reco + n * 8 : OOB, 0, AUX_SYS);
            else __builtin_amdgcn_raw_ptr_buffer_atomic_add_i32(1, rctl, lane == 0 ? OFF_RDY + n * 4 : OOB, 0, 0);
            owe = ~0u;
        }
    };
    auto draw = [&](unsigned n) -> unsigned {
        unsigned t = 0;
        if (lane == 0) t = __hip_atomic_fetch_add((LAS unsigned*)(lds + LDS_RING + R_NEXT), n, __ATOMIC_RELAXED, __HIP_MEMORY_SCOPE_WORKGROUP);
        return t;
    };
    auto look = [&](unsigned k) -> unsigned {
        return *(volatile LAS unsigned*)(lds + LDS_RING + (lane < 8 ? R_DESC + ((k * 16 + lane * 4) & 511) : (lane & 1) * 4));
    };
    auto release = [&](unsigned s) { *(volatile LAS unsigned*)(lds + LDS_RING + R_FREE + (s & 31) * 4) = (s >> 5) + 1; };
    unsigned k = rfl(draw(1)), d = look(k);
    for (;;) {
        TIC;
        for (Patience pt; rdl(d, 8) <= (k | (rdl(d, 3) & 1)); d = look(k)) {
            announce();
            if (pt.over(myws, call)) return;
        }
        if (k > 32 && k < 360) TOC(0);
        if (k >= rdl(d, 9)) break;
        const unsigned r0 = rdl(d, 2), r1 = rdl(d, 6), w = rdl(d, 3), kind = w & 3, n = (w >> 3) & 1023;
        const unsigned tk = draw(kind == K_SUM ? 2 : 1);
        LAS const char* in = lds + (k & 31) * 4096;
        auto next = [&]() { k = kind == K_SUM ? (rfl(tk) + 1) & ~1u : rfl(tk); d = look(k); announce(); };
        if (kind == K_COPY) {
            u32x4 P[4];
#pragma unroll
            for (int i = 0; i < 4; ++i) P[i] = *(LAS const u32x4*)(in + i * 1024 + lane * 16);
            FENCE; release(k); next();
            store_chunk(rd, pko + n * 3072, pko + PKO_MAIN + n * 1024, lane, P, r0);
            if (!(w & 4)) { owe = w; owe_r0 = r0; }
            continue;
        }
        if (kind & k & 1) { next(); continue; }
        unsigned D[16];
        decode(in, r0, cw, lane, D, [&]() { release(k); });
        if (kind == K_UNPACK) {
            next();
            store_in(myws, D, w, lane);
            if (!(w & 4)) owe = w;
            continue;
        }
        float v[32];
        add_chunk(v, D, true);
        decode(in + 4096, r1, cw, lane, D, [&]() { release(k + 1); });
        next();
        add_chunk(v, D, false);
        store_out(rout, (rank >> 1) ^ quad, n, v, lane, w & 4);
    }
    announce();
    if (wid == 2) TOUT(16);
}

__device__ __forceinline__ void reduce_own(const Params& p, char* myws, unsigned call, const Codec& cw, LAS char* lds, int wid, int lane) {
    const unsigned par = call & 1;
    const rsrc_t rctl = mk(myws, CTL_BYTES), rout = mk(p.out, IN_BYTES), rs = mk(myws + OFF_PKO + par * PKO_PAR, PKO_PAR);
    for (unsigned it = 0;; ++it) {
        LAS unsigned* bx = (LAS unsigned*)(lds + LDS_RING) + (it & 1) * 64;
        if (wid == 0) {
            const unsigned g = rfl(__builtin_amdgcn_raw_ptr_buffer_atomic_add_i32(1, rctl, lane == 0 ? OFF_TKT + 11 * 128 : OOB, 0, 0)) - (call - 1) * (128 + NGEMM - CPY);
            const unsigned s = lane >> 3, ro = lane < 40 ? OFF_RECO + ((par * 12 + (s ? 7 + s : 0)) * 1024 + g * 8 + (lane & 7)) * 8 : lane == 40 ? OFF_TMO - 4 : OOB;
            u32x2 rec = {0, 0};
            const unsigned long long t0 = __builtin_amdgcn_s_memrealtime();
            bool dead = false;
            for (unsigned sp = 0; g < 128 && !dead;) {
                rec = __builtin_amdgcn_raw_buffer_load_b64(rctl, ro, 0, AUX_SYS);
                const unsigned long long m = __ballot(rec[1] == call);
                dead = (m >> 40) & 1;
                if ((m & 0xFFFFFFFFFFull) == 0xFFFFFFFFFFull) break;
                if (!dead) dead = spin_fail<SLEEP_O>(myws, call, t0, sp);
            }
            bx[lane] = lane < 40 ? rec[0] : lane == 40 ? g : dead;
            WAIT_L(0);
        }
        BAR;
        FENCE;
        const unsigned bv = bx[lane], g = rdl(bv, 40), r0 = rdl(bv, wid), n = g * 8 + wid;
        const bool dead = rdl(bv, 41);
        if (g >= 128) break;
        load_chunk(rs, dead ? OOB : n * 3072, is_raw(r0) && !dead ? PKO_MAIN + n * 1024 : OOB, r0, lds + wid * 4096, lane);
        u32x4 Q[4][4];
#pragma unroll
        for (int s = 0; s < 4; ++s)
#pragma unroll
            for (int i = 0; i < 4; ++i)
                Q[s][i] = __builtin_amdgcn_raw_buffer_load_b128(rs, lane * 16, i < 3 ? ((8 + s) * 1024 + n) * 3072 + i * 1024 : PKO_MAIN + ((8 + s) * 1024 + n) * 1024, AUX_SYS);
        WAIT_V(0);
        unsigned D[16];
        float v[32];
        decode(lds + wid * 4096, r0, cw, lane, D, []() {});
        add_chunk(v, D, true);
#pragma unroll
        for (int s = 0; s < 4; ++s) {
#pragma unroll
            for (int i = 0; i < 16; ++i) D[i] = Q[s][i >> 2][i & 3];
            add_chunk(v, D, false);
        }
        store_out(rout, (unsigned)p.rank >> 1, n, v, lane, dead);
    }
}

__global__ __launch_bounds__(512, 2) void gemm_rs_kernel(const Params p) {
    extern __shared__ __attribute__((aligned(16))) char shm[];
    LAS char* lds = (LAS char*)shm;
    const int tid = threadIdx.x, wid = rfl(tid >> 6), lane = tid & 63;
    const unsigned b = blockIdx.x, rank = p.rank & 7;
    char* const myws = p.ws[rank];
    STAMP(0);
    const unsigned z = b & 7, slot = b >> 3, s2 = slot * 2 + (z & 1);
    const unsigned g0 = b >= NGEMM ? b - 28 : z >= 6 ? (slot < 12 ? s2 : 112 + s2) : z >= 2 ? (z < 4 ? 108 : 52) + (slot < 14 ? s2 : s2 - 56) : slot < 14 ? 1024 : 140 + s2;
    if (DELAY0 && g0 >= URGENT) __builtin_amdgcn_s_sleep(DELAY0);
    u32x4 R[4];
    pack_load(p, g0, wid, lane, R);
    if (tid < 64) *(LAS unsigned*)(lds + LDS_STAGE + 256 + tid * 4) = 0;
    if (tid < 64) *(LAS unsigned*)(lds + LDS_RING + tid * 4) = 0;
    *(LAS unsigned*)(lds + LDS_LUT + tid * 4) = lut_entry(tid);
    unsigned* callp = (unsigned*)(myws + OFF_CALL + b * 128);
    const unsigned call = rfl(__hip_atomic_load(callp, __ATOMIC_RELAXED, __HIP_MEMORY_SCOPE_AGENT)) + 1;
    const unsigned par = call & 1;
    const Codec cw = {lds + LDS_STAGE + 1024 + wid * CW_BYTES, lds + LDS_LUT};
    const unsigned en0 = p.order.e[lane], en1 = p.order.e[64 + lane];
    {
        unsigned G = g0, it = 0;
        unsigned long long jobs = 0x1834506CA8A8C4E4ull;
        WAIT_L(0); BAR;
        if (tid == 128) __hip_atomic_store(callp, call, __ATOMIC_RELAXED, __HIP_MEMORY_SCOPE_AGENT);
        while (G < 1024) {
            const unsigned Gn = g0 < ((jobs >> 8) & 255) ? G + ((unsigned)jobs & 255) : 1024;
            jobs >>= 8;
            unsigned D[16];
#pragma unroll
            for (int i = 0; i < 16; ++i) D[i] = R[i >> 2][i & 3];
            u32x4 P[4];
            const unsigned r0 = encode(D, pick_top(D), cw, lane, P);
            const PackJob j = pack_job(G, wid);
            if (G < 8 * NPUSH) {
                if (PDELAY && G >= 16) __builtin_amdgcn_s_sleep(PDELAY);
#pragma unroll
                for (int o = 0; o < 2; ++o) {
                    const unsigned h = j.hi * 4 + o * 2 + j.kk, c = 8192 + rank * 128 + (j.x ^ o) * 16 + (G & 1) * 8 + wid;
                    store_chunk(mk(p.ws[h & 7] + OFF_PKI + par * PKI_PAR, (h >> 1) != (rank >> 1) ? PKI_PAR : 0), c * 3072, PKI_MAIN + c * 1024, lane, P, r0);
                }
            } else store_chunk(mk(myws + OFF_PKI + par * PKI_PAR, PKI_PAR), j.gc * 3072, PKI_MAIN + j.gc * 1024, lane, P, r0);
            SCHED;
            pack_load(p, Gn, wid, lane, R);
            SCHED;
            WAIT_V(4);
            LAS u32x2* lrec = (LAS u32x2*)(lds + LDS_STAGE + (it & 1) * 64);
            if (lane == 0) lrec[wid] = (u32x2){r0, call};
            WAIT_L(0); BAR;
            if (wid < 2) {
                const unsigned h = j.mtx ? j.hi * 4 + wid * 2 + j.kk : wid * 4 + j.hi * 2 + j.kk;
                if ((h >> 1) != (rank >> 1) && lane < 8) {
                    const u32x2 r = lrec[lane];
                    __builtin_amdgcn_raw_buffer_store_b64(r, mk(p.ws[h & 7], CTL_BYTES), lane * 8, OFF_RECI + ((par * 8 + rank) * 2048 + j.rec_idx) * 8, AUX_SYS);
                }
            }
            if (it == INBOX_IT && g0 - INBOX_G0 < 12 * NPUSH) {
                const unsigned id = (g0 - INBOX_G0) * 8 + wid, si = id / (16 * NPUSH), jj = id % (16 * NPUSH), ch = 8192 + ((rank ^ (si + 2)) & 7) * 128 + jj;
                const Piece pc = describe(64 | jj >> 4, si, rank, par);
                bool dead = false;
                const unsigned q0 = rfl(records(myws, call, pc.f[0] + (jj & 15) * 8, 1, dead));
                load_chunk(mk(myws + OFF_PKI + par * PKI_PAR, PKI_PAR), ch * 3072, is_raw(q0) ? PKI_MAIN + ch * 1024 : OOB, q0, lds + wid * 4096, lane);
                __builtin_amdgcn_s_waitcnt(0x0F70);
                decode(lds + wid * 4096, q0, cw, lane, D, []() {});
                store_in(myws, D, in_word(pc.f[2], jj & 15), lane);
                WAIT_V(0);
                __builtin_amdgcn_raw_ptr_buffer_atomic_add_i32(1, mk(myws, CTL_BYTES), lane == 0 && !dead ? OFF_RDY + ((pc.f[2] >> 3) & 1023) * 4 : OOB, 0, 0);
            }
            G = Gn; ++it;
        }
    }
    STAMP(1);
    const unsigned gi = rank >> 2, gj = (rank >> 1) & 1;
    unsigned role = b - NGEMM;
    unsigned tile = g0 - 136, kq = 0;
    const int wr = wid >> 2, wc = wid & 3, fr = lane & 15, fq = lane >> 4;
    const unsigned voffA = (wid * 8 + (lane >> 3)) * 2048 + (((lane & 7) ^ ((wid * 4 + (lane >> 4)) & 7)) * 16);
    const unsigned voffB = (wid * 4 + (lane >> 4)) * 16384 + (((lane & 15) ^ (((lane >> 4) << 2) | (wid & 2))) * 16);
    const unsigned ldsw = wid * 1024;
    int aoff[2], boff[2];
#pragma unroll
    for (int k = 0; k < 2; ++k) aoff[k] = (wr * 64 + fr) * 128 + (((4 * k + fq) ^ ((fr >> 1) & 7)) * 16);
#pragma unroll
    for (int n = 0; n < 2; ++n) { const int q = (lane & 15) >> 2, pp = lane & 3;
        boff[n] = (int)(unsigned)(size_t)lds + 65536 + (8 * fq + q) * 256 + (((4 * wc + 2 * n + (pp >> 1)) ^ ((q << 2) | ((fq & 1) << 1))) * 16) + 8 * (pp & 1); }
    const rsrc_t rctl = mk(myws, CTL_BYTES);
#define AUX_G AUX_AGENT
#define HT 16384
#define SA(bb, h) (((bb) * 2 + (h)) * HT)
#define SB(bb, h) ((4 + (bb) * 2 + (h)) * HT)
#define STAGE_A(bb, h, rs, so) do { _Pragma("unroll") for (int _i = 0; _i < 2; ++_i) \
        __builtin_amdgcn_raw_ptr_buffer_load_lds(rs, (LAS void*)(lds + SA(bb, h) + ldsw + _i * 8192), 16, voffA, (so) + (h) * 262144 + _i * 131072, 0, AUX_G); } while (0)
#define STAGE_B(bb, h, rs, so) do { _Pragma("unroll") for (int _i = 0; _i < 2; ++_i) \
        __builtin_amdgcn_raw_ptr_buffer_load_lds(rs, (LAS void*)(lds + SB(bb, h) + ldsw + _i * 8192), 16, voffB, (so) + (h) * 256 + _i * 524288, 0, AUX_G); } while (0)
#define LDA(dst, bb, h) do { _Pragma("unroll") for (int m = 0; m < 4; ++m) _Pragma("unroll") for (int k = 0; k < 2; ++k) \
        dst[m][k] = *(const LAS bf16x8*)(lds + SA(bb, h) + aoff[k] + m * 2048); } while (0)
#define LDB(T, bb, h) asm volatile( \
        "ds_read_b64_tr_b16 %0, %8 offset:%10\n\tds_read_b64_tr_b16 %1, %8 offset:%11\n\tds_read_b64_tr_b16 %2, %8 offset:%12\n\tds_read_b64_tr_b16 %3, %8 offset:%13\n\t" \
        "ds_read_b64_tr_b16 %4, %9 offset:%10\n\tds_read_b64_tr_b16 %5, %9 offset:%11\n\tds_read_b64_tr_b16 %6, %9 offset:%12\n\tds_read_b64_tr_b16 %7, %9 offset:%13" \
        : "=&v"(T[0]), "=&v"(T[1]), "=&v"(T[2]), "=&v"(T[3]), "=&v"(T[4]), "=&v"(T[5]), "=&v"(T[6]), "=&v"(T[7]) \
        : "v"(boff[0]), "v"(boff[1]), "i"(SB(bb, h) - 65536), "i"(SB(bb, h) - 65536 + 1024), "i"(SB(bb, h) - 65536 + 8192), "i"(SB(bb, h) - 65536 + 9216) : "memory")
#define LDB_USE(dst, T) do { asm volatile("s_waitcnt lgkmcnt(0)" : "+v"(T[0]), "+v"(T[1]), "+v"(T[2]), "+v"(T[3]), "+v"(T[4]), "+v"(T[5]), "+v"(T[6]), "+v"(T[7]) :: "memory"); \
        _Pragma("unroll") for (int _j = 0; _j < 4; ++_j) dst[_j >> 1][_j & 1] = __builtin_shufflevector(T[2 * _j], T[2 * _j + 1], 0, 1, 2, 3, 4, 5, 6, 7); } while (0)
#define MMA(ai, bj, At, Bt) do { __builtin_amdgcn_s_setprio(1); _Pragma("unroll") for (int m = 0; m < 4; ++m) _Pragma("unroll") for (int n = 0; n < 2; ++n) _Pragma("unroll") for (int k = 0; k < 2; ++k) \
        acc[ai][bj][m][n] = __builtin_amdgcn_mfma_f32_16x16x32_bf16(Bt[n][k], At[m][k], acc[ai][bj][m][n], 0, 0, 0); __builtin_amdgcn_s_setprio(0); } while (0)
    constexpr unsigned AK = 128, BK = 1048576;
    for (int un = tile < 32 ? -1 : 0; un < 2 && b < NGEMM; ++un) {
        const unsigned d = un ? 0 : 1 + slot % 7, s0 = un ? kq * 8 : 0, nst = un ? 8 : 32;
        const unsigned ut = un ? tile & 1 : z & 1, uct = un ? (tile >> 1) & 15 : (z >> 1) * 4 + slot / 7;
        const unsigned rt = ut * 8 + d;
        const unsigned aown = ((rank ^ d) * 1024 + gi * 512 + ut * 256) * 2048, bown = (gj * 4096 + uct * 256) * 2;
        const unsigned arem = rt * 256 * 2048, brem = uct * 512;
        const unsigned rdy_off = OFF_RDY + (lane < 32 ? rt : 16 + uct) * 128 + (lane & 31) * 4;
        auto step_a = [&](unsigned s) -> unsigned { const unsigned r = s - 8, q = (r * 43) >> 8, si = r - q * 6;
            return s < 8 ? aown + s * 256 : arem + (si >> 1) * 8 * MiB + (si & 1) * 1024 + q * 256; };
        auto step_b = [&](unsigned s) -> unsigned { const unsigned r = s - 8, q = (r * 43) >> 8, si = r - q * 6;
            return s < 8 ? bown + s * 128 * 16384 : brem + ((si >> 1) * 512 + q * 128) * 16384 + (si & 1) * 8192; };
        LAS unsigned* lseq = (LAS unsigned*)(lds + LDS_STAGE + 256 + (un & 1) * 128);
        const unsigned want = un ? 0xFFu << s0 : 0xFFFFFFFFu;
        unsigned seen = want & 0xFFu, seqv = (lane < 8 && s0 == 0) ? lane + 1 : 0;
        auto get_step = [&](unsigned k) -> unsigned {
            unsigned e = rdl(seqv, k);
            if (e == 0) {
                if (wid == 0 && (unsigned)__builtin_popcount(seen) <= k) {
                    const unsigned long long t0 = __builtin_amdgcn_s_memrealtime();
                    unsigned fresh;
                    for (unsigned sp = 0;;) {
                        const unsigned v = __builtin_amdgcn_raw_buffer_load_b32(rctl, rdy_off, 0, AUX_AGENT);
                        const unsigned long long m = __ballot((int)(v - call * 16u) >= 0);
                        fresh = (((unsigned)m & (unsigned)(m >> 32)) << 8) & want & ~seen;
                        if (fresh) break;
                        if (spin_fail<SLEEP_G>(myws, call, t0, sp)) { fresh = want & ~seen; break; }
                    }
                    if (lane < 32 && ((fresh >> lane) & 1))
                        __hip_atomic_store(lseq + __builtin_popcount(seen) + __builtin_popcount(fresh & ((1u << lane) - 1)), (unsigned)lane + 1, __ATOMIC_RELEASE, __HIP_MEMORY_SCOPE_WORKGROUP);
                    seen |= fresh;
                }
                for (;;) {
                    seqv = __hip_atomic_load(lseq + (lane & 31), __ATOMIC_ACQUIRE, __HIP_MEMORY_SCOPE_WORKGROUP);
                    e = rdl(seqv, k);
                    if (e) break;
                    __builtin_amdgcn_s_sleep(1);
                }
                asm volatile("" ::: "memory");
            }
            return (e - 1) & 31;
        };
        f32x4 acc[2][2][4][2];
#pragma unroll
        for (int a = 0; a < 2; ++a)
#pragma unroll
            for (int bb = 0; bb < 2; ++bb)
#pragma unroll
                for (int m = 0; m < 4; ++m)
#pragma unroll
                    for (int n = 0; n < 2; ++n) acc[a][bb][m][n] = (f32x4){0.f, 0.f, 0.f, 0.f};
        bf16x8 At[4][2], B0[2][2], B1[2][2]; s16x4 TB[8];
        unsigned sn = get_step(0);
        rsrc_t ra = mk(sn < 8 ? p.x : myws + OFF_XS, sn < 8 ? IN_BYTES : XS_BYTES);
        unsigned sa = step_a(sn);
        {
            const rsrc_t rb = mk(sn < 8 ? p.w : myws + OFF_WS, sn < 8 ? IN_BYTES : WS_BYTES);
            const unsigned sb = step_b(sn);
            STAGE_B(0, 0, rb, sb); STAGE_A(0, 0, ra, sa); STAGE_B(0, 1, rb, sb); STAGE_A(0, 1, ra, sa);
            if (wr == 1) BAR;
            WAIT_V(4); BAR;
            STAGE_B(1, 0, rb, sb + BK); STAGE_A(1, 0, ra, sa + AK); STAGE_B(1, 1, rb, sb + BK);
            WAIT_V(6); BAR;
        }
        for (unsigned s = 0; s < nst; ++s) {
            if (s + 1 < nst) sn = get_step(s + 1);
            const rsrc_t rna = mk(sn < 8 ? p.x : myws + OFF_XS, sn < 8 ? IN_BYTES : XS_BYTES);
            const rsrc_t rnb = mk(sn < 8 ? p.w : myws + OFF_WS, sn < 8 ? IN_BYTES : WS_BYTES);
            const unsigned na = step_a(sn), nb = step_b(sn);
            LDB(TB, 0, 0); SCHED; LDA(At, 0, 0); STAGE_A(1, 1, ra, sa + AK);
            WAIT_L(8); BAR; LDB_USE(B0, TB); MMA(0, 0, At, B0); BAR; SCHED;
            LDB(TB, 0, 1); STAGE_B(0, 0, rnb, nb);
            BAR; LDB_USE(B1, TB); MMA(0, 1, At, B1); BAR;
            LDA(At, 0, 1); STAGE_A(0, 0, rna, na);
            BAR; WAIT_L(0); MMA(1, 0, At, B0); BAR; SCHED;
            STAGE_B(0, 1, rnb, nb);
            WAIT_V(6); BAR; MMA(1, 1, At, B1); BAR;
            LDB(TB, 1, 0); SCHED; LDA(At, 1, 0); STAGE_A(0, 1, rna, na);
            WAIT_L(8); BAR; LDB_USE(B0, TB); MMA(0, 0, At, B0); BAR; SCHED;
            LDB(TB, 1, 1); STAGE_B(1, 0, rnb, nb + BK);
            BAR; LDB_USE(B1, TB); MMA(0, 1, At, B1); BAR;
            LDA(At, 1, 1); STAGE_A(1, 0, rna, na + AK);
            BAR; WAIT_L(0); MMA(1, 0, At, B0); BAR; SCHED;
            STAGE_B(1, 1, rnb, nb + BK);
            WAIT_V(6); BAR; MMA(1, 1, At, B1); BAR;
            ra = rna; sa = na;
        }
        if (wr == 0) BAR;
        STAMP(4 + 2 * (un & 1));
        {
            const unsigned dst = (rank ^ d) & 7;
            const unsigned c0 = (un ? 8 + kq : d) * 1024 + uct * 64 + ut * 8 + wid;
            unsigned tk = 0;
            if (un == 0 && wid == 0) tk = __builtin_amdgcn_raw_ptr_buffer_atomic_add_i32(1, rctl, lane == 0 ? OFF_TKT + 10 * 128 : OOB, 0, 0);
            const rsrc_t rpk = mk(myws + OFF_PKO + par * PKO_PAR, PKO_PAR);
            unsigned top = 1, rp = 0;
            const unsigned tmo = __hip_atomic_load((unsigned*)(myws + OFF_TMO), __ATOMIC_RELAXED, __HIP_MEMORY_SCOPE_AGENT);
            auto publish = [&](unsigned c, unsigned r0) {
                __builtin_amdgcn_raw_buffer_store_b64((u32x2){r0, call}, mk(p.ws[dst], CTL_BYTES), tmo != call && lane == 0 ? OFF_RECO + (par * 12 * 1024 + c) * 8 : OOB, 0, AUX_SYS);
                SCHED;
            };
#pragma unroll
            for (int u = 0; u < 4; ++u) {
                unsigned D[16];
#pragma unroll
                for (int m = 0; m < 4; ++m)
#pragma unroll
                    for (int n = 0; n < 2; ++n) {
                        const f32x4 a = acc[u >> 1][u & 1][m][n];
                        D[m * 4 + n * 2] = cvt_pk_bf16(a[0], a[1]); D[m * 4 + n * 2 + 1] = cvt_pk_bf16(a[2], a[3]);
                    }
                u32x4 P[4];
                unsigned r0 = RAW << 16;
                if (un) {
#pragma unroll
                    for (int i = 0; i < 16; ++i) P[i >> 2][i & 3] = D[i];
                } else {
                    if (u == 0) top = pick_top(D);
                    r0 = encode(D, top, cw, lane, P);
                }
                const unsigned c = c0 + u * 16;
                SCHED;
                store_chunk(rpk, c * 3072, PKO_MAIN + c * 1024, lane, P, r0);
                SCHED;
                if (u == 1) WAIT_V(4);
                if (u > 1) WAIT_V(5);
                if (u) publish(c - 16, rp);
                rp = r0;
            }
            WAIT_V(1);
            publish(c0 + 48, rp);
            if (un == 0 && tid == 0) *(LAS unsigned*)(lds + LDS_STAGE + 128) = tk - (call - 1) * NGEMM + 32;
            WAIT_L(0);
        }
        BAR;
        STAMP(5 + 2 * (un & 1));
        if (un) continue;
        role = rfl(*(volatile LAS unsigned*)(lds + LDS_STAGE + 128));
        if (role < 32 + CPY) break;
        role += 256 - (32 + CPY);
        if (role >= 256 + 96) break;
        tile = (role - 256) / 3; kq = 1 + (role - 256) % 3;
    }
    if (role < 32 + CPY) ring(p, myws, call, lds, cw, role, en0, en1, wid, lane);
    else reduce_own(p, myws, call, cw, lds, wid, lane);
}

}

extern "C" void kernel_launch(int rank, int nranks, ncclComm_t comm, hipStream_t stream,
                              void* const* d_in, const int* in_sizes, int n_in,
                              void* d_out, int out_size,
                              const int* in_dtypes, int out_dtype,
                              void* const* d_out_all,
                              void* const* d_ws_all, size_t ws_bytes) {
    (void)comm; (void)in_dtypes; (void)out_dtype; (void)d_out_all;
    if (nranks != NR || (unsigned)rank >= (unsigned)NR || n_in < 2 || ws_bytes < WS_NEED || out_size != 1024 * 8192 ||
        in_sizes[0] != 8192 * 1024 || in_sizes[1] != 1024 * 8192) {
        fprintf(stderr, "kernel_launch: unexpected problem geometry\n");
        return;
    }
    HIP_CHECK(hipFuncSetAttribute((const void*)gemm_rs_kernel, hipFuncAttributeMaxDynamicSharedMemorySize, LDS_BYTES));
    Params p;
    p.x = (const char*)d_in[0]; p.w = (const char*)d_in[1]; p.out = (char*)d_out; p.rank = rank;
    constexpr Order order = make_order();
    p.order = order;
    for (int r = 0; r < NR; ++r) p.ws[r] = (char*)d_ws_all[r];
    hipLaunchKernelGGL(gemm_rs_kernel, dim3(256), dim3(512), LDS_BYTES, stream, p);
    HIP_CHECK(hipGetLastError());
}
```

```cpp
#include <hip/hip_runtime.h>
#include <rccl/rccl.h>
#include <cstdio>

#define HIP_CHECK(expr) do { hipError_t _e = (expr); if (_e != hipSuccess) { \
    fprintf(stderr, "HIP error %s at %s:%d\n", hipGetErrorString(_e), __FILE__, __LINE__); \
    return; } } while (0)

namespace {

#define LAS __attribute__((address_space(3)))
typedef short bf16x8 __attribute__((ext_vector_type(8)));
typedef short s16x4 __attribute__((ext_vector_type(4)));
typedef float f32x4 __attribute__((ext_vector_type(4)));
typedef unsigned u32x4 __attribute__((ext_vector_type(4)));
typedef unsigned u32x2 __attribute__((ext_vector_type(2)));
typedef unsigned short u16x2 __attribute__((ext_vector_type(2)));
typedef __amdgpu_buffer_rsrc_t rsrc_t;

constexpr int NR = 8;
constexpr unsigned MiB = 1u << 20;
constexpr unsigned OFF_CALL = 0;
constexpr unsigned OFF_TMO = 32768;
constexpr unsigned LSTR = 4352;
constexpr unsigned OFF_RDY = 786432;
constexpr unsigned OFF_TKT = OFF_RDY + 32 * LSTR;
static_assert(OFF_TKT + 12 * LSTR <= 1u << 20, "");
__device__ __forceinline__ unsigned rdy(unsigned n) { return OFF_RDY + (n >> 5) * LSTR + (n & 31) * 4; }
constexpr unsigned OFF_RECI = 65536;
constexpr unsigned OFF_RECO = 327680;
constexpr unsigned CTL_BYTES = MiB;
constexpr unsigned OFF_PKI = 1 * MiB, PKI_MAIN = 27 * MiB, PKI_PAR = 36 * MiB;
constexpr unsigned OFF_PKO = 73 * MiB, PKO_MAIN = 36 * MiB, PKO_PAR = 48 * MiB;
constexpr unsigned OFF_XS = 169 * MiB, XS_BYTES = 24 * MiB;
constexpr unsigned OFF_WS = 193 * MiB, WS_BYTES = 24 * MiB;
constexpr unsigned WS_NEED = 217 * MiB;
constexpr unsigned IN_BYTES = 16 * MiB;
constexpr unsigned OOB = 0x80000000u;
constexpr unsigned long long DEADLINE = 30000000ull;

constexpr int LDS_STAGE = 131072, LDS_LUT = LDS_STAGE + 1024 + 8 * 2128, LDS_RING = LDS_LUT + 2048, LDS_BYTES = LDS_RING + 1024;
constexpr int NGEMM = 224;
constexpr int NPW = 5;
constexpr int DEPTH = 4;
constexpr unsigned AHEAD = 4;
constexpr unsigned TAILG = 40;
constexpr unsigned CPY = 3;
constexpr unsigned GD = 120, GATE = 48;
constexpr int NPUSH = 4;
constexpr unsigned URGENT = 72;
constexpr int AUX_SYS = 17, AUX_AGENT = 16;
constexpr int SLOPE = 28, PAIR = 24, XSTEP = 22, GROUPT = 156;

__device__ __forceinline__ unsigned xpiece(unsigned x) { return (unsigned)(0x80A2FD75EC64B931ull >> (4 * x)) & 15; }
struct Order { unsigned char e[128]; };
constexpr Order make_order() {
    Order o{}; int key[128] = {}; int n = 0;
    for (int q = 0; q < 4; ++q) {
        for (int x = 0; x < 16; ++x) {
            o.e[n] = (unsigned char)(64 | q << 4 | x);
            key[n++] = (q == 0 && x < NPUSH) ? 200000 : x >= 14 ? 100000 + q : !q ? -1 : x < 2 ? q * GROUPT - 1 : q * GROUPT + 1 + (x - 2) / 2 * XSTEP; }
        for (int c = 0; c < 16; ++c) { o.e[n] = (unsigned char)(q << 4 | c); key[n++] = q * GROUPT + c * SLOPE - (c & 1) * PAIR; }
    }
    for (int i = 1; i < 128; ++i)
        for (int j = i; j > 0 && key[j - 1] > key[j]; --j) {
            const int k = key[j]; key[j] = key[j - 1]; key[j - 1] = k; const unsigned char e = o.e[j]; o.e[j] = o.e[j - 1]; o.e[j - 1] = e; }
    return o;
}

struct Params {
    const char* x; const char* w; char* out; char* ws[NR]; int rank; Order order;
};

__device__ __forceinline__ rsrc_t mk(const void* p, unsigned bytes) { return __builtin_amdgcn_make_buffer_rsrc((void*)p, 0, bytes, 0x00020000); }
__device__ __forceinline__ unsigned rfl(unsigned v) { return __builtin_amdgcn_readfirstlane(v); }
__device__ __forceinline__ unsigned rdl(unsigned v, unsigned l) { return __builtin_amdgcn_readlane(v, l); }
#define WAIT_V(n) asm volatile("s_waitcnt vmcnt(" #n ")" ::: "memory")
#define WAIT_L(n) asm volatile("s_waitcnt lgkmcnt(" #n ")" ::: "memory")
#define BAR __builtin_amdgcn_s_barrier()
#define SCHED __builtin_amdgcn_sched_barrier(0)


__device__ __forceinline__ unsigned cvt_pk_bf16(float lo, float hi) {
    unsigned r; asm("v_cvt_pk_bf16_f32 %0, %1, %2" : "=v"(r) : "v"(lo), "v"(hi)); return r;
}
__device__ __forceinline__ unsigned pk_sub(unsigned a, unsigned b) { return __builtin_bit_cast(unsigned, __builtin_bit_cast(u16x2, a) - __builtin_bit_cast(u16x2, b)); }
__device__ __forceinline__ unsigned bfi(unsigned m, unsigned a, unsigned b) { return (a & m) | (b & ~m); }
__device__ __forceinline__ unsigned rotr(unsigned a, unsigned n) { return __builtin_amdgcn_alignbit(a, a, n); }
template <int CTRL> __device__ __forceinline__ float dpp_f(float v) { return __builtin_bit_cast(float, __builtin_amdgcn_update_dpp(0, __builtin_bit_cast(int, v), CTRL, 0xf, 0xf, true)); }
__device__ __forceinline__ float rdl_f(float v, unsigned l) { return __builtin_bit_cast(float, rdl(__builtin_bit_cast(unsigned, v), l)); }

constexpr unsigned RAW = 255, EB = 1040, CW_BYTES = EB + 1024 + 64;
struct Codec { LAS char* cw; LAS const char* lut; };
__device__ __forceinline__ bool is_raw(unsigned r0) { return ((r0 >> 16) & 255) == RAW; }
__device__ __forceinline__ unsigned plane2_bytes(unsigned r0) { return 704 + 4 * ((r0 >> 16) & 255); }
__device__ __forceinline__ unsigned lut_entry(unsigned t) {
    unsigned k = 0, e = t < 256 ? 0 : 0x0C0C0C0Cu;
#pragma unroll
    for (unsigned b = 0; b < 4; ++b) {
        const unsigned c = (t >> (2 * b)) & 3;
        if (t < 256) e |= (c ? 4 + c : k) << (8 * b); else if (!c) e ^= (12 ^ b) << (8 * k);
        k += !c;
    }
    return e;
}
__device__ __forceinline__ unsigned scan_add(unsigned v) {
    int s = v;
    s += __builtin_amdgcn_update_dpp(0, s, 0x111, 0xf, 0xf, false); s += __builtin_amdgcn_update_dpp(0, s, 0x112, 0xf, 0xf, false);
    s += __builtin_amdgcn_update_dpp(0, s, 0x114, 0xf, 0xf, false); s += __builtin_amdgcn_update_dpp(0, s, 0x118, 0xf, 0xf, false);
    s += __builtin_amdgcn_update_dpp(0, s, 0x142, 0xa, 0xf, false); s += __builtin_amdgcn_update_dpp(0, s, 0x143, 0xc, 0xf, false);
    return s;
}
__device__ __forceinline__ unsigned pick_top(const unsigned (&D)[16]) {
    float sm = 0.f;
#pragma unroll
    for (int i = 0; i < 16; i += 4) sm += __builtin_fabsf(__builtin_bit_cast(float, D[i] << 16)) + __builtin_fabsf(__builtin_bit_cast(float, D[i] & 0xFFFF0000u));
    sm += dpp_f<0xB1>(sm); sm += dpp_f<0x4E>(sm); sm += dpp_f<0x141>(sm); sm += dpp_f<0x140>(sm);
    const float top = ((rdl_f(sm, 0) + rdl_f(sm, 16)) + (rdl_f(sm, 32) + rdl_f(sm, 48))) * (20.6f / 512.f);
    return (rfl(__builtin_bit_cast(unsigned, top) >> 15) & 0xFFFFu) | 1u;
}
#define FENCE asm volatile("" ::: "memory")
template <class Fn>
__device__ __forceinline__ unsigned encode(const unsigned (&D)[16], unsigned top, const Codec& k, int lane, u32x4 (&P)[4], Fn mid) {
    LAS char* const cw = k.cw;
    const unsigned top2 = top * 0x10001u;
    unsigned F[8], sel[8], C[2] = {0, 0};
#pragma unroll
    for (int i = 0; i < 8; ++i) {
        const unsigned a = pk_sub(top2, rotr(D[2 * i], 31)), b = pk_sub(top2, rotr(D[2 * i + 1], 31));
        P[i >> 2][i & 3] = __builtin_amdgcn_perm(b, a, 0x06040200u);
        F[i] = __builtin_amdgcn_perm(b, a, 0x07050301u);
        const unsigned c = __builtin_amdgcn_perm(0x00000302u, 0x01000000u, F[i]);
        const unsigned cb = __builtin_amdgcn_udot4((c ^ (c >> 2)) & 0x03030303u, 0x40100401u, 0, false);
        sel[i] = *(LAS const unsigned*)(k.lut + 1024 + cb * 4);
        C[i >> 2] |= cb << (8 * (i & 3));
    }
    const unsigned m[2] = {~(C[0] | C[0] >> 1) & 0x55555555u, ~(C[1] | C[1] >> 1) & 0x55555555u};
    const unsigned k0 = __builtin_popcount(m[0]), kk = k0 + __builtin_popcount(m[1]), incl = scan_add(kk), cnt = rdl(incl, 63);
    P[2] = (u32x4){F[0], F[1], F[2], F[3]}; P[3] = (u32x4){F[4], F[5], F[6], F[7]};
    if (cnt > 1024) { mid(); return top | RAW << 16; }
    const unsigned fill = lane * 16u >= cnt ? 0x02020202u : 0;
    *(LAS u32x4*)(cw + EB + lane * 16) = (u32x4){fill, fill, fill, fill};
    FENCE;
    *(LAS unsigned char*)(cw + EB + cnt + lane) = 2;
    *(LAS u32x2*)(cw + lane * 8) = (u32x2){C[0], C[1]};
    FENCE;
    const unsigned e1 = (unsigned)(size_t)cw + EB - 1 + incl;
    unsigned carry = 0;
#pragma unroll
    for (int i = 0; i < 8; ++i) {
        const unsigned p = e1 - kk + (i >= 4 ? k0 : 0) + __builtin_popcount(m[i >> 2] & ((1u << (8 * (i & 3))) - 1)), pk = __builtin_amdgcn_perm(0u, F[i], sel[i]);
        __hip_atomic_fetch_or((LAS unsigned*)(size_t)(p & ~3u), __builtin_amdgcn_alignbyte(pk, 0u, ~p) | carry, __ATOMIC_RELAXED, __HIP_MEMORY_SCOPE_WAVEFRONT);
        carry = __builtin_amdgcn_alignbyte(0u, pk, ~p);
    }
    __hip_atomic_fetch_or((LAS unsigned*)(size_t)(e1 & ~3u), carry, __ATOMIC_RELAXED, __HIP_MEMORY_SCOPE_WAVEFRONT);
    FENCE;
    mid();
    unsigned M = 0, ovf = 0;
    auto dense = [&](int r) -> unsigned {
        const u32x2 f = *(LAS u32x2*)(cw + EB + r * 512 + lane * 8);
        const unsigned t0 = __builtin_amdgcn_perm(0u, 0x00074040u, f[0]), t1 = __builtin_amdgcn_perm(0u, 0x00074040u, f[1]);
        M |= bfi(0x40404040u, t0, t1 << 1) >> (6 - 2 * r) & 0x03030303u << (2 * r);
        return ((f[0] ^ t0) & 0x07070707u) | ((f[1] ^ t1) & 0x07070707u) << 3;
    };
    const unsigned x = dense(0), y = bfi(0xC0C0C0u, __umul24(x >> 24, 0x41040u), x);
    *(LAS unsigned short*)(cw + 512 + lane * 2) = (unsigned short)y;
    *(LAS unsigned char*)(cw + 640 + lane) = (unsigned char)(y >> 16);
    if (cnt > 512) { *(LAS unsigned*)(cw + 704 + lane * 4) = dense(1); ovf = (cnt - 505) >> 3; }
    FENCE;
    unsigned n = 0;
    for (;;) {
        const unsigned long long act = __ballot(M != 0);
        if (!act) break;
        if (M) {
            const unsigned off = ovf + n + __builtin_amdgcn_mbcnt_hi((unsigned)(act >> 32), __builtin_amdgcn_mbcnt_lo((unsigned)act, 0));
            const unsigned bt = __builtin_ctz(M), pos = (bt & 2) * 256 + lane * 8 + (bt & 1) * 4 + (bt >> 3);
            *(LAS unsigned*)(cw + 704 + (off < 80 ? off : 80) * 4) = pos | (unsigned)*(LAS unsigned char*)(cw + EB + pos) << 16;
            M &= M - 1;
        }
        n += __builtin_popcountll(act);
        if (n > 64 || ovf + n > 80) return top | RAW << 16;
    }
    FENCE;
    P[2] = *(LAS u32x4*)(cw + lane * 16);
    FENCE;
    return top | (ovf + n) << 16 | n << 24;
}
template <class Fn>
__device__ __forceinline__ void decode(LAS const char* in, unsigned r0, const Codec& k, int lane, unsigned (&D)[16], Fn done) {
    LAS char* const cw = k.cw;
    LAS const char* const p2 = in + 2048;
    const unsigned n = r0 >> 24, ovf = ((r0 >> 16) & 255) - n, top2 = (r0 & 0xFFFFu) * 0x10001u;
    unsigned F[8];
    u32x4 L[2];
    FENCE;
    if (is_raw(r0)) {
        const u32x4 f0 = *(LAS const u32x4*)(p2 + lane * 16), f1 = *(LAS const u32x4*)(p2 + 1024 + lane * 16);
        L[0] = *(LAS const u32x4*)(in + lane * 16); L[1] = *(LAS const u32x4*)(in + 1024 + lane * 16);
        FENCE; done(); FENCE;
#pragma unroll
        for (int i = 0; i < 4; ++i) { F[i] = f0[i]; F[i + 4] = f1[i]; }
    } else {
        const u32x2 C = *(LAS const u32x2*)(p2 + lane * 8);
        const unsigned y = *(LAS const unsigned short*)(p2 + 512 + lane * 2) | (unsigned)*(LAS const unsigned char*)(p2 + 640 + lane) << 16;
        const unsigned xo = *(LAS const unsigned*)(p2 + 704 + lane * 4), e = *(LAS const unsigned*)(p2 + 704 + ((ovf + lane) & 127) * 4);
        L[0] = *(LAS const u32x4*)(in + lane * 16); L[1] = *(LAS const u32x4*)(in + 1024 + lane * 16);
        FENCE; done(); FENCE;
        auto spread = [&](int r, unsigned x) {
            *(LAS u32x2*)(cw + EB + r * 512 + lane * 8) = (u32x2){__builtin_amdgcn_perm(0x0706020Cu, 0x0B0A0908u, x & 0x07070707u),
                                                                  __builtin_amdgcn_perm(0x0706020Cu, 0x0B0A0908u, (x >> 3) & 0x07070707u)};
        };
        unsigned sel[8];
#pragma unroll
        for (int i = 0; i < 8; ++i) sel[i] = *(LAS const unsigned*)(k.lut + ((C[i >> 2] >> (8 * (i & 3))) & 255) * 4);
        spread(0, bfi(0x3F000000u, __umul24(y & 0xC0C0C0u, 0x41040u), y));
        if (ovf) spread(1, xo);
        FENCE;
        *(LAS unsigned char*)(cw + ((unsigned)lane < n ? EB + (e & 1023) : 1024)) = (unsigned char)(e >> 16);
        FENCE;
        const unsigned m[2] = {~(C[0] | C[0] >> 1) & 0x55555555u, ~(C[1] | C[1] >> 1) & 0x55555555u};
        const unsigned k0 = __builtin_popcount(m[0]), kk = k0 + __builtin_popcount(m[1]), p0 = (unsigned)(size_t)cw + EB + scan_add(kk) - kk;
#pragma unroll
        for (int i = 0; i < 8; ++i) {
            const unsigned p = p0 + (i >= 4 ? k0 : 0) + __builtin_popcount(m[i >> 2] & ((1u << (8 * (i & 3))) - 1));
            const LAS unsigned* w = (const LAS unsigned*)(size_t)(p & ~3u);
            F[i] = __builtin_amdgcn_perm(0x05040300u, __builtin_amdgcn_alignbyte(w[1], w[0], p), sel[i]);
        }
        FENCE;
    }
#pragma unroll
    for (int i = 0; i < 8; ++i) {
        D[2 * i] = rotr(pk_sub(top2, __builtin_amdgcn_perm(F[i], L[i >> 2][i & 3], 0x05010400u)), 1);
        D[2 * i + 1] = rotr(pk_sub(top2, __builtin_amdgcn_perm(F[i], L[i >> 2][i & 3], 0x07030602u)), 1);
    }
}
__device__ __forceinline__ void store_chunk(rsrc_t rs, unsigned main_off, unsigned esc_off, int lane, const u32x4 (&P)[4], unsigned r0) {
#pragma unroll
    for (int i = 0; i < 2; ++i) __builtin_amdgcn_raw_buffer_store_b128(P[i], rs, main_off + lane * 16 + i * 1024, 0, AUX_SYS);
    __builtin_amdgcn_raw_buffer_store_b128(P[2], rs, lane * 16u < plane2_bytes(r0) ? main_off + lane * 16 + 2048 : OOB, 0, AUX_SYS);
    __builtin_amdgcn_raw_buffer_store_b128(P[3], rs, is_raw(r0) ? esc_off + lane * 16 : OOB, 0, AUX_SYS);
}
__device__ __forceinline__ void load_chunk(rsrc_t rs, unsigned main_off, unsigned esc_off, unsigned r0, LAS char* to, int lane) {
    if (esc_off != OOB) __builtin_amdgcn_raw_ptr_buffer_load_lds(rs, (LAS void*)(to + 3072), 16, lane * 16, esc_off, 0, AUX_SYS);
    __builtin_amdgcn_raw_ptr_buffer_load_lds(rs, (LAS void*)to, 16, lane * 16, main_off, 0, AUX_SYS);
    __builtin_amdgcn_raw_ptr_buffer_load_lds(rs, (LAS void*)(to + 1024), 16, lane * 16 + 1024, main_off, 0, AUX_SYS);
    __builtin_amdgcn_raw_ptr_buffer_load_lds(rs, (LAS void*)(to + 2048), 16, lane * 16u < plane2_bytes(r0) ? lane * 16 + 2048 : OOB, main_off, 0, AUX_SYS);
}

template <int SLEEP>
__device__ __forceinline__ bool spin_fail(char* myws, unsigned call, unsigned long long t0, unsigned& n) {
    unsigned* tmo = (unsigned*)(myws + OFF_TMO);
    if ((++n & 63) == 0 &&
        (rfl(__hip_atomic_load(tmo, __ATOMIC_RELAXED, __HIP_MEMORY_SCOPE_AGENT)) == call || __builtin_amdgcn_s_memrealtime() - t0 > DEADLINE)) {
        __hip_atomic_store(tmo, call, __ATOMIC_RELAXED, __HIP_MEMORY_SCOPE_AGENT);
        return true;
    }
    __builtin_amdgcn_s_sleep(SLEEP);
    return false;
}

struct PackJob { unsigned src_off, stride, gc, rec_idx, x; int mtx, kk, hi; };
__device__ __forceinline__ PackJob pack_job(unsigned G, int wid) {
    PackJob j; const unsigned q = G >> 8, ch = (G & 1) * 8 + wid;
    j.mtx = ((G >> 7) & 1) ^ 1; j.x = (G >> 3) & 15; j.hi = (G >> 2) & 1; j.kk = (G >> 1) & 1; const unsigned ks = j.kk * 4 + q;
    unsigned t16, tile;
    if (j.mtx) { const unsigned e = xpiece(j.x), sh = (j.hi * 4 + j.kk) ^ (e & 7), t = e >> 3;
        tile = sh * 4 + j.hi * 2 + t; t16 = sh * 2 + t; j.src_off = (tile * 256 + ch * 16) * 2048 + ks * 256; j.stride = 2048; }
    else { tile = j.hi * 16 + j.x; t16 = j.x; j.src_off = (ks * 128 + (ch >> 1) * 16) * 16384 + tile * 512 + (ch & 1) * 256; j.stride = 16384; }
    j.gc = (j.mtx * 256 + tile * 8 + ks) * 16 + ch;
    j.rec_idx = (j.mtx * 64 + t16 * 4 + q) * 16 + (G & 1) * 8;
    return j;
}
__device__ __forceinline__ void pack_load(const Params& p, unsigned G, int wid, int lane, u32x4 (&R)[4]) {
    const PackJob j = pack_job(G < 1024 ? G : 0, wid);
    const rsrc_t rs = mk(j.mtx ? p.x : p.w, G < 1024 ? IN_BYTES : 0);
    const unsigned vo = (lane >> 4) * j.stride + (lane & 15) * 16;
#pragma unroll
    for (int i = 0; i < 4; ++i) R[i] = __builtin_amdgcn_raw_buffer_load_b128(rs, vo, j.src_off + i * 4 * j.stride, 0);
}

__device__ __forceinline__ void add_chunk(float (&v)[32], const unsigned (&D)[16], bool first) {
#pragma unroll
    for (int i = 0; i < 16; ++i) {
        const float a = __builtin_bit_cast(float, D[i] << 16), b = __builtin_bit_cast(float, D[i] & 0xFFFF0000u);
        v[2 * i] = first ? a : v[2 * i] + a; v[2 * i + 1] = first ? b : v[2 * i + 1] + b;
    }
}
__device__ __forceinline__ void store_out(rsrc_t rout, unsigned pr, unsigned n, const float (&v)[32], int lane, bool dead) {
    const unsigned t = (n >> 3) & 1, ct = ((n >> 6) & 14) | ((n >> 4) & 1), w = n & 7, u = (n >> 5) & 3;
    const int fr = lane & 15, fq = lane >> 4;
    const unsigned vo = fr * 16384 + ((fq & 1) * 16 + (fq >> 1) * 8) * 2;
    const unsigned so = ((pr >> 1) * 512 + t * 256 + (u >> 1) * 128 + (w >> 2) * 64) * 16384 + ((pr & 1) * 4096 + ct * 256 + (u & 1) * 128 + (w & 3) * 32) * 2;
#pragma unroll
    for (int m = 0; m < 4; ++m) {
        unsigned d[4];
#pragma unroll
        for (int h = 0; h < 4; ++h) {
            const float a = v[8 * m + 2 * h], b = v[8 * m + 2 * h + 1];
            d[h] = cvt_pk_bf16(a < 0.f ? 0.f : a, b < 0.f ? 0.f : b);
        }
        const auto s0 = __builtin_amdgcn_permlane16_swap(d[0], d[2], false, false);
        const auto s1 = __builtin_amdgcn_permlane16_swap(d[1], d[3], false, false);
        u32x4 o = {s0[0], s1[0], s0[1], s1[1]};
        if (dead) o = (u32x4){0x7FC07FC0u, 0x7FC07FC0u, 0x7FC07FC0u, 0x7FC07FC0u};
        __builtin_amdgcn_raw_buffer_store_b128(o, rout, vo + so + m * 16 * 16384, 0, AUX_AGENT);
    }
}

enum { K_UNPACK = 0, K_SUM = 1, K_COPY = 2, K_END = 3 };
enum { R_FULL = 0, R_ISSUED = 4, R_AVAIL = 8, R_NEXT = 12, R_FREE = 128, R_DESC = 256 };
__device__ __forceinline__ unsigned lds_ld2(unsigned a, unsigned b, unsigned& vb) {
    unsigned v; asm volatile("ds_read_b32 %0, %2\n\tds_read_b32 %1, %3\n\ts_waitcnt lgkmcnt(0)" : "=&v"(v), "=&v"(vb) : "v"(a), "v"(b) : "memory"); return v; }
__device__ __forceinline__ void lds_st(unsigned a, unsigned v) { asm volatile("ds_write_b32 %0, %1" :: "v"(a), "v"(v) : "memory"); }
struct Patience {
    unsigned n = 0; unsigned long long t0 = 0;
    __device__ __forceinline__ bool over(char* myws, unsigned call) {
        if ((n++ & 1023) == 0) {
            const unsigned long long t = __builtin_amdgcn_s_memrealtime();
            if (n == 1) t0 = t;
            if (t - t0 > 2 * DEADLINE) { __hip_atomic_store((unsigned*)(myws + OFF_TMO), call, __ATOMIC_RELAXED, __HIP_MEMORY_SCOPE_AGENT); return true; }
        }
        __builtin_amdgcn_s_sleep(1);
        return false;
    }
};
struct Piece { unsigned f[3]; };
__device__ __forceinline__ Piece describe(unsigned en, unsigned si, unsigned rank, unsigned par) {
    const unsigned mtx = (en >> 6) & 1, q = (en >> 4) & 3, ct = en & 15, g = (rank ^ (si + 2)) & 7;
    const unsigned y = (ct + 10 + 2 * si) % 12, x = !q || ct >= 14 ? ct : ct < 2 ? 2 * ct : y ? y + 2 : 1;
    const unsigned e = xpiece(x), t = e >> 3, d = e & 7, sh = rank ^ d, ks = (rank & 1) * 4 + q;
    const unsigned tile32 = mtx ? sh * 4 + (rank >> 2) * 2 + t : (rank & 2) * 8 + ct, t16 = mtx ? sh * 2 + t : ct;
    const unsigned dst = mtx ? (si >> 1) * 8 * MiB + (t * 8 + d) * 256 * 2048 + (si & 1) * 1024 + q * 256
                             : XS_BYTES + ((si >> 1) * 512 + q * 128) * 16384 + (si & 1) * 8192 + ct * 512;
    return {{OFF_RECI + ((par * 8 + g) * 2048 + (mtx * 64 + t16 * 4 + q) * 16) * 8, (mtx * 256 + tile32 * 8 + ks) * 16,
             dst << 5 | ((mtx ? t * 8 + d : 16 + ct) * 32 + q * 6 + si) << 3}};
}
__device__ __forceinline__ unsigned in_word(unsigned w, unsigned ch) { return w + ((w < XS_BYTES << 5 ? ch * 32768 : (ch >> 1) * 262144 + (ch & 1) * 256) << 5); }
__device__ __forceinline__ unsigned records(char* myws, unsigned call, unsigned ro, unsigned need, bool& dead) {
    const unsigned long long t0 = __builtin_amdgcn_s_memrealtime();
    for (unsigned sp = 0; !dead;) {
        const u32x2 rec = __builtin_amdgcn_raw_buffer_load_b64(mk(myws, CTL_BYTES), ro, 0, AUX_SYS);
        if (((unsigned)__ballot(rec[1] == call) & need) == need) return rec[0];
        dead = spin_fail<4>(myws, call, t0, sp);
    }
    return 0;
}
__device__ __forceinline__ void store_in(char* myws, const unsigned (&D)[16], unsigned w, int lane) {
    const unsigned dst = w >> 13 << 8, st = dst < XS_BYTES ? 2048 : 16384, vo = (lane >> 4) * st + (lane & 15) * 16 + dst;
#pragma unroll
    for (int i = 0; i < 4; ++i)
        __builtin_amdgcn_raw_buffer_store_b128((u32x4){D[4 * i], D[4 * i + 1], D[4 * i + 2], D[4 * i + 3]}, mk(myws + OFF_XS, XS_BYTES + WS_BYTES), vo + i * 4 * st, 0, AUX_AGENT);
}
__device__ __forceinline__ void ring(const Params& p, char* myws, unsigned call, LAS char* lds, const Codec& cw, unsigned c, unsigned en0, unsigned en1, int wid, int lane) {
    const unsigned rank = p.rank & 7, par = call & 1;
    const bool cp = c >= 30;
    const unsigned si = c % 6, g = (rank ^ (cp ? 1 : si + 2)) & 7, quad = 1 + (si >> 1), ctl = (unsigned)(size_t)lds + LDS_RING;
    const unsigned pki = OFF_PKI + par * PKI_PAR, pko = OFF_PKO + par * PKO_PAR, reco = OFF_RECO + par * 12 * 1024 * 8;
    const rsrc_t rctl = mk(myws, CTL_BYTES);
    if (wid == 0) {
        __builtin_amdgcn_s_setprio(3);
        unsigned long long src0 = (unsigned long long)p.ws[g], src1 = (unsigned long long)p.ws[g ^ 1];
        asm volatile("" : "+s"(src0), "+s"(src1));
        unsigned av = 0, k0 = 0, d = 0, lim = 0;
        unsigned long long raw = 0;
        int full = 0;
        auto post = [&](unsigned issued) { asm volatile("ds_write_b64 %0, %1" :: "v"(ctl + R_FULL), "v"((u32x2){(unsigned)full, issued}) : "memory"); };
        for (unsigned k = 0;; k += 4) {
            if (av <= k) {
                k0 = k;
                for (Patience pt;;) {
                    const unsigned q = rfl(lds_ld2(ctl + R_AVAIL, ctl + R_DESC + ((k * 16 + lane * 4) & 511), d));
                    av = q < k + 16 ? q : k + 16;
                    if (av > k) break;
                    if (full < (int)k) { WAIT_V(0); full = k; post(k); } else if (pt.over(myws, call)) return;
                }
                lim = plane2_bytes(d); raw = __ballot((lane & 3) == 1 && d != OOB);
            }
            const unsigned l = (k - k0) * 4, w = rdl(d, l + 3);
            if ((w & 3) == K_END) {
                WAIT_V(6); full = max(full, (int)k - 2); post(k);
                __builtin_amdgcn_s_waitcnt(0x0F70);
                full = 0x7FFFFFFF; post(k);
                break;
            }
            const rsrc_t rs[2] = {mk((void*)(w & si & 1 ? src1 : src0), WS_NEED), mk((void*)(w & ~si & 1 ? src1 : src0), WS_NEED)};
            LAS char* const to = lds + (k & 31) * 4096;
            if ((raw >> l) & 0xFFFF)
#pragma unroll
                for (int i = 0; i < 4; ++i) __builtin_amdgcn_raw_ptr_buffer_load_lds(rs[i & 1], (LAS void*)(to + i * 4096 + 3072), 16, lane * 16, rdl(d, l + 4 * i + 1), 0, AUX_SYS);
#pragma unroll
            for (int i = 0; i < 4; ++i) {
                const unsigned o0 = rdl(d, l + 4 * i), v2 = lane * 16u < rdl(lim, l + 4 * i + 2) ? lane * 16 : OOB;
                SCHED;
                asm volatile("s_waitcnt vmcnt(%0)" :: "i"(3 * (DEPTH - 1)) : "memory");
                __builtin_amdgcn_raw_ptr_buffer_load_lds(rs[i & 1], (LAS void*)(to + i * 4096), 16, lane * 16, o0, 0, AUX_SYS);
                __builtin_amdgcn_raw_ptr_buffer_load_lds(rs[i & 1], (LAS void*)(to + i * 4096), 16, lane * 16, o0, 1024, AUX_SYS);
                __builtin_amdgcn_raw_ptr_buffer_load_lds(rs[i & 1], (LAS void*)(to + i * 4096), 16, v2, o0, 2048, AUX_SYS);
                SCHED;
                full = max(full, (int)k + i + 1 - DEPTH);
                post(k + i + 1);
            }
        }
    }
    if (wid == 1) {
        const Piece tb[2] = {describe(en0, si, rank, par), describe(en1, si, rank, par)};
        auto piece = [&](int f, unsigned pc) -> unsigned { const unsigned a = rdl(tb[0].f[f], pc & 63), b = rdl(tb[1].f[f], pc & 63); return pc & 64 ? b : a; };
        constexpr unsigned NG_IN = (128 - NPUSH) * 2;
        const unsigned big = (cp ? GD : 256) - TAILG / 2, ng_out = big + TAILG, l = lane & 7;
        unsigned kind = cp ? K_COPY : K_UNPACK, gr = cp ? c - 30 : c / 6, tk = 0, ro = OOB, ch = 0, w = 0;
        u32x2 rec = {0, 0};
        bool dead = false;
        auto ask = [&]() {
            if (kind == K_UNPACK && gr >= NG_IN) { kind = K_SUM; gr = (c / 6) * 2 + (si & 1); }
            if (kind != K_UNPACK && gr >= ng_out) { kind = K_END; return; }
            if (kind == K_UNPACK) { const unsigned h = (gr & 1) * 8 + l; ro = piece(0, gr >> 1) + h * 8; ch = piece(1, gr >> 1) + h; w = in_word(piece(2, gr >> 1), h); }
            else {
                const unsigned sl = (gr < big ? gr * 8 : big * 4 + gr * 4) + l, n = (cp ? sl : sl >> 1) & 1023;
                ch = (cp ? 1 : 2 * quad + (l & 1)) * 1024 + n; ro = reco + ch * 8; w = kind | n << 3;
            }
            rec = __builtin_amdgcn_raw_buffer_load_b64(rctl, lane < 8 ? ro : lane == 8 ? OFF_TMO - 4 : OOB, 0, AUX_SYS);
            tk = __builtin_amdgcn_raw_ptr_buffer_atomic_add_i32(1, rctl, lane == 0 ? OFF_TKT + (kind == K_UNPACK ? si : cp ? 6 : 6 + quad) * LSTR : OOB, 0, 0);
        };
        ask();
        for (unsigned k = 0;;) {
            const bool in = kind == K_UNPACK, end = kind == K_END;
            const unsigned cnt = in || gr < big ? 8 : 4, need = (1u << cnt) - 1;
            u32x4 d = {OOB, OOB, rec[0], K_END};
            if (!end) {
                const unsigned m = (unsigned)__ballot(rec[1] == call);
                dead |= (m >> 8) & 1;
                if ((m & need) != need) d[2] = records(myws, call, lane < 8 ? ro : OOB, need, dead);
                d[3] = dead ? w | 4 : w;
                if (!dead) { d[0] = (in ? pki : pko) + ch * 3072; if (is_raw(d[2])) d[1] = (in ? pki + PKI_MAIN : pko + PKO_MAIN) + ch * 1024; }
                gr = rfl(tk) - (call - 1) * (in ? NG_IN : ng_out) + (in ? NPW : cp ? 2 + CPY : 2 * NPW);
                ask();
            }
            for (Patience pt;;) {
                unsigned fr;
                const unsigned s = k + l;
                if (rfl(lds_ld2(ctl + R_ISSUED, ctl + R_FREE + (s & 31) * 4, fr)) + AHEAD >= k && !__ballot(fr * 32 + 32 <= s)) break;
                if (pt.over(myws, call)) return;
            }
            if (lane < 8) *(LAS u32x4*)(lds + LDS_RING + R_DESC + ((k + l) & 31) * 16) = d;
            k += cnt;
            lds_st(ctl + R_AVAIL, k);
            if (end) break;
        }
    }
    const rsrc_t rd = mk(myws, WS_NEED), rout = mk(p.out, IN_BYTES);
    unsigned owe = ~0u, owe_r0 = 0;
    auto announce = [&]() {
        if (owe != ~0u) {
            WAIT_V(0);
            const unsigned n = (owe >> 3) & 1023;
            if (cp) __builtin_amdgcn_raw_buffer_store_b64((u32x2){owe_r0, call}, rctl, lane == 0 ? reco + n * 8 : OOB, 0, AUX_SYS);
            else __builtin_amdgcn_raw_ptr_buffer_atomic_add_i32(1, rctl, lane == 0 ? rdy(n) : OOB, 0, 0);
            owe = ~0u;
        }
    };
    auto draw = [&](unsigned n) -> unsigned {
        unsigned t = 0;
        if (lane == 0) t = __hip_atomic_fetch_add((LAS unsigned*)(lds + LDS_RING + R_NEXT), n, __ATOMIC_RELAXED, __HIP_MEMORY_SCOPE_WORKGROUP);
        return t;
    };
    auto look = [&](unsigned k) -> unsigned {
        return *(volatile LAS unsigned*)(lds + LDS_RING + (lane < 8 ? R_DESC + ((k * 16 + lane * 4) & 511) : (lane & 1) * 4));
    };
    auto release = [&](unsigned s) { *(volatile LAS unsigned*)(lds + LDS_RING + R_FREE + (s & 31) * 4) = (s >> 5) + 1; };
    unsigned k = rfl(draw(1)), d = look(k);
    for (;;) {
        for (Patience pt; rdl(d, 8) <= (k | (rdl(d, 3) & 1)); d = look(k)) {
            announce();
            if (pt.over(myws, call)) return;
        }
        if (k >= rdl(d, 9)) break;
        const unsigned r0 = rdl(d, 2), r1 = rdl(d, 6), w = rdl(d, 3), kind = w & 3, n = (w >> 3) & 1023;
        const unsigned tk = draw(kind == K_SUM ? 2 : 1);
        LAS const char* in = lds + (k & 31) * 4096;
        auto next = [&]() { k = kind == K_SUM ? (rfl(tk) + 1) & ~1u : rfl(tk); d = look(k); announce(); };
        if (kind == K_COPY) {
            u32x4 P[4];
#pragma unroll
            for (int i = 0; i < 4; ++i) P[i] = *(LAS const u32x4*)(in + i * 1024 + lane * 16);
            FENCE; release(k); next();
            store_chunk(rd, pko + n * 3072, pko + PKO_MAIN + n * 1024, lane, P, r0);
            if (!(w & 4)) { owe = w; owe_r0 = r0; }
            continue;
        }
        if (kind & k & 1) { next(); continue; }
        unsigned D[16];
        decode(in, r0, cw, lane, D, [&]() { release(k); });
        if (kind == K_UNPACK) {
            next();
            store_in(myws, D, w, lane);
            if (!(w & 4)) owe = w;
            continue;
        }
        float v[32];
        add_chunk(v, D, true);
        decode(in + 4096, r1, cw, lane, D, [&]() { release(k + 1); });
        next();
        add_chunk(v, D, false);
        store_out(rout, (rank >> 1) ^ quad, n, v, lane, w & 4);
    }
    announce();
}

__device__ __forceinline__ void reduce_own(const Params& p, char* myws, unsigned call, const Codec& cw, LAS char* lds, int wid, int lane) {
    const unsigned par = call & 1;
    const rsrc_t rctl = mk(myws, CTL_BYTES), rout = mk(p.out, IN_BYTES), rs = mk(myws + OFF_PKO + par * PKO_PAR, PKO_PAR);
    for (unsigned it = 0;; ++it) {
        LAS unsigned* bx = (LAS unsigned*)(lds + LDS_RING) + (it & 1) * 64;
        if (wid == 0) {
            const unsigned g = rfl(__builtin_amdgcn_raw_ptr_buffer_atomic_add_i32(1, rctl, lane == 0 ? OFF_TKT + 11 * LSTR : OOB, 0, 0)) - (call - 1) * (128 + NGEMM - CPY);
            const unsigned s = lane >> 3, n = lane < 40 ? (s ? 7 + s : g >= GD) * 1024 + g * 8 + (lane & 7) : g >= GD ? GD * 8 - GATE : g * 8;
            const unsigned ro = lane <= 40 ? OFF_RECO + (par * 12 * 1024 + n) * 8 : lane == 41 ? OFF_TMO - 4 : OOB;
            u32x2 rec = {0, 0};
            const unsigned long long t0 = __builtin_amdgcn_s_memrealtime();
            bool dead = false;
            for (unsigned sp = 0; g < 128 && !dead;) {
                rec = __builtin_amdgcn_raw_buffer_load_b64(rctl, ro, 0, AUX_SYS);
                const unsigned long long m = __ballot(rec[1] == call);
                dead = (m >> 41) & 1;
                if ((m & 0x1FFFFFFFFFFull) == 0x1FFFFFFFFFFull) break;
                if (!dead) dead = spin_fail<32>(myws, call, t0, sp);
            }
            bx[lane] = lane < 40 ? rec[0] : lane == 40 ? g : dead;
            WAIT_L(0);
        }
        BAR;
        FENCE;
        const unsigned bv = bx[lane], g = rdl(bv, 40), r0 = rdl(bv, wid), n = g * 8 + wid;
        const bool dead = rdl(bv, 41);
        if (g >= 128) break;
        const unsigned c = g >= GD ? 1024 + n : n;
        load_chunk(mk((g >= GD ? p.ws[(p.rank ^ 1) & 7] : myws) + OFF_PKO + par * PKO_PAR, PKO_PAR), dead ? OOB : c * 3072,
                   is_raw(r0) && !dead ? PKO_MAIN + c * 1024 : OOB, r0, lds + wid * 4096, lane);
        u32x4 Q[4][4];
#pragma unroll
        for (int s = 0; s < 4; ++s)
#pragma unroll
            for (int i = 0; i < 4; ++i)
                Q[s][i] = __builtin_amdgcn_raw_buffer_load_b128(rs, lane * 16, i < 3 ? ((8 + s) * 1024 + n) * 3072 + i * 1024 : PKO_MAIN + ((8 + s) * 1024 + n) * 1024, AUX_SYS);
        WAIT_V(0);
        unsigned D[16];
        float v[32];
        decode(lds + wid * 4096, r0, cw, lane, D, []() {});
        add_chunk(v, D, true);
#pragma unroll
        for (int s = 0; s < 4; ++s) {
#pragma unroll
            for (int i = 0; i < 16; ++i) D[i] = Q[s][i >> 2][i & 3];
            add_chunk(v, D, false);
        }
        store_out(rout, (unsigned)p.rank >> 1, n, v, lane, dead);
    }
}

__global__ __launch_bounds__(512, 2) void gemm_rs_kernel(const Params p) {
    extern __shared__ __attribute__((aligned(16))) char shm[];
    LAS char* lds = (LAS char*)shm;
    const int tid = threadIdx.x, wid = rfl(tid >> 6), lane = tid & 63;
    const unsigned b = blockIdx.x, rank = p.rank & 7;
    char* const myws = p.ws[rank];
    const unsigned z = b & 7, slot = b >> 3, s2 = slot * 2 + (z & 1);
    const unsigned g0 = b >= NGEMM ? 1024 : z >= 6 ? (slot < 12 ? s2 : 84 + s2) : z >= 4 ? (slot < 14 ? 52 + s2 : s2 - 4) : z >= 2 ? (slot < 14 ? 140 : 52) + s2 : slot < 14 ? 1024 : 140 + s2;
    if (g0 - URGENT < 1024 - URGENT) __builtin_amdgcn_s_sleep(50);
    u32x4 R[4];
    pack_load(p, g0, wid, lane, R);
    if (tid < 64) *(LAS unsigned*)(lds + LDS_STAGE + 256 + tid * 4) = 0;
    if (tid < 64) *(LAS unsigned*)(lds + LDS_RING + tid * 4) = 0;
    *(LAS unsigned*)(lds + LDS_LUT + tid * 4) = lut_entry(tid);
    unsigned* callp = (unsigned*)(myws + OFF_CALL + b * 128);
    const unsigned call = rfl(__hip_atomic_load(callp, __ATOMIC_RELAXED, __HIP_MEMORY_SCOPE_AGENT)) + 1;
    const unsigned par = call & 1;
    const Codec cw = {lds + LDS_STAGE + 1024 + wid * CW_BYTES, lds + LDS_LUT};
    const unsigned en0 = p.order.e[lane], en1 = p.order.e[64 + lane];
    {
        unsigned G = g0, it = 0;
        unsigned long long jobs = 0x1834508CA8A8C4C4ull;
        WAIT_L(0); BAR;
        if (tid == 128) __hip_atomic_store(callp, call, __ATOMIC_RELAXED, __HIP_MEMORY_SCOPE_AGENT);
        while (G < 1024) {
            const unsigned Gn = g0 < ((jobs >> 8) & 255) ? G + ((unsigned)jobs & 255) : 1024;
            jobs >>= 8;
            unsigned D[16];
#pragma unroll
            for (int i = 0; i < 16; ++i) D[i] = R[i >> 2][i & 3];
            u32x4 P[4];
            const unsigned r0 = encode(D, pick_top(D), cw, lane, P, []() {});
            const PackJob j = pack_job(G, wid);
            if (G < 8 * NPUSH) {
#pragma unroll
                for (int o = 0; o < 2; ++o) {
                    const unsigned h = j.hi * 4 + o * 2 + j.kk, c = 8192 + rank * 128 + (j.x ^ o) * 16 + (G & 1) * 8 + wid;
                    store_chunk(mk(p.ws[h & 7] + OFF_PKI + par * PKI_PAR, (h >> 1) != (rank >> 1) ? PKI_PAR : 0), c * 3072, PKI_MAIN + c * 1024, lane, P, r0);
                }
            } else store_chunk(mk(myws + OFF_PKI + par * PKI_PAR, PKI_PAR), j.gc * 3072, PKI_MAIN + j.gc * 1024, lane, P, r0);
            SCHED;
            pack_load(p, Gn, wid, lane, R);
            SCHED;
            WAIT_V(4);
            LAS u32x2* lrec = (LAS u32x2*)(lds + LDS_STAGE + (it & 1) * 64);
            if (lane == 0) lrec[wid] = (u32x2){r0, call};
            WAIT_L(0); BAR;
            if (wid < 2) {
                const unsigned h = j.mtx ? j.hi * 4 + wid * 2 + j.kk : wid * 4 + j.hi * 2 + j.kk;
                if ((h >> 1) != (rank >> 1) && lane < 8) {
                    const u32x2 r = lrec[lane];
                    __builtin_amdgcn_raw_buffer_store_b64(r, mk(p.ws[h & 7], CTL_BYTES), lane * 8, OFF_RECI + ((par * 8 + rank) * 2048 + j.rec_idx) * 8, AUX_SYS);
                }
            }
            if (it == 1 && g0 < 12 * NPUSH) {
                const unsigned id = g0 * 8 + wid, si = id / (16 * NPUSH), jj = id % (16 * NPUSH), ch = 8192 + ((rank ^ (si + 2)) & 7) * 128 + jj;
                const Piece pc = describe(64 | jj >> 4, si, rank, par);
                bool dead = false;
                const unsigned q0 = rfl(records(myws, call, pc.f[0] + (jj & 15) * 8, 1, dead));
                load_chunk(mk(myws + OFF_PKI + par * PKI_PAR, PKI_PAR), ch * 3072, is_raw(q0) ? PKI_MAIN + ch * 1024 : OOB, q0, lds + wid * 4096, lane);
                __builtin_amdgcn_s_waitcnt(0x0F70);
                decode(lds + wid * 4096, q0, cw, lane, D, []() {});
                store_in(myws, D, in_word(pc.f[2], jj & 15), lane);
                WAIT_V(0);
                __builtin_amdgcn_raw_ptr_buffer_atomic_add_i32(1, mk(myws, CTL_BYTES), lane == 0 && !dead ? rdy((pc.f[2] >> 3) & 1023) : OOB, 0, 0);
            }
            G = Gn; ++it;
        }
    }
    const unsigned gi = rank >> 2, gj = (rank >> 1) & 1;
    unsigned role = b - NGEMM;
    unsigned tile = g0 - 108, kq = 0;
    const int wr = wid >> 2, wc = wid & 3, fr = lane & 15, fq = lane >> 4;
    const unsigned voffA = (wid * 8 + (lane >> 3)) * 2048 + (((lane & 7) ^ ((wid * 4 + (lane >> 4)) & 7)) * 16);
    const unsigned voffB = (wid * 4 + (lane >> 4)) * 16384 + (((lane & 15) ^ (((lane >> 4) << 2) | (wid & 2))) * 16);
    const unsigned ldsw = wid * 1024;
    int aoff[2], boff[2];
#pragma unroll
    for (int k = 0; k < 2; ++k) aoff[k] = (wr * 64 + fr) * 128 + (((4 * k + fq) ^ ((fr >> 1) & 7)) * 16);
#pragma unroll
    for (int n = 0; n < 2; ++n) { const int q = (lane & 15) >> 2, pp = lane & 3;
        boff[n] = (int)(unsigned)(size_t)lds + 65536 + (8 * fq + q) * 256 + (((4 * wc + 2 * n + (pp >> 1)) ^ ((q << 2) | ((fq & 1) << 1))) * 16) + 8 * (pp & 1); }
    const rsrc_t rctl = mk(myws, CTL_BYTES);
#define HT 16384
#define SA(bb, h) (((bb) * 2 + (h)) * HT)
#define SB(bb, h) ((4 + (bb) * 2 + (h)) * HT)
#define STAGE_A(bb, h, rs, so) do { _Pragma("unroll") for (int _i = 0; _i < 2; ++_i) \
        __builtin_amdgcn_raw_ptr_buffer_load_lds(rs, (LAS void*)(lds + SA(bb, h) + ldsw + _i * 8192), 16, voffA, (so) + (h) * 262144 + _i * 131072, 0, AUX_AGENT); } while (0)
#define STAGE_B(bb, h, rs, so) do { _Pragma("unroll") for (int _i = 0; _i < 2; ++_i) \
        __builtin_amdgcn_raw_ptr_buffer_load_lds(rs, (LAS void*)(lds + SB(bb, h) + ldsw + _i * 8192), 16, voffB, (so) + (h) * 256 + _i * 524288, 0, AUX_AGENT); } while (0)
#define LDA(dst, bb, h) do { _Pragma("unroll") for (int m = 0; m < 4; ++m) _Pragma("unroll") for (int k = 0; k < 2; ++k) \
        dst[m][k] = *(const LAS bf16x8*)(lds + SA(bb, h) + aoff[k] + m * 2048); } while (0)
#define LDB(T, bb, h) asm volatile( \
        "ds_read_b64_tr_b16 %0, %8 offset:%10\n\tds_read_b64_tr_b16 %1, %8 offset:%11\n\tds_read_b64_tr_b16 %2, %8 offset:%12\n\tds_read_b64_tr_b16 %3, %8 offset:%13\n\t" \
        "ds_read_b64_tr_b16 %4, %9 offset:%10\n\tds_read_b64_tr_b16 %5, %9 offset:%11\n\tds_read_b64_tr_b16 %6, %9 offset:%12\n\tds_read_b64_tr_b16 %7, %9 offset:%13" \
        : "=&v"(T[0]), "=&v"(T[1]), "=&v"(T[2]), "=&v"(T[3]), "=&v"(T[4]), "=&v"(T[5]), "=&v"(T[6]), "=&v"(T[7]) \
        : "v"(boff[0]), "v"(boff[1]), "i"(SB(bb, h) - 65536), "i"(SB(bb, h) - 65536 + 1024), "i"(SB(bb, h) - 65536 + 8192), "i"(SB(bb, h) - 65536 + 9216) : "memory")
#define LDB_USE(dst, T) do { asm volatile("s_waitcnt lgkmcnt(0)" : "+v"(T[0]), "+v"(T[1]), "+v"(T[2]), "+v"(T[3]), "+v"(T[4]), "+v"(T[5]), "+v"(T[6]), "+v"(T[7]) :: "memory"); \
        _Pragma("unroll") for (int _j = 0; _j < 4; ++_j) dst[_j >> 1][_j & 1] = __builtin_shufflevector(T[2 * _j], T[2 * _j + 1], 0, 1, 2, 3, 4, 5, 6, 7); } while (0)
#define MMA(ai, bj, At, Bt) do { __builtin_amdgcn_s_setprio(1); _Pragma("unroll") for (int m = 0; m < 4; ++m) _Pragma("unroll") for (int n = 0; n < 2; ++n) _Pragma("unroll") for (int k = 0; k < 2; ++k) \
        acc[ai][bj][m][n] = __builtin_amdgcn_mfma_f32_16x16x32_bf16(Bt[n][k], At[m][k], acc[ai][bj][m][n], 0, 0, 0); __builtin_amdgcn_s_setprio(0); } while (0)
    constexpr unsigned AK = 128, BK = 1048576;
    for (int un = tile < 32 ? -1 : 0; un < 2 && b < NGEMM; ++un) {
        const unsigned d = un ? 0 : 1 + slot % 7, s0 = un ? kq * 8 : 0, nst = un ? 8 : 32;
        const unsigned ut = un ? tile & 1 : z & 1, uct = un ? (tile >> 1) & 15 : (z >> 1) * 4 + slot / 7;
        const unsigned rt = ut * 8 + d;
        const unsigned aown = ((rank ^ d) * 1024 + gi * 512 + ut * 256) * 2048, bown = (gj * 4096 + uct * 256) * 2;
        const unsigned arem = rt * 256 * 2048, brem = uct * 512;
        const unsigned rdy_off = rdy((lane < 32 ? rt : 16 + uct) * 32 + (lane & 31));
        auto step_a = [&](unsigned s) -> unsigned { const unsigned r = s - 8, q = (r * 43) >> 8, si = r - q * 6;
            return s < 8 ? aown + s * 256 : arem + (si >> 1) * 8 * MiB + (si & 1) * 1024 + q * 256; };
        auto step_b = [&](unsigned s) -> unsigned { const unsigned r = s - 8, q = (r * 43) >> 8, si = r - q * 6;
            return s < 8 ? bown + s * 128 * 16384 : brem + ((si >> 1) * 512 + q * 128) * 16384 + (si & 1) * 8192; };
        LAS unsigned* lseq = (LAS unsigned*)(lds + LDS_STAGE + 256 + (un & 1) * 128);
        const unsigned want = un ? 0xFFu << s0 : 0xFFFFFFFFu;
        unsigned seen = want & 0xFFu, seqv = (lane < 8 && s0 == 0) ? lane + 1 : 0;
        auto get_step = [&](unsigned k) -> unsigned {
            unsigned e = rdl(seqv, k);
            if (e == 0) {
                if (wid == 0 && (unsigned)__builtin_popcount(seen) <= k) {
                    const unsigned long long t0 = __builtin_amdgcn_s_memrealtime();
                    unsigned fresh;
                    for (unsigned sp = 0;;) {
                        const unsigned v = __builtin_amdgcn_raw_buffer_load_b32(rctl, rdy_off, 0, AUX_AGENT);
                        const unsigned long long m = __ballot((int)(v - call * 16u) >= 0);
                        fresh = (((unsigned)m & (unsigned)(m >> 32)) << 8) & want & ~seen;
                        if (fresh) break;
                        if (spin_fail<48>(myws, call, t0, sp)) { fresh = want & ~seen; break; }
                        if (z >= 2 || un) __builtin_amdgcn_s_sleep(60);
                    }
                    if (lane < 32 && ((fresh >> lane) & 1))
                        __hip_atomic_store(lseq + __builtin_popcount(seen) + __builtin_popcount(fresh & ((1u << lane) - 1)), (unsigned)lane + 1, __ATOMIC_RELEASE, __HIP_MEMORY_SCOPE_WORKGROUP);
                    seen |= fresh;
                }
                for (Patience pt;;) {
                    seqv = __hip_atomic_load(lseq + (lane & 31), __ATOMIC_ACQUIRE, __HIP_MEMORY_SCOPE_WORKGROUP);
                    e = rdl(seqv, k);
                    if (e || pt.over(myws, call)) break;
                }
                asm volatile("" ::: "memory");
            }
            return (e - 1) & 31;
        };
        f32x4 acc[2][2][4][2];
#pragma unroll
        for (int a = 0; a < 2; ++a)
#pragma unroll
            for (int bb = 0; bb < 2; ++bb)
#pragma unroll
                for (int m = 0; m < 4; ++m)
#pragma unroll
                    for (int n = 0; n < 2; ++n) acc[a][bb][m][n] = (f32x4){0.f, 0.f, 0.f, 0.f};
        bf16x8 At[4][2], B0[2][2], B1[2][2]; s16x4 TB[8];
        unsigned sn = get_step(0);
        rsrc_t ra = mk(sn < 8 ? p.x : myws + OFF_XS, sn < 8 ? IN_BYTES : XS_BYTES);
        unsigned sa = step_a(sn);
        {
            const rsrc_t rb = mk(sn < 8 ? p.w : myws + OFF_WS, sn < 8 ? IN_BYTES : WS_BYTES);
            const unsigned sb = step_b(sn);
            STAGE_B(0, 0, rb, sb); STAGE_A(0, 0, ra, sa); STAGE_B(0, 1, rb, sb); STAGE_A(0, 1, ra, sa);
            if (wr == 1) BAR;
            WAIT_V(4); BAR;
            STAGE_B(1, 0, rb, sb + BK); STAGE_A(1, 0, ra, sa + AK); STAGE_B(1, 1, rb, sb + BK);
            WAIT_V(6); BAR;
            LDB(TB, 0, 0); LDB_USE(B0, TB);
        }
        for (unsigned s = 0; s < nst; ++s) {
            if (s + 1 < nst) sn = get_step(s + 1);
            const rsrc_t rna = mk(sn < 8 ? p.x : myws + OFF_XS, sn < 8 ? IN_BYTES : XS_BYTES);
            const rsrc_t rnb = mk(sn < 8 ? p.w : myws + OFF_WS, sn < 8 ? IN_BYTES : WS_BYTES);
            const unsigned na = step_a(sn), nb = step_b(sn);
            LDA(At, 0, 0); STAGE_A(1, 1, ra, sa + AK);
            WAIT_V(10); BAR; WAIT_L(0); MMA(0, 0, At, B0); BAR; SCHED;
            LDB(TB, 0, 1); STAGE_B(0, 0, rnb, nb);
            WAIT_V(10); BAR; LDB_USE(B1, TB); MMA(0, 1, At, B1); BAR;
            LDA(At, 0, 1); STAGE_A(0, 0, rna, na);
            WAIT_V(10); BAR; WAIT_L(0); MMA(1, 0, At, B0); BAR; SCHED;
            LDB(TB, 1, 0); STAGE_B(0, 1, rnb, nb);
            WAIT_V(10); BAR; SCHED; MMA(1, 1, At, B1); SCHED; LDB_USE(B0, TB); BAR;
            LDA(At, 1, 0); STAGE_A(0, 1, rna, na);
            WAIT_V(10); BAR; WAIT_L(0); MMA(0, 0, At, B0); BAR; SCHED;
            LDB(TB, 1, 1); STAGE_B(1, 0, rnb, nb + BK);
            WAIT_V(10); BAR; LDB_USE(B1, TB); MMA(0, 1, At, B1); BAR;
            LDA(At, 1, 1); STAGE_A(1, 0, rna, na + AK);
            WAIT_V(10); BAR; WAIT_L(0); MMA(1, 0, At, B0); BAR; SCHED;
            LDB(TB, 0, 0); STAGE_B(1, 1, rnb, nb + BK);
            WAIT_V(10); BAR; SCHED; MMA(1, 1, At, B1); SCHED; LDB_USE(B0, TB); BAR;
            ra = rna; sa = na;
        }
        if (wr == 0) BAR;
        {
            const unsigned dst = (rank ^ d) & 7;
            const unsigned c0 = (un ? 8 + kq : d) * 1024 + (uct >> 1) * 128 + (uct & 1) * 16 + ut * 8 + wid;
            unsigned tk = 0;
            if (un == 0 && wid == 0) tk = __builtin_amdgcn_raw_ptr_buffer_atomic_add_i32(1, rctl, lane == 0 ? OFF_TKT + 10 * LSTR : OOB, 0, 0);
            const rsrc_t rpk = mk(myws + OFF_PKO + par * PKO_PAR, PKO_PAR);
            unsigned top = 1, rp = 0;
            const unsigned tmo = __hip_atomic_load((unsigned*)(myws + OFF_TMO), __ATOMIC_RELAXED, __HIP_MEMORY_SCOPE_AGENT);
            auto publish = [&](unsigned c, unsigned r0, int n) {
                __builtin_amdgcn_raw_buffer_store_b64((u32x2){r0, call}, mk(p.ws[dst], CTL_BYTES), tmo != call && lane < n ? OFF_RECO + (par * 12 * 1024 + c - lane * 32) * 8 : OOB, 0, AUX_SYS);
            };
#pragma unroll
            for (int u = 0; u < 4; ++u) {
                unsigned D[16];
#pragma unroll
                for (int m = 0; m < 4; ++m)
#pragma unroll
                    for (int n = 0; n < 2; ++n) {
                        const f32x4 a = acc[u >> 1][u & 1][m][n];
                        D[m * 4 + n * 2] = cvt_pk_bf16(a[0], a[1]); D[m * 4 + n * 2 + 1] = cvt_pk_bf16(a[2], a[3]);
                    }
                u32x4 P[4];
                unsigned r0 = RAW << 16;
                const unsigned c = c0 + u * 32;
                if (un) {
#pragma unroll
                    for (int i = 0; i < 16; ++i) P[i >> 2][i & 3] = D[i];
                } else {
                    if (u == 0) top = pick_top(D);
                    r0 = encode(D, top, cw, lane, P, [&]() { if (u) { WAIT_V(0); publish(c - 32, rp, 1); } });
                }
                store_chunk(rpk, c * 3072, PKO_MAIN + c * 1024, lane, P, r0);
                rp = r0;
            }
            WAIT_V(0);
            publish(c0 + 96, rp, un ? 4 : 1);
            if (un == 0 && tid == 0) *(LAS unsigned*)(lds + LDS_STAGE + 128) = tk - (call - 1) * NGEMM + 32;
            WAIT_L(0);
        }
        BAR;
        if (un) continue;
        role = rfl(*(volatile LAS unsigned*)(lds + LDS_STAGE + 128));
        if (role < 32 + CPY) break;
        role += 256 - (32 + CPY);
        if (role >= 256 + 96) break;
        tile = (role - 256) / 3; kq = 1 + (role - 256) % 3;
    }
    if (role < 32 + CPY) ring(p, myws, call, lds, cw, role, en0, en1, wid, lane);
    else reduce_own(p, myws, call, cw, lds, wid, lane);
}

}

extern "C" void kernel_launch(int rank, int nranks, ncclComm_t comm, hipStream_t stream,
                              void* const* d_in, const int* in_sizes, int n_in,
                              void* d_out, int out_size,
                              const int* in_dtypes, int out_dtype,
                              void* const* d_out_all,
                              void* const* d_ws_all, size_t ws_bytes) {
    (void)comm; (void)in_dtypes; (void)out_dtype; (void)d_out_all;
    if (nranks != NR || (unsigned)rank >= (unsigned)NR || n_in < 2 || ws_bytes < WS_NEED || out_size != 1024 * 8192 ||
        in_sizes[0] != 8192 * 1024 || in_sizes[1] != 1024 * 8192) {
        fprintf(stderr, "kernel_launch: unexpected problem geometry\n");
        return;
    }
    HIP_CHECK(hipFuncSetAttribute((const void*)gemm_rs_kernel, hipFuncAttributeMaxDynamicSharedMemorySize, LDS_BYTES));
    Params p;
    p.x = (const char*)d_in[0]; p.w = (const char*)d_in[1]; p.out = (char*)d_out; p.rank = rank;
    constexpr Order order = make_order();
    p.order = order;
    for (int r = 0; r < NR; ++r) p.ws[r] = (char*)d_ws_all[r];
    hipLaunchKernelGGL(gemm_rs_kernel, dim3(256), dim3(512), LDS_BYTES, stream, p);
    HIP_CHECK(hipGetLastError());
}
```

```cpp
#include <hip/hip_runtime.h>
#include <rccl/rccl.h>
#include <cstdio>

#define HIP_CHECK(expr) do { hipError_t _e = (expr); if (_e != hipSuccess) { \
    fprintf(stderr, "HIP error %s at %s:%d\n", hipGetErrorString(_e), __FILE__, __LINE__); \
    return; } } while (0)

namespace {

#define LAS __attribute__((address_space(3)))
typedef short bf16x8 __attribute__((ext_vector_type(8)));
typedef short s16x4 __attribute__((ext_vector_type(4)));
typedef float f32x4 __attribute__((ext_vector_type(4)));
typedef unsigned u32x4 __attribute__((ext_vector_type(4)));
typedef unsigned u32x2 __attribute__((ext_vector_type(2)));
typedef unsigned short u16x2 __attribute__((ext_vector_type(2)));
typedef __amdgpu_buffer_rsrc_t rsrc_t;

constexpr int NR = 8;
constexpr unsigned MiB = 1u << 20;
constexpr unsigned OFF_CALL = 0;
constexpr unsigned OFF_TMO = 32768;
constexpr unsigned LSTR = 4352;
constexpr unsigned OFF_RDY = 786432;
constexpr unsigned OFF_TKT = OFF_RDY + 32 * LSTR;
static_assert(OFF_TKT + 12 * LSTR <= 1u << 20, "");
__device__ __forceinline__ unsigned rdy(unsigned n) { return OFF_RDY + (n >> 5) * LSTR + (n & 31) * 4; }
constexpr unsigned OFF_RECI = 65536;
constexpr unsigned OFF_RECO = 327680;
constexpr unsigned CTL_BYTES = MiB;
constexpr unsigned OFF_PKI = 1 * MiB, PKI_MAIN = 27 * MiB, PKI_PAR = 36 * MiB;
constexpr unsigned OFF_PKO = 73 * MiB, PKO_MAIN = 36 * MiB, PKO_PAR = 48 * MiB;
constexpr unsigned OFF_XS = 169 * MiB, XS_BYTES = 24 * MiB;
constexpr unsigned OFF_WS = 193 * MiB, WS_BYTES = 24 * MiB;
constexpr unsigned WS_NEED = 217 * MiB;
constexpr unsigned IN_BYTES = 16 * MiB;
constexpr unsigned OOB = 0x80000000u;
constexpr unsigned long long DEADLINE = 30000000ull;

constexpr int LDS_STAGE = 131072, LDS_LUT = LDS_STAGE + 1024 + 8 * 2128, LDS_RING = LDS_LUT + 2048, LDS_BYTES = LDS_RING + 1024;
constexpr int NGEMM = 224;
constexpr int NPW = 5;
constexpr int DEPTH = 4;
constexpr unsigned AHEAD = 4;
constexpr unsigned TAILG = 40;
constexpr unsigned CPY = 3;
constexpr unsigned GD = 120, GATE = 48;
constexpr int NPUSH = 4;
#define OLDMAP 0
#define INBOX_IT 3
constexpr unsigned URGENT = 72;
constexpr int AUX_SYS = 17, AUX_AGENT = 16;
constexpr int SLOPE = 28, PAIR = 24, XSTEP = 22, GROUPT = 156;

__device__ __forceinline__ unsigned xpiece(unsigned x) { return (unsigned)(0x80A2FD75EC64B931ull >> (4 * x)) & 15; }
struct Order { unsigned char e[128]; };
constexpr Order make_order() {
    Order o{}; int key[128] = {}; int n = 0;
    for (int q = 0; q < 4; ++q) {
        for (int x = 0; x < 16; ++x) {
            o.e[n] = (unsigned char)(64 | q << 4 | x);
            key[n++] = (q == 0 && x < NPUSH) ? 200000 : x >= 14 ? 100000 + q : !q ? -1 : x < 2 ? q * GROUPT - 1 : q * GROUPT + 1 + (x - 2) / 2 * XSTEP; }
        for (int c = 0; c < 16; ++c) { o.e[n] = (unsigned char)(q << 4 | c); key[n++] = q * GROUPT + c * SLOPE - (c & 1) * PAIR; }
    }
    for (int i = 1; i < 128; ++i)
        for (int j = i; j > 0 && key[j - 1] > key[j]; --j) {
            const int k = key[j]; key[j] = key[j - 1]; key[j - 1] = k; const unsigned char e = o.e[j]; o.e[j] = o.e[j - 1]; o.e[j - 1] = e; }
    return o;
}

struct Params {
    const char* x; const char* w; char* out; char* ws[NR]; int rank; Order order;
};

__device__ __forceinline__ rsrc_t mk(const void* p, unsigned bytes) { return __builtin_amdgcn_make_buffer_rsrc((void*)p, 0, bytes, 0x00020000); }
__device__ __forceinline__ unsigned rfl(unsigned v) { return __builtin_amdgcn_readfirstlane(v); }
__device__ __forceinline__ unsigned rdl(unsigned v, unsigned l) { return __builtin_amdgcn_readlane(v, l); }
#define WAIT_V(n) asm volatile("s_waitcnt vmcnt(" #n ")" ::: "memory")
#define WAIT_L(n) asm volatile("s_waitcnt lgkmcnt(" #n ")" ::: "memory")
#define BAR __builtin_amdgcn_s_barrier()
#define SCHED __builtin_amdgcn_sched_barrier(0)

__device__ __forceinline__ unsigned cvt_pk_bf16(float lo, float hi) {
    unsigned r; asm("v_cvt_pk_bf16_f32 %0, %1, %2" : "=v"(r) : "v"(lo), "v"(hi)); return r;
}
__device__ __forceinline__ unsigned pk_sub(unsigned a, unsigned b) { return __builtin_bit_cast(unsigned, __builtin_bit_cast(u16x2, a) - __builtin_bit_cast(u16x2, b)); }
__device__ __forceinline__ unsigned bfi(unsigned m, unsigned a, unsigned b) { return (a & m) | (b & ~m); }
__device__ __forceinline__ unsigned rotr(unsigned a, unsigned n) { return __builtin_amdgcn_alignbit(a, a, n); }
template <int CTRL> __device__ __forceinline__ float dpp_f(float v) { return __builtin_bit_cast(float, __builtin_amdgcn_update_dpp(0, __builtin_bit_cast(int, v), CTRL, 0xf, 0xf, true)); }
__device__ __forceinline__ float rdl_f(float v, unsigned l) { return __builtin_bit_cast(float, rdl(__builtin_bit_cast(unsigned, v), l)); }

constexpr unsigned RAW = 255, EB = 1040, CW_BYTES = EB + 1024 + 64;
struct Codec { LAS char* cw; LAS const char* lut; };
__device__ __forceinline__ bool is_raw(unsigned r0) { return ((r0 >> 16) & 255) == RAW; }
__device__ __forceinline__ unsigned plane2_bytes(unsigned r0) { return 704 + 4 * ((r0 >> 16) & 255); }
__device__ __forceinline__ unsigned lut_entry(unsigned t) {
    unsigned k = 0, e = t < 256 ? 0 : 0x0C0C0C0Cu;
#pragma unroll
    for (unsigned b = 0; b < 4; ++b) {
        const unsigned c = (t >> (2 * b)) & 3;
        if (t < 256) e |= (c ? 4 + c : k) << (8 * b); else if (!c) e ^= (12 ^ b) << (8 * k);
        k += !c;
    }
    return e;
}
__device__ __forceinline__ unsigned scan_add(unsigned v) {
    int s = v;
    s += __builtin_amdgcn_update_dpp(0, s, 0x111, 0xf, 0xf, false); s += __builtin_amdgcn_update_dpp(0, s, 0x112, 0xf, 0xf, false);
    s += __builtin_amdgcn_update_dpp(0, s, 0x114, 0xf, 0xf, false); s += __builtin_amdgcn_update_dpp(0, s, 0x118, 0xf, 0xf, false);
    s += __builtin_amdgcn_update_dpp(0, s, 0x142, 0xa, 0xf, false); s += __builtin_amdgcn_update_dpp(0, s, 0x143, 0xc, 0xf, false);
    return s;
}
__device__ __forceinline__ unsigned pick_top(const unsigned (&D)[16]) {
    float sm = 0.f;
#pragma unroll
    for (int i = 0; i < 16; i += 4) sm += __builtin_fabsf(__builtin_bit_cast(float, D[i] << 16)) + __builtin_fabsf(__builtin_bit_cast(float, D[i] & 0xFFFF0000u));
    sm += dpp_f<0xB1>(sm); sm += dpp_f<0x4E>(sm); sm += dpp_f<0x141>(sm); sm += dpp_f<0x140>(sm);
    const float top = ((rdl_f(sm, 0) + rdl_f(sm, 16)) + (rdl_f(sm, 32) + rdl_f(sm, 48))) * (20.6f / 512.f);
    return (rfl(__builtin_bit_cast(unsigned, top) >> 15) & 0xFFFFu) | 1u;
}
#define FENCE asm volatile("" ::: "memory")
template <class Fn>
__device__ __forceinline__ unsigned encode(const unsigned (&D)[16], unsigned top, const Codec& k, int lane, u32x4 (&P)[4], Fn mid) {
    LAS char* const cw = k.cw;
    const unsigned top2 = top * 0x10001u;
    unsigned F[8], sel[8], C[2] = {0, 0};
#pragma unroll
    for (int i = 0; i < 8; ++i) {
        const unsigned a = pk_sub(top2, rotr(D[2 * i], 31)), b = pk_sub(top2, rotr(D[2 * i + 1], 31));
        P[i >> 2][i & 3] = __builtin_amdgcn_perm(b, a, 0x06040200u);
        F[i] = __builtin_amdgcn_perm(b, a, 0x07050301u);
        const unsigned c = __builtin_amdgcn_perm(0x00000302u, 0x01000000u, F[i]);
        const unsigned cb = __builtin_amdgcn_udot4((c ^ (c >> 2)) & 0x03030303u, 0x40100401u, 0, false);
        sel[i] = *(LAS const unsigned*)(k.lut + 1024 + cb * 4);
        C[i >> 2] |= cb << (8 * (i & 3));
    }
    const unsigned m[2] = {~(C[0] | C[0] >> 1) & 0x55555555u, ~(C[1] | C[1] >> 1) & 0x55555555u};
    const unsigned k0 = __builtin_popcount(m[0]), kk = k0 + __builtin_popcount(m[1]), incl = scan_add(kk), cnt = rdl(incl, 63);
    P[2] = (u32x4){F[0], F[1], F[2], F[3]}; P[3] = (u32x4){F[4], F[5], F[6], F[7]};
    mid();
    if (cnt > 1024) return top | RAW << 16;
    const unsigned fill = lane * 16u >= cnt ? 0x02020202u : 0;
    *(LAS u32x4*)(cw + EB + lane * 16) = (u32x4){fill, fill, fill, fill};
    FENCE;
    *(LAS unsigned char*)(cw + EB + cnt + lane) = 2;
    *(LAS u32x2*)(cw + lane * 8) = (u32x2){C[0], C[1]};
    FENCE;
    const unsigned e1 = (unsigned)(size_t)cw + EB - 1 + incl;
    unsigned carry = 0;
#pragma unroll
    for (int i = 0; i < 8; ++i) {
        const unsigned p = e1 - kk + (i >= 4 ? k0 : 0) + __builtin_popcount(m[i >> 2] & ((1u << (8 * (i & 3))) - 1)), pk = __builtin_amdgcn_perm(0u, F[i], sel[i]);
        __hip_atomic_fetch_or((LAS unsigned*)(size_t)(p & ~3u), __builtin_amdgcn_alignbyte(pk, 0u, ~p) | carry, __ATOMIC_RELAXED, __HIP_MEMORY_SCOPE_WAVEFRONT);
        carry = __builtin_amdgcn_alignbyte(0u, pk, ~p);
    }
    __hip_atomic_fetch_or((LAS unsigned*)(size_t)(e1 & ~3u), carry, __ATOMIC_RELAXED, __HIP_MEMORY_SCOPE_WAVEFRONT);
    FENCE;
    unsigned M = 0, ovf = 0;
    auto dense = [&](int r) -> unsigned {
        const u32x2 f = *(LAS u32x2*)(cw + EB + r * 512 + lane * 8);
        const unsigned t0 = __builtin_amdgcn_perm(0u, 0x00074040u, f[0]), t1 = __builtin_amdgcn_perm(0u, 0x00074040u, f[1]);
        M |= bfi(0x40404040u, t0, t1 << 1) >> (6 - 2 * r) & 0x03030303u << (2 * r);
        return ((f[0] ^ t0) & 0x07070707u) | ((f[1] ^ t1) & 0x07070707u) << 3;
    };
    const unsigned x = dense(0), y = bfi(0xC0C0C0u, __umul24(x >> 24, 0x41040u), x);
    *(LAS unsigned short*)(cw + 512 + lane * 2) = (unsigned short)y;
    *(LAS unsigned char*)(cw + 640 + lane) = (unsigned char)(y >> 16);
    if (cnt > 512) { *(LAS unsigned*)(cw + 704 + lane * 4) = dense(1); ovf = (cnt - 505) >> 3; }
    FENCE;
    unsigned n = 0;
    for (;;) {
        const unsigned long long act = __ballot(M != 0);
        if (!act) break;
        if (M) {
            const unsigned off = ovf + n + __builtin_amdgcn_mbcnt_hi((unsigned)(act >> 32), __builtin_amdgcn_mbcnt_lo((unsigned)act, 0));
            const unsigned bt = __builtin_ctz(M), pos = (bt & 2) * 256 + lane * 8 + (bt & 1) * 4 + (bt >> 3);
            *(LAS unsigned*)(cw + 704 + (off < 80 ? off : 80) * 4) = pos | (unsigned)*(LAS unsigned char*)(cw + EB + pos) << 16;
            M &= M - 1;
        }
        n += __builtin_popcountll(act);
        if (n > 64 || ovf + n > 80) return top | RAW << 16;
    }
    FENCE;
    P[2] = *(LAS u32x4*)(cw + lane * 16);
    FENCE;
    return top | (ovf + n) << 16 | n << 24;
}
template <class Fn>
__device__ __forceinline__ void decode(LAS const char* in, unsigned r0, const Codec& k, int lane, unsigned (&D)[16], Fn done) {
    LAS char* const cw = k.cw;
    LAS const char* const p2 = in + 2048;
    const unsigned n = r0 >> 24, ovf = ((r0 >> 16) & 255) - n, top2 = (r0 & 0xFFFFu) * 0x10001u;
    unsigned F[8];
    u32x4 L[2];
    FENCE;
    if (is_raw(r0)) {
        const u32x4 f0 = *(LAS const u32x4*)(p2 + lane * 16), f1 = *(LAS const u32x4*)(p2 + 1024 + lane * 16);
        L[0] = *(LAS const u32x4*)(in + lane * 16); L[1] = *(LAS const u32x4*)(in + 1024 + lane * 16);
        FENCE; done(); FENCE;
#pragma unroll
        for (int i = 0; i < 4; ++i) { F[i] = f0[i]; F[i + 4] = f1[i]; }
    } else {
        const u32x2 C = *(LAS const u32x2*)(p2 + lane * 8);
        const unsigned y = *(LAS const unsigned short*)(p2 + 512 + lane * 2) | (unsigned)*(LAS const unsigned char*)(p2 + 640 + lane) << 16;
        const unsigned xo = *(LAS const unsigned*)(p2 + 704 + lane * 4), e = *(LAS const unsigned*)(p2 + 704 + ((ovf + lane) & 127) * 4);
        L[0] = *(LAS const u32x4*)(in + lane * 16); L[1] = *(LAS const u32x4*)(in + 1024 + lane * 16);
        FENCE; done(); FENCE;
        auto spread = [&](int r, unsigned x) {
            *(LAS u32x2*)(cw + EB + r * 512 + lane * 8) = (u32x2){__builtin_amdgcn_perm(0x0706020Cu, 0x0B0A0908u, x & 0x07070707u),
                                                                  __builtin_amdgcn_perm(0x0706020Cu, 0x0B0A0908u, (x >> 3) & 0x07070707u)};
        };
        unsigned sel[8];
#pragma unroll
        for (int i = 0; i < 8; ++i) sel[i] = *(LAS const unsigned*)(k.lut + ((C[i >> 2] >> (8 * (i & 3))) & 255) * 4);
        spread(0, bfi(0x3F000000u, __umul24(y & 0xC0C0C0u, 0x41040u), y));
        if (ovf) spread(1, xo);
        FENCE;
        *(LAS unsigned char*)(cw + ((unsigned)lane < n ? EB + (e & 1023) : 1024)) = (unsigned char)(e >> 16);
        FENCE;
        const unsigned m[2] = {~(C[0] | C[0] >> 1) & 0x55555555u, ~(C[1] | C[1] >> 1) & 0x55555555u};
        const unsigned k0 = __builtin_popcount(m[0]), kk = k0 + __builtin_popcount(m[1]), p0 = (unsigned)(size_t)cw + EB + scan_add(kk) - kk;
#pragma unroll
        for (int i = 0; i < 8; ++i) {
            const unsigned p = p0 + (i >= 4 ? k0 : 0) + __builtin_popcount(m[i >> 2] & ((1u << (8 * (i & 3))) - 1));
            const LAS unsigned* w = (const LAS unsigned*)(size_t)(p & ~3u);
            F[i] = __builtin_amdgcn_perm(0x05040300u, __builtin_amdgcn_alignbyte(w[1], w[0], p), sel[i]);
        }
        FENCE;
    }
#pragma unroll
    for (int i = 0; i < 8; ++i) {
        D[2 * i] = rotr(pk_sub(top2, __builtin_amdgcn_perm(F[i], L[i >> 2][i & 3], 0x05010400u)), 1);
        D[2 * i + 1] = rotr(pk_sub(top2, __builtin_amdgcn_perm(F[i], L[i >> 2][i & 3], 0x07030602u)), 1);
    }
}
__device__ __forceinline__ void store_chunk(rsrc_t rs, unsigned main_off, unsigned esc_off, int lane, const u32x4 (&P)[4], unsigned r0) {
#pragma unroll
    for (int i = 0; i < 2; ++i) __builtin_amdgcn_raw_buffer_store_b128(P[i], rs, main_off + lane * 16 + i * 1024, 0, AUX_SYS);
    __builtin_amdgcn_raw_buffer_store_b128(P[2], rs, lane * 16u < plane2_bytes(r0) ? main_off + lane * 16 + 2048 : OOB, 0, AUX_SYS);
    __builtin_amdgcn_raw_buffer_store_b128(P[3], rs, is_raw(r0) ? esc_off + lane * 16 : OOB, 0, AUX_SYS);
}
__device__ __forceinline__ void load_chunk(rsrc_t rs, unsigned main_off, unsigned esc_off, unsigned r0, LAS char* to, int lane) {
    if (esc_off != OOB) __builtin_amdgcn_raw_ptr_buffer_load_lds(rs, (LAS void*)(to + 3072), 16, lane * 16, esc_off, 0, AUX_SYS);
    __builtin_amdgcn_raw_ptr_buffer_load_lds(rs, (LAS void*)to, 16, lane * 16, main_off, 0, AUX_SYS);
    __builtin_amdgcn_raw_ptr_buffer_load_lds(rs, (LAS void*)(to + 1024), 16, lane * 16 + 1024, main_off, 0, AUX_SYS);
    __builtin_amdgcn_raw_ptr_buffer_load_lds(rs, (LAS void*)(to + 2048), 16, lane * 16u < plane2_bytes(r0) ? lane * 16 + 2048 : OOB, main_off, 0, AUX_SYS);
}

template <int SLEEP>
__device__ __forceinline__ bool spin_fail(char* myws, unsigned call, unsigned long long t0, unsigned& n) {
    unsigned* tmo = (unsigned*)(myws + OFF_TMO);
    if ((++n & 63) == 0 &&
        (rfl(__hip_atomic_load(tmo, __ATOMIC_RELAXED, __HIP_MEMORY_SCOPE_AGENT)) == call || __builtin_amdgcn_s_memrealtime() - t0 > DEADLINE)) {
        __hip_atomic_store(tmo, call, __ATOMIC_RELAXED, __HIP_MEMORY_SCOPE_AGENT);
        return true;
    }
    __builtin_amdgcn_s_sleep(SLEEP);
    return false;
}

struct PackJob { unsigned src_off, stride, gc, rec_idx, x; int mtx, kk, hi; };
__device__ __forceinline__ PackJob pack_job(unsigned G, int wid) {
    PackJob j; const unsigned q = G >> 8, ch = (G & 1) * 8 + wid;
    j.mtx = ((G >> 7) & 1) ^ 1; j.x = (G >> 3) & 15; j.hi = (G >> 2) & 1; j.kk = (G >> 1) & 1; const unsigned ks = j.kk * 4 + q;
    unsigned t16, tile;
    if (j.mtx) { const unsigned e = xpiece(j.x), sh = (j.hi * 4 + j.kk) ^ (e & 7), t = e >> 3;
        tile = sh * 4 + j.hi * 2 + t; t16 = sh * 2 + t; j.src_off = (tile * 256 + ch * 16) * 2048 + ks * 256; j.stride = 2048; }
    else { tile = j.hi * 16 + j.x; t16 = j.x; j.src_off = (ks * 128 + (ch >> 1) * 16) * 16384 + tile * 512 + (ch & 1) * 256; j.stride = 16384; }
    j.gc = (j.mtx * 256 + tile * 8 + ks) * 16 + ch;
    j.rec_idx = (j.mtx * 64 + t16 * 4 + q) * 16 + (G & 1) * 8;
    return j;
}
__device__ __forceinline__ void pack_load(const Params& p, unsigned G, int wid, int lane, u32x4 (&R)[4]) {
    const PackJob j = pack_job(G < 1024 ? G : 0, wid);
    const rsrc_t rs = mk(j.mtx ? p.x : p.w, G < 1024 ? IN_BYTES : 0);
    const unsigned vo = (lane >> 4) * j.stride + (lane & 15) * 16;
#pragma unroll
    for (int i = 0; i < 4; ++i) R[i] = __builtin_amdgcn_raw_buffer_load_b128(rs, vo, j.src_off + i * 4 * j.stride, 0);
}

__device__ __forceinline__ void add_chunk(float (&v)[32], const unsigned (&D)[16], bool first) {
#pragma unroll
    for (int i = 0; i < 16; ++i) {
        const float a = __builtin_bit_cast(float, D[i] << 16), b = __builtin_bit_cast(float, D[i] & 0xFFFF0000u);
        v[2 * i] = first ? a : v[2 * i] + a; v[2 * i + 1] = first ? b : v[2 * i + 1] + b;
    }
}
__device__ __forceinline__ void store_out(rsrc_t rout, unsigned pr, unsigned n, const float (&v)[32], int lane, bool dead) {
    const unsigned t = (n >> 3) & 1, ct = ((n >> 6) & 14) | ((n >> 4) & 1), w = n & 7, u = (n >> 5) & 3;
    const int fr = lane & 15, fq = lane >> 4;
    const unsigned vo = fr * 16384 + ((fq & 1) * 16 + (fq >> 1) * 8) * 2;
    const unsigned so = ((pr >> 1) * 512 + t * 256 + (u >> 1) * 128 + (w >> 2) * 64) * 16384 + ((pr & 1) * 4096 + ct * 256 + (u & 1) * 128 + (w & 3) * 32) * 2;
#pragma unroll
    for (int m = 0; m < 4; ++m) {
        unsigned d[4];
#pragma unroll
        for (int h = 0; h < 4; ++h) {
            const float a = v[8 * m + 2 * h], b = v[8 * m + 2 * h + 1];
            d[h] = cvt_pk_bf16(a < 0.f ? 0.f : a, b < 0.f ? 0.f : b);
        }
        const auto s0 = __builtin_amdgcn_permlane16_swap(d[0], d[2], false, false);
        const auto s1 = __builtin_amdgcn_permlane16_swap(d[1], d[3], false, false);
        u32x4 o = {s0[0], s1[0], s0[1], s1[1]};
        if (dead) o = (u32x4){0x7FC07FC0u, 0x7FC07FC0u, 0x7FC07FC0u, 0x7FC07FC0u};
        __builtin_amdgcn_raw_buffer_store_b128(o, rout, vo + so + m * 16 * 16384, 0, AUX_AGENT);
    }
}

enum { K_UNPACK = 0, K_SUM = 1, K_COPY = 2, K_END = 3 };
enum { R_FULL = 0, R_ISSUED = 4, R_AVAIL = 8, R_NEXT = 12, R_FREE = 128, R_DESC = 256 };
__device__ __forceinline__ unsigned lds_ld2(unsigned a, unsigned b, unsigned& vb) {
    unsigned v; asm volatile("ds_read_b32 %0, %2\n\tds_read_b32 %1, %3\n\ts_waitcnt lgkmcnt(0)" : "=&v"(v), "=&v"(vb) : "v"(a), "v"(b) : "memory"); return v; }
__device__ __forceinline__ void lds_st(unsigned a, unsigned v) { asm volatile("ds_write_b32 %0, %1" :: "v"(a), "v"(v) : "memory"); }
struct Patience {
    unsigned n = 0; unsigned long long t0 = 0;
    __device__ __forceinline__ bool over(char* myws, unsigned call) {
        if ((n++ & 1023) == 0) {
            const unsigned long long t = __builtin_amdgcn_s_memrealtime();
            if (n == 1) t0 = t;
            if (t - t0 > 2 * DEADLINE) { __hip_atomic_store((unsigned*)(myws + OFF_TMO), call, __ATOMIC_RELAXED, __HIP_MEMORY_SCOPE_AGENT); return true; }
        }
        __builtin_amdgcn_s_sleep(1);
        return false;
    }
};
struct Piece { unsigned f[3]; };
__device__ __forceinline__ Piece describe(unsigned en, unsigned si, unsigned rank, unsigned par) {
    const unsigned mtx = (en >> 6) & 1, q = (en >> 4) & 3, ct = en & 15, g = (rank ^ (si + 2)) & 7;
    const unsigned y = (ct + 10 + 2 * si) % 12, x = !q || ct >= 14 ? ct : ct < 2 ? 2 * ct : y ? y + 2 : 1;
    const unsigned e = xpiece(x), t = e >> 3, d = e & 7, sh = rank ^ d, ks = (rank & 1) * 4 + q;
    const unsigned tile32 = mtx ? sh * 4 + (rank >> 2) * 2 + t : (rank & 2) * 8 + ct, t16 = mtx ? sh * 2 + t : ct;
    const unsigned dst = mtx ? (si >> 1) * 8 * MiB + (t * 8 + d) * 256 * 2048 + (si & 1) * 1024 + q * 256
                             : XS_BYTES + ((si >> 1) * 512 + q * 128) * 16384 + (si & 1) * 8192 + ct * 512;
    return {{OFF_RECI + ((par * 8 + g) * 2048 + (mtx * 64 + t16 * 4 + q) * 16) * 8, (mtx * 256 + tile32 * 8 + ks) * 16,
             dst << 5 | ((mtx ? t * 8 + d : 16 + ct) * 32 + q * 6 + si) << 3}};
}
__device__ __forceinline__ unsigned in_word(unsigned w, unsigned ch) { return w + ((w < XS_BYTES << 5 ? ch * 32768 : (ch >> 1) * 262144 + (ch & 1) * 256) << 5); }
__device__ __forceinline__ unsigned records(char* myws, unsigned call, unsigned ro, unsigned need, bool& dead) {
    const unsigned long long t0 = __builtin_amdgcn_s_memrealtime();
    for (unsigned sp = 0; !dead;) {
        const u32x2 rec = __builtin_amdgcn_raw_buffer_load_b64(mk(myws, CTL_BYTES), ro, 0, AUX_SYS);
        if (((unsigned)__ballot(rec[1] == call) & need) == need) return rec[0];
        dead = spin_fail<4>(myws, call, t0, sp);
    }
    return 0;
}
__device__ __forceinline__ void store_in(char* myws, const unsigned (&D)[16], unsigned w, int lane) {
    const unsigned dst = w >> 13 << 8, st = dst < XS_BYTES ? 2048 : 16384, vo = (lane >> 4) * st + (lane & 15) * 16 + dst;
#pragma unroll
    for (int i = 0; i < 4; ++i)
        __builtin_amdgcn_raw_buffer_store_b128((u32x4){D[4 * i], D[4 * i + 1], D[4 * i + 2], D[4 * i + 3]}, mk(myws + OFF_XS, XS_BYTES + WS_BYTES), vo + i * 4 * st, 0, AUX_AGENT);
}
__device__ __forceinline__ void ring(const Params& p, char* myws, unsigned call, LAS char* lds, const Codec& cw, unsigned c, unsigned en0, unsigned en1, int wid, int lane) {
    const unsigned rank = p.rank & 7, par = call & 1;
    const bool cp = c >= 30;
    const unsigned si = c % 6, g = (rank ^ (cp ? 1 : si + 2)) & 7, quad = 1 + (si >> 1), ctl = (unsigned)(size_t)lds + LDS_RING;
    const unsigned pki = OFF_PKI + par * PKI_PAR, pko = OFF_PKO + par * PKO_PAR, reco = OFF_RECO + par * 12 * 1024 * 8;
    const rsrc_t rctl = mk(myws, CTL_BYTES);
    if (wid == 0) {
        __builtin_amdgcn_s_setprio(3);
        unsigned long long src0 = (unsigned long long)p.ws[g], src1 = (unsigned long long)p.ws[g ^ 1];
        asm volatile("" : "+s"(src0), "+s"(src1));
        unsigned av = 0, k0 = 0, d = 0, lim = 0;
        unsigned long long raw = 0;
        int full = 0;
        auto post = [&](unsigned issued) { asm volatile("ds_write_b64 %0, %1" :: "v"(ctl + R_FULL), "v"((u32x2){(unsigned)full, issued}) : "memory"); };
        for (unsigned k = 0;; k += 4) {
            if (av <= k) {
                k0 = k;
                for (Patience pt;;) {
                    const unsigned q = rfl(lds_ld2(ctl + R_AVAIL, ctl + R_DESC + ((k * 16 + lane * 4) & 511), d));
                    av = q < k + 16 ? q : k + 16;
                    if (av > k) break;
                    if (full < (int)k) { WAIT_V(0); full = k; post(k); } else if (pt.over(myws, call)) return;
                }
                lim = plane2_bytes(d); raw = __ballot((lane & 3) == 1 && d != OOB);
            }
            const unsigned l = (k - k0) * 4, w = rdl(d, l + 3);
            if ((w & 3) == K_END) {
                WAIT_V(6); full = max(full, (int)k - 2); post(k);
                __builtin_amdgcn_s_waitcnt(0x0F70);
                full = 0x7FFFFFFF; post(k);
                break;
            }
            const rsrc_t rs[2] = {mk((void*)(w & si & 1 ? src1 : src0), WS_NEED), mk((void*)(w & ~si & 1 ? src1 : src0), WS_NEED)};
            LAS char* const to = lds + (k & 31) * 4096;
            if ((raw >> l) & 0xFFFF)
#pragma unroll
                for (int i = 0; i < 4; ++i) __builtin_amdgcn_raw_ptr_buffer_load_lds(rs[i & 1], (LAS void*)(to + i * 4096 + 3072), 16, lane * 16, rdl(d, l + 4 * i + 1), 0, AUX_SYS);
#pragma unroll
            for (int i = 0; i < 4; ++i) {
                const unsigned o0 = rdl(d, l + 4 * i), v2 = lane * 16u < rdl(lim, l + 4 * i + 2) ? lane * 16 : OOB;
                SCHED;
                asm volatile("s_waitcnt vmcnt(%0)" :: "i"(3 * (DEPTH - 1)) : "memory");
                __builtin_amdgcn_raw_ptr_buffer_load_lds(rs[i & 1], (LAS void*)(to + i * 4096), 16, lane * 16, o0, 0, AUX_SYS);
                __builtin_amdgcn_raw_ptr_buffer_load_lds(rs[i & 1], (LAS void*)(to + i * 4096), 16, lane * 16, o0, 1024, AUX_SYS);
                __builtin_amdgcn_raw_ptr_buffer_load_lds(rs[i & 1], (LAS void*)(to + i * 4096), 16, v2, o0, 2048, AUX_SYS);
                SCHED;
                full = max(full, (int)k + i + 1 - DEPTH);
                post(k + i + 1);
            }
        }
    }
    if (wid == 1) {
        const Piece tb[2] = {describe(en0, si, rank, par), describe(en1, si, rank, par)};
        auto piece = [&](int f, unsigned pc) -> unsigned { const unsigned a = rdl(tb[0].f[f], pc & 63), b = rdl(tb[1].f[f], pc & 63); return pc & 64 ? b : a; };
        constexpr unsigned NG_IN = (128 - NPUSH) * 2;
        const unsigned big = (cp ? GD : 256) - TAILG / 2, ng_out = big + TAILG, l = lane & 7;
        unsigned kind = cp ? K_COPY : K_UNPACK, gr = cp ? c - 30 : c / 6, tk = 0, ro = OOB, ch = 0, w = 0;
        u32x2 rec = {0, 0};
        bool dead = false;
        auto ask = [&]() {
            if (kind == K_UNPACK && gr >= NG_IN) { kind = K_SUM; gr = (c / 6) * 2 + (si & 1); }
            if (kind != K_UNPACK && gr >= ng_out) { kind = K_END; return; }
            if (kind == K_UNPACK) { const unsigned h = (gr & 1) * 8 + l; ro = piece(0, gr >> 1) + h * 8; ch = piece(1, gr >> 1) + h; w = in_word(piece(2, gr >> 1), h); }
            else {
                const unsigned sl = (gr < big ? gr * 8 : big * 4 + gr * 4) + l, n = (cp ? sl : sl >> 1) & 1023;
                ch = (cp ? 1 : 2 * quad + (l & 1)) * 1024 + n; ro = reco + ch * 8; w = kind | n << 3;
            }
            rec = __builtin_amdgcn_raw_buffer_load_b64(rctl, lane < 8 ? ro : lane == 8 ? OFF_TMO - 4 : OOB, 0, AUX_SYS);
            tk = __builtin_amdgcn_raw_ptr_buffer_atomic_add_i32(1, rctl, lane == 0 ? OFF_TKT + (kind == K_UNPACK ? si : cp ? 6 : 6 + quad) * LSTR : OOB, 0, 0);
        };
        ask();
        for (unsigned k = 0;;) {
            const bool in = kind == K_UNPACK, end = kind == K_END;
            const unsigned cnt = in || gr < big ? 8 : 4, need = (1u << cnt) - 1;
            u32x4 d = {OOB, OOB, rec[0], K_END};
            if (!end) {
                const unsigned m = (unsigned)__ballot(rec[1] == call);
                dead |= (m >> 8) & 1;
                if ((m & need) != need) d[2] = records(myws, call, lane < 8 ? ro : OOB, need, dead);
                d[3] = dead ? w | 4 : w;
                if (!dead) { d[0] = (in ? pki : pko) + ch * 3072; if (is_raw(d[2])) d[1] = (in ? pki + PKI_MAIN : pko + PKO_MAIN) + ch * 1024; }
                gr = rfl(tk) - (call - 1) * (in ? NG_IN : ng_out) + (in ? NPW : cp ? 2 + CPY : 2 * NPW);
                ask();
            }
            for (Patience pt;;) {
                unsigned fr;
                const unsigned s = k + l;
                if (rfl(lds_ld2(ctl + R_ISSUED, ctl + R_FREE + (s & 31) * 4, fr)) + AHEAD >= k && !__ballot(fr * 32 + 32 <= s)) break;
                if (pt.over(myws, call)) return;
            }
            if (lane < 8) *(LAS u32x4*)(lds + LDS_RING + R_DESC + ((k + l) & 31) * 16) = d;
            k += cnt;
            lds_st(ctl + R_AVAIL, k);
            if (end) break;
        }
    }
    const rsrc_t rd = mk(myws, WS_NEED), rout = mk(p.out, IN_BYTES);
    unsigned owe = ~0u, owe_r0 = 0;
    auto announce = [&]() {
        if (owe != ~0u) {
            WAIT_V(0);
            const unsigned n = (owe >> 3) & 1023;
            if (cp) __builtin_amdgcn_raw_buffer_store_b64((u32x2){owe_r0, call}, rctl, lane == 0 ? reco + n * 8 : OOB, 0, AUX_SYS);
            else __builtin_amdgcn_raw_ptr_buffer_atomic_add_i32(1, rctl, lane == 0 ? rdy(n) : OOB, 0, 0);
            owe = ~0u;
        }
    };
    auto draw = [&](unsigned n) -> unsigned {
        unsigned t = 0;
        if (lane == 0) t = __hip_atomic_fetch_add((LAS unsigned*)(lds + LDS_RING + R_NEXT), n, __ATOMIC_RELAXED, __HIP_MEMORY_SCOPE_WORKGROUP);
        return t;
    };
    auto look = [&](unsigned k) -> unsigned {
        return *(volatile LAS unsigned*)(lds + LDS_RING + (lane < 8 ? R_DESC + ((k * 16 + lane * 4) & 511) : (lane & 1) * 4));
    };
    auto release = [&](unsigned s) { *(volatile LAS unsigned*)(lds + LDS_RING + R_FREE + (s & 31) * 4) = (s >> 5) + 1; };
    unsigned k = rfl(draw(1)), d = look(k);
    for (;;) {
        for (Patience pt; rdl(d, 8) <= (k | (rdl(d, 3) & 1)); d = look(k)) {
            announce();
            if (pt.over(myws, call)) return;
        }
        if (k >= rdl(d, 9)) break;
        const unsigned r0 = rdl(d, 2), r1 = rdl(d, 6), w = rdl(d, 3), kind = w & 3, n = (w >> 3) & 1023;
        const unsigned tk = draw(kind == K_SUM ? 2 : 1);
        LAS const char* in = lds + (k & 31) * 4096;
        auto next = [&]() { k = kind == K_SUM ? (rfl(tk) + 1) & ~1u : rfl(tk); d = look(k); announce(); };
        if (kind == K_COPY) {
            u32x4 P[4];
#pragma unroll
            for (int i = 0; i < 4; ++i) P[i] = *(LAS const u32x4*)(in + i * 1024 + lane * 16);
            FENCE; release(k); next();
            store_chunk(rd, pko + n * 3072, pko + PKO_MAIN + n * 1024, lane, P, r0);
            if (!(w & 4)) { owe = w; owe_r0 = r0; }
            continue;
        }
        if (kind & k & 1) { next(); continue; }
        unsigned D[16];
        decode(in, r0, cw, lane, D, [&]() { release(k); });
        if (kind == K_UNPACK) {
            next();
            store_in(myws, D, w, lane);
            if (!(w & 4)) owe = w;
            continue;
        }
        float v[32];
        add_chunk(v, D, true);
        decode(in + 4096, r1, cw, lane, D, [&]() { release(k + 1); });
        next();
        add_chunk(v, D, false);
        store_out(rout, (rank >> 1) ^ quad, n, v, lane, w & 4);
    }
    announce();
}

__device__ __forceinline__ void reduce_own(const Params& p, char* myws, unsigned call, const Codec& cw, LAS char* lds, int wid, int lane) {
    const unsigned par = call & 1;
    const rsrc_t rctl = mk(myws, CTL_BYTES), rout = mk(p.out, IN_BYTES), rs = mk(myws + OFF_PKO + par * PKO_PAR, PKO_PAR);
    for (unsigned it = 0;; ++it) {
        LAS unsigned* bx = (LAS unsigned*)(lds + LDS_RING) + (it & 1) * 64;
        if (wid == 0) {
            const unsigned g = rfl(__builtin_amdgcn_raw_ptr_buffer_atomic_add_i32(1, rctl, lane == 0 ? OFF_TKT + 11 * LSTR : OOB, 0, 0)) - (call - 1) * (128 + NGEMM - CPY);
            const unsigned s = lane >> 3, n = lane < 40 ? (s ? 7 + s : g >= GD) * 1024 + g * 8 + (lane & 7) : g >= GD ? GD * 8 - GATE : g * 8;
            const unsigned ro = lane <= 40 ? OFF_RECO + (par * 12 * 1024 + n) * 8 : lane == 41 ? OFF_TMO - 4 : OOB;
            u32x2 rec = {0, 0};
            const unsigned long long t0 = __builtin_amdgcn_s_memrealtime();
            bool dead = false;
            for (unsigned sp = 0; g < 128 && !dead;) {
                rec = __builtin_amdgcn_raw_buffer_load_b64(rctl, ro, 0, AUX_SYS);
                const unsigned long long m = __ballot(rec[1] == call);
                dead = (m >> 41) & 1;
                if ((m & 0x1FFFFFFFFFFull) == 0x1FFFFFFFFFFull) break;
                if (!dead) dead = spin_fail<32>(myws, call, t0, sp);
            }
            bx[lane] = lane < 40 ? rec[0] : lane == 40 ? g : dead;
            WAIT_L(0);
        }
        BAR;
        FENCE;
        const unsigned bv = bx[lane], g = rdl(bv, 40), r0 = rdl(bv, wid), n = g * 8 + wid;
        const bool dead = rdl(bv, 41);
        if (g >= 128) break;
        const unsigned c = g >= GD ? 1024 + n : n;
        load_chunk(mk((g >= GD ? p.ws[(p.rank ^ 1) & 7] : myws) + OFF_PKO + par * PKO_PAR, PKO_PAR), dead ? OOB : c * 3072,
                   is_raw(r0) && !dead ? PKO_MAIN + c * 1024 : OOB, r0, lds + wid * 4096, lane);
        u32x4 Q[4][4];
#pragma unroll
        for (int s = 0; s < 4; ++s)
#pragma unroll
            for (int i = 0; i < 4; ++i)
                Q[s][i] = __builtin_amdgcn_raw_buffer_load_b128(rs, lane * 16, i < 3 ? ((8 + s) * 1024 + n) * 3072 + i * 1024 : PKO_MAIN + ((8 + s) * 1024 + n) * 1024, AUX_SYS);
        WAIT_V(0);
        unsigned D[16];
        float v[32];
        decode(lds + wid * 4096, r0, cw, lane, D, []() {});
        add_chunk(v, D, true);
#pragma unroll
        for (int s = 0; s < 4; ++s) {
#pragma unroll
            for (int i = 0; i < 16; ++i) D[i] = Q[s][i >> 2][i & 3];
            add_chunk(v, D, false);
        }
        store_out(rout, (unsigned)p.rank >> 1, n, v, lane, dead);
    }
}

__global__ __launch_bounds__(512, 2) void gemm_rs_kernel(const Params p) {
    extern __shared__ __attribute__((aligned(16))) char shm[];
    LAS char* lds = (LAS char*)shm;
    const int tid = threadIdx.x, wid = rfl(tid >> 6), lane = tid & 63;
    const unsigned b = blockIdx.x, rank = p.rank & 7;
    char* const myws = p.ws[rank];
    const unsigned z = b & 7, slot = b >> 3, s2 = slot * 2 + (z & 1);
    const unsigned g0 = b >= NGEMM ? 1024 : z >= 6 ? (slot < 12 ? s2 : 84 + s2) : z >= 4 ? (slot < 14 ? 52 + s2 : s2 - 4) : z >= 2 ? (slot < 14 ? 140 : 52) + s2 : !OLDMAP || slot < 14 ? 1024 : 140 + s2;
    if (g0 - URGENT < 1024 - URGENT) __builtin_amdgcn_s_sleep(50);
    u32x4 R[4];
    pack_load(p, g0, wid, lane, R);
    if (tid < 64) *(LAS unsigned*)(lds + LDS_STAGE + 256 + tid * 4) = 0;
    if (tid < 64) *(LAS unsigned*)(lds + LDS_RING + tid * 4) = 0;
    *(LAS unsigned*)(lds + LDS_LUT + tid * 4) = lut_entry(tid);
    unsigned* callp = (unsigned*)(myws + OFF_CALL + b * 128);
    const unsigned call = rfl(__hip_atomic_load(callp, __ATOMIC_RELAXED, __HIP_MEMORY_SCOPE_AGENT)) + 1;
    const unsigned par = call & 1;
    const Codec cw = {lds + LDS_STAGE + 1024 + wid * CW_BYTES, lds + LDS_LUT};
    const unsigned en0 = p.order.e[lane], en1 = p.order.e[64 + lane];
    {
        unsigned G = g0, it = 0;
        unsigned long long jobs = OLDMAP ? 0x1834508CA8A8C4C4ull : 0x18346CA8A8A8A8A8ull;
        asm volatile("" :: "s"(call) : "memory");
        WAIT_L(0); BAR;
        if (tid == 128) __hip_atomic_store(callp, call, __ATOMIC_RELAXED, __HIP_MEMORY_SCOPE_AGENT);
        unsigned Gp = 0, rp = 0, np = 0;
        auto publish = [&]() {
            LAS u32x2* lrec = (LAS u32x2*)(lds + LDS_STAGE + (np++ & 1) * 64);
            if (lane == 0) lrec[wid] = (u32x2){rp, call};
            WAIT_L(0); BAR;
            if (wid < 2) {
                const PackJob j = pack_job(Gp, wid);
                const unsigned h = j.mtx ? j.hi * 4 + wid * 2 + j.kk : wid * 4 + j.hi * 2 + j.kk;
                if ((h >> 1) != (rank >> 1) && lane < 8) {
                    const u32x2 r = lrec[lane];
                    __builtin_amdgcn_raw_buffer_store_b64(r, mk(p.ws[h & 7], CTL_BYTES), lane * 8, OFF_RECI + ((par * 8 + rank) * 2048 + j.rec_idx) * 8, AUX_SYS);
                }
            }
        };
        while (G < 1024) {
            const unsigned Gn = g0 < ((jobs >> 8) & 255) ? G + ((unsigned)jobs & 255) : 1024;
            jobs >>= 8;
            unsigned D[16];
#pragma unroll
            for (int i = 0; i < 16; ++i) D[i] = R[i >> 2][i & 3];
            asm volatile("" : "+v"(D[0]), "+v"(D[4]), "+v"(D[8]), "+v"(D[12]));
            SCHED;
            pack_load(p, it ? Gn : 1024, wid, lane, R);
            SCHED;
            u32x4 P[4];
            const unsigned r0 = encode(D, pick_top(D), cw, lane, P, [&]() {
                __builtin_amdgcn_s_waitcnt(0x0F70);
                FENCE;
                if (it > 1) publish();
            });
            const PackJob j = pack_job(G, wid);
            if (G < 8 * NPUSH) {
#pragma unroll
                for (int o = 0; o < 2; ++o) {
                    const unsigned h = j.hi * 4 + o * 2 + j.kk, c = 8192 + rank * 128 + (j.x ^ o) * 16 + (G & 1) * 8 + wid;
                    store_chunk(mk(p.ws[h & 7] + OFF_PKI + par * PKI_PAR, (h >> 1) != (rank >> 1) ? PKI_PAR : 0), c * 3072, PKI_MAIN + c * 1024, lane, P, r0);
                }
            } else store_chunk(mk(myws + OFF_PKI + par * PKI_PAR, PKI_PAR), j.gc * 3072, PKI_MAIN + j.gc * 1024, lane, P, r0);
            Gp = G; rp = r0;
            if (it == 0 || Gn >= 1024) {
                SCHED; pack_load(p, Gn, wid, lane, R); SCHED;
                WAIT_V(4); publish();
                __builtin_amdgcn_s_waitcnt(0x0F70);
            }
            if (it == INBOX_IT && g0 < 12 * NPUSH) {
                const unsigned id = g0 * 8 + wid, si = id / (16 * NPUSH), jj = id % (16 * NPUSH), ch = 8192 + ((rank ^ (si + 2)) & 7) * 128 + jj;
                const Piece pc = describe(64 | jj >> 4, si, rank, par);
                bool dead = false;
                const unsigned q0 = rfl(records(myws, call, pc.f[0] + (jj & 15) * 8, 1, dead));
                load_chunk(mk(myws + OFF_PKI + par * PKI_PAR, PKI_PAR), ch * 3072, is_raw(q0) ? PKI_MAIN + ch * 1024 : OOB, q0, lds + wid * 4096, lane);
                __builtin_amdgcn_s_waitcnt(0x0F70);
                decode(lds + wid * 4096, q0, cw, lane, D, []() {});
                store_in(myws, D, in_word(pc.f[2], jj & 15), lane);
                WAIT_V(0);
                __builtin_amdgcn_raw_ptr_buffer_atomic_add_i32(1, mk(myws, CTL_BYTES), lane == 0 && !dead ? rdy((pc.f[2] >> 3) & 1023) : OOB, 0, 0);
            }
            G = Gn; ++it;
        }
    }
    const unsigned gi = rank >> 2, gj = (rank >> 1) & 1;
    unsigned role = b - NGEMM;
    unsigned tile = g0 - 108, kq = 0;
    const int wr = wid >> 2, wc = wid & 3, fr = lane & 15, fq = lane >> 4;
    const unsigned voffA = (wid * 8 + (lane >> 3)) * 2048 + (((lane & 7) ^ ((wid * 4 + (lane >> 4)) & 7)) * 16);
    const unsigned voffB = (wid * 4 + (lane >> 4)) * 16384 + (((lane & 15) ^ (((lane >> 4) << 2) | (wid & 2))) * 16);
    const unsigned ldsw = wid * 1024;
    int aoff[2], boff[2];
#pragma unroll
    for (int k = 0; k < 2; ++k) aoff[k] = (wr * 64 + fr) * 128 + (((4 * k + fq) ^ ((fr >> 1) & 7)) * 16);
#pragma unroll
    for (int n = 0; n < 2; ++n) { const int q = (lane & 15) >> 2, pp = lane & 3;
        boff[n] = (int)(unsigned)(size_t)lds + 65536 + (8 * fq + q) * 256 + (((4 * wc + 2 * n + (pp >> 1)) ^ ((q << 2) | ((fq & 1) << 1))) * 16) + 8 * (pp & 1); }
    const rsrc_t rctl = mk(myws, CTL_BYTES);
#define HT 16384
#define SA(bb, h) (((bb) * 2 + (h)) * HT)
#define SB(bb, h) ((4 + (bb) * 2 + (h)) * HT)
#define STAGE_A(bb, h, rs, so) do { _Pragma("unroll") for (int _i = 0; _i < 2; ++_i) \
        __builtin_amdgcn_raw_ptr_buffer_load_lds(rs, (LAS void*)(lds + SA(bb, h) + ldsw + _i * 8192), 16, voffA, (so) + (h) * 262144 + _i * 131072, 0, AUX_AGENT); } while (0)
#define STAGE_B(bb, h, rs, so) do { _Pragma("unroll") for (int _i = 0; _i < 2; ++_i) \
        __builtin_amdgcn_raw_ptr_buffer_load_lds(rs, (LAS void*)(lds + SB(bb, h) + ldsw + _i * 8192), 16, voffB, (so) + (h) * 256 + _i * 524288, 0, AUX_AGENT); } while (0)
#define LDA(dst, bb, h) do { _Pragma("unroll") for (int m = 0; m < 4; ++m) _Pragma("unroll") for (int k = 0; k < 2; ++k) \
        dst[m][k] = *(const LAS bf16x8*)(lds + SA(bb, h) + aoff[k] + m * 2048); } while (0)
#define LDB(T, bb, h) asm volatile( \
        "ds_read_b64_tr_b16 %0, %8 offset:%10\n\tds_read_b64_tr_b16 %1, %8 offset:%11\n\tds_read_b64_tr_b16 %2, %8 offset:%12\n\tds_read_b64_tr_b16 %3, %8 offset:%13\n\t" \
        "ds_read_b64_tr_b16 %4, %9 offset:%10\n\tds_read_b64_tr_b16 %5, %9 offset:%11\n\tds_read_b64_tr_b16 %6, %9 offset:%12\n\tds_read_b64_tr_b16 %7, %9 offset:%13" \
        : "=&v"(T[0]), "=&v"(T[1]), "=&v"(T[2]), "=&v"(T[3]), "=&v"(T[4]), "=&v"(T[5]), "=&v"(T[6]), "=&v"(T[7]) \
        : "v"(boff[0]), "v"(boff[1]), "i"(SB(bb, h) - 65536), "i"(SB(bb, h) - 65536 + 1024), "i"(SB(bb, h) - 65536 + 8192), "i"(SB(bb, h) - 65536 + 9216) : "memory")
#define LDB_USE(dst, T) do { asm volatile("s_waitcnt lgkmcnt(0)" : "+v"(T[0]), "+v"(T[1]), "+v"(T[2]), "+v"(T[3]), "+v"(T[4]), "+v"(T[5]), "+v"(T[6]), "+v"(T[7]) :: "memory"); \
        _Pragma("unroll") for (int _j = 0; _j < 4; ++_j) dst[_j >> 1][_j & 1] = __builtin_shufflevector(T[2 * _j], T[2 * _j + 1], 0, 1, 2, 3, 4, 5, 6, 7); } while (0)
#define MMA(ai, bj, At, Bt) do { __builtin_amdgcn_s_setprio(1); _Pragma("unroll") for (int m = 0; m < 4; ++m) _Pragma("unroll") for (int n = 0; n < 2; ++n) _Pragma("unroll") for (int k = 0; k < 2; ++k) \
        acc[ai][bj][m][n] = __builtin_amdgcn_mfma_f32_16x16x32_bf16(Bt[n][k], At[m][k], acc[ai][bj][m][n], 0, 0, 0); __builtin_amdgcn_s_setprio(0); } while (0)
    constexpr unsigned AK = 128, BK = 1048576;
    for (int un = tile < 32 ? -1 : 0; un < 2 && b < NGEMM; ++un) {
        const unsigned d = un ? 0 : 1 + slot % 7, s0 = un ? kq * 8 : 0, nst = un ? 8 : 32;
        const unsigned ut = un ? tile & 1 : z & 1, uct = un ? (tile >> 1) & 15 : (z >> 1) * 4 + slot / 7;
        const unsigned rt = ut * 8 + d;
        const unsigned aown = ((rank ^ d) * 1024 + gi * 512 + ut * 256) * 2048, bown = (gj * 4096 + uct * 256) * 2;
        const unsigned arem = rt * 256 * 2048, brem = uct * 512;
        const unsigned rdy_off = rdy((lane < 32 ? rt : 16 + uct) * 32 + (lane & 31));
        auto step_a = [&](unsigned s) -> unsigned { const unsigned r = s - 8, q = (r * 43) >> 8, si = r - q * 6;
            return s < 8 ? aown + s * 256 : arem + (si >> 1) * 8 * MiB + (si & 1) * 1024 + q * 256; };
        auto step_b = [&](unsigned s) -> unsigned { const unsigned r = s - 8, q = (r * 43) >> 8, si = r - q * 6;
            return s < 8 ? bown + s * 128 * 16384 : brem + ((si >> 1) * 512 + q * 128) * 16384 + (si & 1) * 8192; };
        LAS unsigned* lseq = (LAS unsigned*)(lds + LDS_STAGE + 256 + (un & 1) * 128);
        const unsigned want = un ? 0xFFu << s0 : 0xFFFFFFFFu;
        unsigned seen = want & 0xFFu, seqv = (lane < 8 && s0 == 0) ? lane + 1 : 0;
        auto get_step = [&](unsigned k) -> unsigned {
            unsigned e = rdl(seqv, k);
            if (e == 0) {
                if (wid == 0 && (unsigned)__builtin_popcount(seen) <= k) {
                    const unsigned long long t0 = __builtin_amdgcn_s_memrealtime();
                    unsigned fresh;
                    for (unsigned sp = 0;;) {
                        const unsigned v = __builtin_amdgcn_raw_buffer_load_b32(rctl, rdy_off, 0, AUX_AGENT);
                        const unsigned long long m = __ballot((int)(v - call * 16u) >= 0);
                        fresh = (((unsigned)m & (unsigned)(m >> 32)) << 8) & want & ~seen;
                        if (fresh) break;
                        if (spin_fail<48>(myws, call, t0, sp)) { fresh = want & ~seen; break; }
                        if (z >= 2 || un) __builtin_amdgcn_s_sleep(60);
                    }
                    if (lane < 32 && ((fresh >> lane) & 1))
                        __hip_atomic_store(lseq + __builtin_popcount(seen) + __builtin_popcount(fresh & ((1u << lane) - 1)), (unsigned)lane + 1, __ATOMIC_RELEASE, __HIP_MEMORY_SCOPE_WORKGROUP);
                    seen |= fresh;
                }
                for (Patience pt;;) {
                    seqv = __hip_atomic_load(lseq + (lane & 31), __ATOMIC_ACQUIRE, __HIP_MEMORY_SCOPE_WORKGROUP);
                    e = rdl(seqv, k);
                    if (e || pt.over(myws, call)) break;
                }
                asm volatile("" ::: "memory");
            }
            return (e - 1) & 31;
        };
        f32x4 acc[2][2][4][2];
#pragma unroll
        for (int a = 0; a < 2; ++a)
#pragma unroll
            for (int bb = 0; bb < 2; ++bb)
#pragma unroll
                for (int m = 0; m < 4; ++m)
#pragma unroll
                    for (int n = 0; n < 2; ++n) acc[a][bb][m][n] = (f32x4){0.f, 0.f, 0.f, 0.f};
        bf16x8 At[4][2], B0[2][2], B1[2][2]; s16x4 TB[8];
        unsigned sn = get_step(0);
        rsrc_t ra = mk(sn < 8 ? p.x : myws + OFF_XS, sn < 8 ? IN_BYTES : XS_BYTES);
        unsigned sa = step_a(sn);
        {
            const rsrc_t rb = mk(sn < 8 ? p.w : myws + OFF_WS, sn < 8 ? IN_BYTES : WS_BYTES);
            const unsigned sb = step_b(sn);
            STAGE_B(0, 0, rb, sb); STAGE_A(0, 0, ra, sa); STAGE_B(0, 1, rb, sb); STAGE_A(0, 1, ra, sa);
            if (wr == 1) BAR;
            WAIT_V(4); BAR;
            STAGE_B(1, 0, rb, sb + BK); STAGE_A(1, 0, ra, sa + AK); STAGE_B(1, 1, rb, sb + BK);
            WAIT_V(6); BAR;
            LDB(TB, 0, 0); LDB_USE(B0, TB);
        }
        for (unsigned s = 0; s < nst; ++s) {
            if (s + 1 < nst) sn = get_step(s + 1);
            const rsrc_t rna = mk(sn < 8 ? p.x : myws + OFF_XS, sn < 8 ? IN_BYTES : XS_BYTES);
            const rsrc_t rnb = mk(sn < 8 ? p.w : myws + OFF_WS, sn < 8 ? IN_BYTES : WS_BYTES);
            const unsigned na = step_a(sn), nb = step_b(sn);
            LDA(At, 0, 0); STAGE_A(1, 1, ra, sa + AK);
            WAIT_V(10); BAR; WAIT_L(0); MMA(0, 0, At, B0); BAR; SCHED;
            LDB(TB, 0, 1); STAGE_B(0, 0, rnb, nb);
            WAIT_V(10); BAR; LDB_USE(B1, TB); MMA(0, 1, At, B1); BAR;
            LDA(At, 0, 1); STAGE_A(0, 0, rna, na);
            WAIT_V(10); BAR; WAIT_L(0); MMA(1, 0, At, B0); BAR; SCHED;
            LDB(TB, 1, 0); STAGE_B(0, 1, rnb, nb);
            WAIT_V(10); BAR; SCHED; MMA(1, 1, At, B1); SCHED; LDB_USE(B0, TB); BAR;
            LDA(At, 1, 0); STAGE_A(0, 1, rna, na);
            WAIT_V(10); BAR; WAIT_L(0); MMA(0, 0, At, B0); BAR; SCHED;
            LDB(TB, 1, 1); STAGE_B(1, 0, rnb, nb + BK);
            WAIT_V(10); BAR; LDB_USE(B1, TB); MMA(0, 1, At, B1); BAR;
            LDA(At, 1, 1); STAGE_A(1, 0, rna, na + AK);
            WAIT_V(10); BAR; WAIT_L(0); MMA(1, 0, At, B0); BAR; SCHED;
            LDB(TB, 0, 0); STAGE_B(1, 1, rnb, nb + BK);
            WAIT_V(10); BAR; SCHED; MMA(1, 1, At, B1); SCHED; LDB_USE(B0, TB); BAR;
            ra = rna; sa = na;
        }
        if (wr == 0) BAR;
        {
            const unsigned dst = (rank ^ d) & 7;
            const unsigned c0 = (un ? 8 + kq : d) * 1024 + (uct >> 1) * 128 + (uct & 1) * 16 + ut * 8 + wid;
            unsigned tk = 0;
            if (un == 0 && wid == 0) tk = __builtin_amdgcn_raw_ptr_buffer_atomic_add_i32(1, rctl, lane == 0 ? OFF_TKT + 10 * LSTR : OOB, 0, 0);
            const rsrc_t rpk = mk(myws + OFF_PKO + par * PKO_PAR, PKO_PAR);
            unsigned top = 1, rp = 0;
            const unsigned tmo = __hip_atomic_load((unsigned*)(myws + OFF_TMO), __ATOMIC_RELAXED, __HIP_MEMORY_SCOPE_AGENT);
            auto publish = [&](unsigned c, unsigned r0, int n) {
                __builtin_amdgcn_raw_buffer_store_b64((u32x2){r0, call}, mk(p.ws[dst], CTL_BYTES), tmo != call && lane < n ? OFF_RECO + (par * 12 * 1024 + c - lane * 32) * 8 : OOB, 0, AUX_SYS);
            };
#pragma unroll
            for (int u = 0; u < 4; ++u) {
                unsigned D[16];
#pragma unroll
                for (int m = 0; m < 4; ++m)
#pragma unroll
                    for (int n = 0; n < 2; ++n) {
                        const f32x4 a = acc[u >> 1][u & 1][m][n];
                        D[m * 4 + n * 2] = cvt_pk_bf16(a[0], a[1]); D[m * 4 + n * 2 + 1] = cvt_pk_bf16(a[2], a[3]);
                    }
                u32x4 P[4];
                unsigned r0 = RAW << 16;
                const unsigned c = c0 + u * 32;
                if (un) {
#pragma unroll
                    for (int i = 0; i < 16; ++i) P[i >> 2][i & 3] = D[i];
                } else {
                    if (u == 0) top = pick_top(D);
                    r0 = encode(D, top, cw, lane, P, [&]() { if (u) { WAIT_V(0); publish(c - 32, rp, 1); } });
                }
                store_chunk(rpk, c * 3072, PKO_MAIN + c * 1024, lane, P, r0);
                rp = r0;
            }
            WAIT_V(0);
            publish(c0 + 96, rp, un ? 4 : 1);
            if (un == 0 && tid == 0) *(LAS unsigned*)(lds + LDS_STAGE + 128) = tk - (call - 1) * NGEMM + 32;
            WAIT_L(0);
        }
        BAR;
        if (un) continue;
        role = rfl(*(volatile LAS unsigned*)(lds + LDS_STAGE + 128));
        if (role < 32 + CPY) break;
        role += 256 - (32 + CPY);
        if (role >= 256 + 96) break;
        tile = (role - 256) / 3; kq = 1 + (role - 256) % 3;
    }
    if (role < 32 + CPY) ring(p, myws, call, lds, cw, role, en0, en1, wid, lane);
    else reduce_own(p, myws, call, cw, lds, wid, lane);
}

}

extern "C" void kernel_launch(int rank, int nranks, ncclComm_t comm, hipStream_t stream,
                              void* const* d_in, const int* in_sizes, int n_in,
                              void* d_out, int out_size,
                              const int* in_dtypes, int out_dtype,
                              void* const* d_out_all,
                              void* const* d_ws_all, size_t ws_bytes) {
    (void)comm; (void)in_dtypes; (void)out_dtype; (void)d_out_all;
    if (nranks != NR || (unsigned)rank >= (unsigned)NR || n_in < 2 || ws_bytes < WS_NEED || out_size != 1024 * 8192 ||
        in_sizes[0] != 8192 * 1024 || in_sizes[1] != 1024 * 8192) {
        fprintf(stderr, "kernel_launch: unexpected problem geometry\n");
        return;
    }
    HIP_CHECK(hipFuncSetAttribute((const void*)gemm_rs_kernel, hipFuncAttributeMaxDynamicSharedMemorySize, LDS_BYTES));
    Params p;
    p.x = (const char*)d_in[0]; p.w = (const char*)d_in[1]; p.out = (char*)d_out; p.rank = rank;
    constexpr Order order = make_order();
    p.order = order;
    for (int r = 0; r < NR; ++r) p.ws[r] = (char*)d_ws_all[r];
    hipLaunchKernelGGL(gemm_rs_kernel, dim3(256), dim3(512), LDS_BYTES, stream, p);
    HIP_CHECK(hipGetLastError());
}
```

```cpp
#include <hip/hip_runtime.h>
#include <rccl/rccl.h>
#include <cstdio>

#define HIP_CHECK(expr) do { hipError_t _e = (expr); if (_e != hipSuccess) { \
    fprintf(stderr, "HIP error %s at %s:%d\n", hipGetErrorString(_e), __FILE__, __LINE__); \
    return; } } while (0)

namespace {

#define LAS __attribute__((address_space(3)))
typedef short bf16x8 __attribute__((ext_vector_type(8)));
typedef short s16x4 __attribute__((ext_vector_type(4)));
typedef float f32x4 __attribute__((ext_vector_type(4)));
typedef unsigned u32x4 __attribute__((ext_vector_type(4)));
typedef unsigned u32x2 __attribute__((ext_vector_type(2)));
typedef unsigned short u16x2 __attribute__((ext_vector_type(2)));
typedef __amdgpu_buffer_rsrc_t rsrc_t;

constexpr int NR = 8;
constexpr unsigned MiB = 1u << 20;
constexpr unsigned OFF_CALL = 0;
constexpr unsigned OFF_TMO = 32768;
constexpr unsigned LSTR = 4352;
constexpr unsigned OFF_RDY = 786432;
constexpr unsigned OFF_TKT = OFF_RDY + 32 * LSTR;
static_assert(OFF_TKT + 12 * LSTR <= 1u << 20, "");
__device__ __forceinline__ unsigned rdy(unsigned n) { return OFF_RDY + (n >> 5) * LSTR + (n & 31) * 4; }
constexpr unsigned OFF_RECI = 65536;
constexpr unsigned OFF_RECO = 327680;
constexpr unsigned CTL_BYTES = MiB;
constexpr unsigned OFF_PKI = 1 * MiB, PKI_MAIN = 27 * MiB, PKI_PAR = 36 * MiB;
constexpr unsigned OFF_PKO = 73 * MiB, PKO_MAIN = 36 * MiB, PKO_PAR = 48 * MiB;
constexpr unsigned OFF_XS = 169 * MiB, XS_BYTES = 24 * MiB;
constexpr unsigned OFF_WS = 193 * MiB, WS_BYTES = 24 * MiB;
constexpr unsigned WS_NEED = 217 * MiB;
constexpr unsigned IN_BYTES = 16 * MiB;
constexpr unsigned OOB = 0x80000000u;
constexpr unsigned long long DEADLINE = 30000000ull;

constexpr int LDS_STAGE = 131072, LDS_LUT = LDS_STAGE + 1024 + 8 * 2128, LDS_RING = LDS_LUT + 2048, LDS_BYTES = LDS_RING + 1024;
constexpr int NGEMM = 224;
constexpr int NPW = 5;
constexpr int DEPTH = 4;
constexpr unsigned AHEAD = 4;
constexpr unsigned TAILG = 40;
constexpr unsigned CPY = 3;
constexpr unsigned GD = 120, GATE = 48;
constexpr int NPUSH = 4;
constexpr unsigned URGENT = 72;
constexpr int AUX_SYS = 17, AUX_AGENT = 16;
constexpr int SLOPE = 28, PAIR = 24, XSTEP = 22, GROUPT = 156;

__device__ __forceinline__ unsigned xpiece(unsigned x) { return (unsigned)(0x80A2FD75EC64B931ull >> (4 * x)) & 15; }
struct Order { unsigned char e[128]; };
constexpr Order make_order() {
    Order o{}; int key[128] = {}; int n = 0;
    for (int q = 0; q < 4; ++q) {
        for (int x = 0; x < 16; ++x) {
            o.e[n] = (unsigned char)(64 | q << 4 | x);
            key[n++] = (q == 0 && x < NPUSH) ? 200000 : x >= 14 ? 100000 + q : !q ? -1 : x < 2 ? q * GROUPT - 1 : q * GROUPT + 1 + (x - 2) / 2 * XSTEP; }
        for (int c = 0; c < 16; ++c) { o.e[n] = (unsigned char)(q << 4 | c); key[n++] = q * GROUPT + c * SLOPE - (c & 1) * PAIR; }
    }
    for (int i = 1; i < 128; ++i)
        for (int j = i; j > 0 && key[j - 1] > key[j]; --j) {
            const int k = key[j]; key[j] = key[j - 1]; key[j - 1] = k; const unsigned char e = o.e[j]; o.e[j] = o.e[j - 1]; o.e[j - 1] = e; }
    return o;
}

struct Params {
    const char* x; const char* w; char* out; char* ws[NR]; int rank; Order order;
};

__device__ __forceinline__ rsrc_t mk(const void* p, unsigned bytes) { return __builtin_amdgcn_make_buffer_rsrc((void*)p, 0, bytes, 0x00020000); }
__device__ __forceinline__ unsigned rfl(unsigned v) { return __builtin_amdgcn_readfirstlane(v); }
__device__ __forceinline__ unsigned rdl(unsigned v, unsigned l) { return __builtin_amdgcn_readlane(v, l); }
#define WAIT_V(n) asm volatile("s_waitcnt vmcnt(" #n ")" ::: "memory")
#define WAIT_L(n) asm volatile("s_waitcnt lgkmcnt(" #n ")" ::: "memory")
#define BAR __builtin_amdgcn_s_barrier()
#define SCHED __builtin_amdgcn_sched_barrier(0)


__device__ __forceinline__ unsigned cvt_pk_bf16(float lo, float hi) {
    unsigned r; asm("v_cvt_pk_bf16_f32 %0, %1, %2" : "=v"(r) : "v"(lo), "v"(hi)); return r;
}
__device__ __forceinline__ unsigned pk_sub(unsigned a, unsigned b) { return __builtin_bit_cast(unsigned, __builtin_bit_cast(u16x2, a) - __builtin_bit_cast(u16x2, b)); }
__device__ __forceinline__ unsigned bfi(unsigned m, unsigned a, unsigned b) { return (a & m) | (b & ~m); }
__device__ __forceinline__ unsigned rotr(unsigned a, unsigned n) { return __builtin_amdgcn_alignbit(a, a, n); }
template <int CTRL> __device__ __forceinline__ float dpp_f(float v) { return __builtin_bit_cast(float, __builtin_amdgcn_update_dpp(0, __builtin_bit_cast(int, v), CTRL, 0xf, 0xf, true)); }
__device__ __forceinline__ float rdl_f(float v, unsigned l) { return __builtin_bit_cast(float, rdl(__builtin_bit_cast(unsigned, v), l)); }

constexpr unsigned RAW = 255, EB = 1040, CW_BYTES = EB + 1024 + 64;
struct Codec { LAS char* cw; LAS const char* lut; };
__device__ __forceinline__ bool is_raw(unsigned r0) { return ((r0 >> 16) & 255) == RAW; }
__device__ __forceinline__ unsigned plane2_bytes(unsigned r0) { return 704 + 4 * ((r0 >> 16) & 255); }
__device__ __forceinline__ unsigned lut_entry(unsigned t) {
    unsigned k = 0, e = t < 256 ? 0 : 0x0C0C0C0Cu;
#pragma unroll
    for (unsigned b = 0; b < 4; ++b) {
        const unsigned c = (t >> (2 * b)) & 3;
        if (t < 256) e |= (c ? 4 + c : k) << (8 * b); else if (!c) e ^= (12 ^ b) << (8 * k);
        k += !c;
    }
    return e;
}
__device__ __forceinline__ unsigned scan_add(unsigned v) {
    int s = v;
    s += __builtin_amdgcn_update_dpp(0, s, 0x111, 0xf, 0xf, false); s += __builtin_amdgcn_update_dpp(0, s, 0x112, 0xf, 0xf, false);
    s += __builtin_amdgcn_update_dpp(0, s, 0x114, 0xf, 0xf, false); s += __builtin_amdgcn_update_dpp(0, s, 0x118, 0xf, 0xf, false);
    s += __builtin_amdgcn_update_dpp(0, s, 0x142, 0xa, 0xf, false); s += __builtin_amdgcn_update_dpp(0, s, 0x143, 0xc, 0xf, false);
    return s;
}
__device__ __forceinline__ unsigned pick_top(const unsigned (&D)[16]) {
    float sm = 0.f;
#pragma unroll
    for (int i = 0; i < 16; i += 4) sm += __builtin_fabsf(__builtin_bit_cast(float, D[i] << 16)) + __builtin_fabsf(__builtin_bit_cast(float, D[i] & 0xFFFF0000u));
    sm += dpp_f<0xB1>(sm); sm += dpp_f<0x4E>(sm); sm += dpp_f<0x141>(sm); sm += dpp_f<0x140>(sm);
    const float top = ((rdl_f(sm, 0) + rdl_f(sm, 16)) + (rdl_f(sm, 32) + rdl_f(sm, 48))) * (20.6f / 512.f);
    return (rfl(__builtin_bit_cast(unsigned, top) >> 15) & 0xFFFFu) | 1u;
}
#define FENCE asm volatile("" ::: "memory")
template <class Fn>
__device__ __forceinline__ unsigned encode(const unsigned (&D)[16], unsigned top, const Codec& k, int lane, u32x4 (&P)[4], Fn mid) {
    LAS char* const cw = k.cw;
    const unsigned top2 = top * 0x10001u;
    unsigned F[8], sel[8], C[2] = {0, 0};
#pragma unroll
    for (int i = 0; i < 8; ++i) {
        const unsigned a = pk_sub(top2, rotr(D[2 * i], 31)), b = pk_sub(top2, rotr(D[2 * i + 1], 31));
        P[i >> 2][i & 3] = __builtin_amdgcn_perm(b, a, 0x06040200u);
        F[i] = __builtin_amdgcn_perm(b, a, 0x07050301u);
        const unsigned c = __builtin_amdgcn_perm(0x00000302u, 0x01000000u, F[i]);
        const unsigned cb = __builtin_amdgcn_udot4((c ^ (c >> 2)) & 0x03030303u, 0x40100401u, 0, false);
        sel[i] = *(LAS const unsigned*)(k.lut + 1024 + cb * 4);
        C[i >> 2] |= cb << (8 * (i & 3));
    }
    const unsigned m[2] = {~(C[0] | C[0] >> 1) & 0x55555555u, ~(C[1] | C[1] >> 1) & 0x55555555u};
    const unsigned k0 = __builtin_popcount(m[0]), kk = k0 + __builtin_popcount(m[1]), incl = scan_add(kk), cnt = rdl(incl, 63);
    P[2] = (u32x4){F[0], F[1], F[2], F[3]}; P[3] = (u32x4){F[4], F[5], F[6], F[7]};
    mid();
    if (cnt > 1024) return top | RAW << 16;
    const unsigned fill = lane * 16u >= cnt ? 0x02020202u : 0;
    *(LAS u32x4*)(cw + EB + lane * 16) = (u32x4){fill, fill, fill, fill};
    FENCE;
    *(LAS unsigned char*)(cw + EB + cnt + lane) = 2;
    *(LAS u32x2*)(cw + lane * 8) = (u32x2){C[0], C[1]};
    FENCE;
    const unsigned e1 = (unsigned)(size_t)cw + EB - 1 + incl;
    unsigned carry = 0;
#pragma unroll
    for (int i = 0; i < 8; ++i) {
        const unsigned p = e1 - kk + (i >= 4 ? k0 : 0) + __builtin_popcount(m[i >> 2] & ((1u << (8 * (i & 3))) - 1)), pk = __builtin_amdgcn_perm(0u, F[i], sel[i]);
        __hip_atomic_fetch_or((LAS unsigned*)(size_t)(p & ~3u), __builtin_amdgcn_alignbyte(pk, 0u, ~p) | carry, __ATOMIC_RELAXED, __HIP_MEMORY_SCOPE_WAVEFRONT);
        carry = __builtin_amdgcn_alignbyte(0u, pk, ~p);
    }
    __hip_atomic_fetch_or((LAS unsigned*)(size_t)(e1 & ~3u), carry, __ATOMIC_RELAXED, __HIP_MEMORY_SCOPE_WAVEFRONT);
    FENCE;
    unsigned M = 0, ovf = 0;
    auto dense = [&](int r) -> unsigned {
        const u32x2 f = *(LAS u32x2*)(cw + EB + r * 512 + lane * 8);
        const unsigned t0 = __builtin_amdgcn_perm(0u, 0x00074040u, f[0]), t1 = __builtin_amdgcn_perm(0u, 0x00074040u, f[1]);
        M |= bfi(0x40404040u, t0, t1 << 1) >> (6 - 2 * r) & 0x03030303u << (2 * r);
        return ((f[0] ^ t0) & 0x07070707u) | ((f[1] ^ t1) & 0x07070707u) << 3;
    };
    const unsigned x = dense(0), y = bfi(0xC0C0C0u, __umul24(x >> 24, 0x41040u), x);
    *(LAS unsigned short*)(cw + 512 + lane * 2) = (unsigned short)y;
    *(LAS unsigned char*)(cw + 640 + lane) = (unsigned char)(y >> 16);
    if (cnt > 512) { *(LAS unsigned*)(cw + 704 + lane * 4) = dense(1); ovf = (cnt - 505) >> 3; }
    FENCE;
    unsigned n = 0;
    for (;;) {
        const unsigned long long act = __ballot(M != 0);
        if (!act) break;
        if (M) {
            const unsigned off = ovf + n + __builtin_amdgcn_mbcnt_hi((unsigned)(act >> 32), __builtin_amdgcn_mbcnt_lo((unsigned)act, 0));
            const unsigned bt = __builtin_ctz(M), pos = (bt & 2) * 256 + lane * 8 + (bt & 1) * 4 + (bt >> 3);
            *(LAS unsigned*)(cw + 704 + (off < 80 ? off : 80) * 4) = pos | (unsigned)*(LAS unsigned char*)(cw + EB + pos) << 16;
            M &= M - 1;
        }
        n += __builtin_popcountll(act);
        if (n > 64 || ovf + n > 80) return top | RAW << 16;
    }
    FENCE;
    P[2] = *(LAS u32x4*)(cw + lane * 16);
    FENCE;
    return top | (ovf + n) << 16 | n << 24;
}
template <class Fn>
__device__ __forceinline__ void decode(LAS const char* in, unsigned r0, const Codec& k, int lane, unsigned (&D)[16], Fn done) {
    LAS char* const cw = k.cw;
    LAS const char* const p2 = in + 2048;
    const unsigned n = r0 >> 24, ovf = ((r0 >> 16) & 255) - n, top2 = (r0 & 0xFFFFu) * 0x10001u;
    unsigned F[8];
    u32x4 L[2];
    FENCE;
    if (is_raw(r0)) {
        const u32x4 f0 = *(LAS const u32x4*)(p2 + lane * 16), f1 = *(LAS const u32x4*)(p2 + 1024 + lane * 16);
        L[0] = *(LAS const u32x4*)(in + lane * 16); L[1] = *(LAS const u32x4*)(in + 1024 + lane * 16);
        FENCE; done(); FENCE;
#pragma unroll
        for (int i = 0; i < 4; ++i) { F[i] = f0[i]; F[i + 4] = f1[i]; }
    } else {
        const u32x2 C = *(LAS const u32x2*)(p2 + lane * 8);
        const unsigned y = *(LAS const unsigned short*)(p2 + 512 + lane * 2) | (unsigned)*(LAS const unsigned char*)(p2 + 640 + lane) << 16;
        const unsigned xo = *(LAS const unsigned*)(p2 + 704 + lane * 4), e = *(LAS const unsigned*)(p2 + 704 + ((ovf + lane) & 127) * 4);
        L[0] = *(LAS const u32x4*)(in + lane * 16); L[1] = *(LAS const u32x4*)(in + 1024 + lane * 16);
        FENCE; done(); FENCE;
        auto spread = [&](int r, unsigned x) {
            *(LAS u32x2*)(cw + EB + r * 512 + lane * 8) = (u32x2){__builtin_amdgcn_perm(0x0706020Cu, 0x0B0A0908u, x & 0x07070707u),
                                                                  __builtin_amdgcn_perm(0x0706020Cu, 0x0B0A0908u, (x >> 3) & 0x07070707u)};
        };
        unsigned sel[8];
#pragma unroll
        for (int i = 0; i < 8; ++i) sel[i] = *(LAS const unsigned*)(k.lut + ((C[i >> 2] >> (8 * (i & 3))) & 255) * 4);
        spread(0, bfi(0x3F000000u, __umul24(y & 0xC0C0C0u, 0x41040u), y));
        if (ovf) spread(1, xo);
        FENCE;
        *(LAS unsigned char*)(cw + ((unsigned)lane < n ? EB + (e & 1023) : 1024)) = (unsigned char)(e >> 16);
        FENCE;
        const unsigned m[2] = {~(C[0] | C[0] >> 1) & 0x55555555u, ~(C[1] | C[1] >> 1) & 0x55555555u};
        const unsigned k0 = __builtin_popcount(m[0]), kk = k0 + __builtin_popcount(m[1]), p0 = (unsigned)(size_t)cw + EB + scan_add(kk) - kk;
#pragma unroll
        for (int i = 0; i < 8; ++i) {
            const unsigned p = p0 + (i >= 4 ? k0 : 0) + __builtin_popcount(m[i >> 2] & ((1u << (8 * (i & 3))) - 1));
            const LAS unsigned* w = (const LAS unsigned*)(size_t)(p & ~3u);
            F[i] = __builtin_amdgcn_perm(0x05040300u, __builtin_amdgcn_alignbyte(w[1], w[0], p), sel[i]);
        }
        FENCE;
    }
#pragma unroll
    for (int i = 0; i < 8; ++i) {
        D[2 * i] = rotr(pk_sub(top2, __builtin_amdgcn_perm(F[i], L[i >> 2][i & 3], 0x05010400u)), 1);
        D[2 * i + 1] = rotr(pk_sub(top2, __builtin_amdgcn_perm(F[i], L[i >> 2][i & 3], 0x07030602u)), 1);
    }
}
__device__ __forceinline__ void store_chunk(rsrc_t rs, unsigned main_off, unsigned esc_off, int lane, const u32x4 (&P)[4], unsigned r0) {
#pragma unroll
    for (int i = 0; i < 2; ++i) __builtin_amdgcn_raw_buffer_store_b128(P[i], rs, main_off + lane * 16 + i * 1024, 0, AUX_SYS);
    __builtin_amdgcn_raw_buffer_store_b128(P[2], rs, lane * 16u < plane2_bytes(r0) ? main_off + lane * 16 + 2048 : OOB, 0, AUX_SYS);
    __builtin_amdgcn_raw_buffer_store_b128(P[3], rs, is_raw(r0) ? esc_off + lane * 16 : OOB, 0, AUX_SYS);
}
__device__ __forceinline__ void load_chunk(rsrc_t rs, unsigned main_off, unsigned esc_off, unsigned r0, LAS char* to, int lane) {
    if (esc_off != OOB) __builtin_amdgcn_raw_ptr_buffer_load_lds(rs, (LAS void*)(to + 3072), 16, lane * 16, esc_off, 0, AUX_SYS);
    __builtin_amdgcn_raw_ptr_buffer_load_lds(rs, (LAS void*)to, 16, lane * 16, main_off, 0, AUX_SYS);
    __builtin_amdgcn_raw_ptr_buffer_load_lds(rs, (LAS void*)(to + 1024), 16, lane * 16 + 1024, main_off, 0, AUX_SYS);
    __builtin_amdgcn_raw_ptr_buffer_load_lds(rs, (LAS void*)(to + 2048), 16, lane * 16u < plane2_bytes(r0) ? lane * 16 + 2048 : OOB, main_off, 0, AUX_SYS);
}

template <int SLEEP>
__device__ __forceinline__ bool spin_fail(char* myws, unsigned call, unsigned long long t0, unsigned& n) {
    unsigned* tmo = (unsigned*)(myws + OFF_TMO);
    if ((++n & 63) == 0 &&
        (rfl(__hip_atomic_load(tmo, __ATOMIC_RELAXED, __HIP_MEMORY_SCOPE_AGENT)) == call || __builtin_amdgcn_s_memrealtime() - t0 > DEADLINE)) {
        __hip_atomic_store(tmo, call, __ATOMIC_RELAXED, __HIP_MEMORY_SCOPE_AGENT);
        return true;
    }
    __builtin_amdgcn_s_sleep(SLEEP);
    return false;
}

struct PackJob { unsigned src_off, stride, gc, rec_idx, x; int mtx, kk, hi; };
__device__ __forceinline__ PackJob pack_job(unsigned G, int wid) {
    PackJob j; const unsigned q = G >> 8, ch = (G & 1) * 8 + wid;
    j.mtx = ((G >> 7) & 1) ^ 1; j.x = (G >> 3) & 15; j.hi = (G >> 2) & 1; j.kk = (G >> 1) & 1; const unsigned ks = j.kk * 4 + q;
    unsigned t16, tile;
    if (j.mtx) { const unsigned e = xpiece(j.x), sh = (j.hi * 4 + j.kk) ^ (e & 7), t = e >> 3;
        tile = sh * 4 + j.hi * 2 + t; t16 = sh * 2 + t; j.src_off = (tile * 256 + ch * 16) * 2048 + ks * 256; j.stride = 2048; }
    else { tile = j.hi * 16 + j.x; t16 = j.x; j.src_off = (ks * 128 + (ch >> 1) * 16) * 16384 + tile * 512 + (ch & 1) * 256; j.stride = 16384; }
    j.gc = (j.mtx * 256 + tile * 8 + ks) * 16 + ch;
    j.rec_idx = (j.mtx * 64 + t16 * 4 + q) * 16 + (G & 1) * 8;
    return j;
}
__device__ __forceinline__ void pack_load(const Params& p, unsigned G, int wid, int lane, u32x4 (&R)[4]) {
    const PackJob j = pack_job(G < 1024 ? G : 0, wid);
    const rsrc_t rs = mk(j.mtx ? p.x : p.w, G < 1024 ? IN_BYTES : 0);
    const unsigned vo = (lane >> 4) * j.stride + (lane & 15) * 16;
#pragma unroll
    for (int i = 0; i < 4; ++i) R[i] = __builtin_amdgcn_raw_buffer_load_b128(rs, vo, j.src_off + i * 4 * j.stride, 0);
}

__device__ __forceinline__ void add_chunk(float (&v)[32], const unsigned (&D)[16], bool first) {
#pragma unroll
    for (int i = 0; i < 16; ++i) {
        const float a = __builtin_bit_cast(float, D[i] << 16), b = __builtin_bit_cast(float, D[i] & 0xFFFF0000u);
        v[2 * i] = first ? a : v[2 * i] + a; v[2 * i + 1] = first ? b : v[2 * i + 1] + b;
    }
}
__device__ __forceinline__ void store_out(rsrc_t rout, unsigned pr, unsigned n, const float (&v)[32], int lane, bool dead) {
    const unsigned t = (n >> 3) & 1, ct = ((n >> 6) & 14) | ((n >> 4) & 1), w = n & 7, u = (n >> 5) & 3;
    const int fr = lane & 15, fq = lane >> 4;
    const unsigned vo = fr * 16384 + ((fq & 1) * 16 + (fq >> 1) * 8) * 2;
    const unsigned so = ((pr >> 1) * 512 + t * 256 + (u >> 1) * 128 + (w >> 2) * 64) * 16384 + ((pr & 1) * 4096 + ct * 256 + (u & 1) * 128 + (w & 3) * 32) * 2;
#pragma unroll
    for (int m = 0; m < 4; ++m) {
        unsigned d[4];
#pragma unroll
        for (int h = 0; h < 4; ++h) {
            const float a = v[8 * m + 2 * h], b = v[8 * m + 2 * h + 1];
            d[h] = cvt_pk_bf16(a < 0.f ? 0.f : a, b < 0.f ? 0.f : b);
        }
        const auto s0 = __builtin_amdgcn_permlane16_swap(d[0], d[2], false, false);
        const auto s1 = __builtin_amdgcn_permlane16_swap(d[1], d[3], false, false);
        u32x4 o = {s0[0], s1[0], s0[1], s1[1]};
        if (dead) o = (u32x4){0x7FC07FC0u, 0x7FC07FC0u, 0x7FC07FC0u, 0x7FC07FC0u};
        __builtin_amdgcn_raw_buffer_store_b128(o, rout, vo + so + m * 16 * 16384, 0, AUX_AGENT);
    }
}

enum { K_UNPACK = 0, K_SUM = 1, K_COPY = 2, K_END = 3 };
enum { R_FULL = 0, R_ISSUED = 4, R_AVAIL = 8, R_NEXT = 12, R_FREE = 128, R_DESC = 256 };
__device__ __forceinline__ unsigned lds_ld2(unsigned a, unsigned b, unsigned& vb) {
    unsigned v; asm volatile("ds_read_b32 %0, %2\n\tds_read_b32 %1, %3\n\ts_waitcnt lgkmcnt(0)" : "=&v"(v), "=&v"(vb) : "v"(a), "v"(b) : "memory"); return v; }
__device__ __forceinline__ void lds_st(unsigned a, unsigned v) { asm volatile("ds_write_b32 %0, %1" :: "v"(a), "v"(v) : "memory"); }
struct Patience {
    unsigned n = 0; unsigned long long t0 = 0;
    __device__ __forceinline__ bool over(char* myws, unsigned call) {
        if ((n++ & 1023) == 0) {
            const unsigned long long t = __builtin_amdgcn_s_memrealtime();
            if (n == 1) t0 = t;
            if (t - t0 > 2 * DEADLINE) { __hip_atomic_store((unsigned*)(myws + OFF_TMO), call, __ATOMIC_RELAXED, __HIP_MEMORY_SCOPE_AGENT); return true; }
        }
        __builtin_amdgcn_s_sleep(1);
        return false;
    }
};
struct Piece { unsigned f[3]; };
__device__ __forceinline__ Piece describe(unsigned en, unsigned si, unsigned rank, unsigned par) {
    const unsigned mtx = (en >> 6) & 1, q = (en >> 4) & 3, ct = en & 15, g = (rank ^ (si + 2)) & 7;
    const unsigned y = (ct + 10 + 2 * si) % 12, x = !q || ct >= 14 ? ct : ct < 2 ? 2 * ct : y ? y + 2 : 1;
    const unsigned e = xpiece(x), t = e >> 3, d = e & 7, sh = rank ^ d, ks = (rank & 1) * 4 + q;
    const unsigned tile32 = mtx ? sh * 4 + (rank >> 2) * 2 + t : (rank & 2) * 8 + ct, t16 = mtx ? sh * 2 + t : ct;
    const unsigned dst = mtx ? (si >> 1) * 8 * MiB + (t * 8 + d) * 256 * 2048 + (si & 1) * 1024 + q * 256
                             : XS_BYTES + ((si >> 1) * 512 + q * 128) * 16384 + (si & 1) * 8192 + ct * 512;
    return {{OFF_RECI + ((par * 8 + g) * 2048 + (mtx * 64 + t16 * 4 + q) * 16) * 8, (mtx * 256 + tile32 * 8 + ks) * 16,
             dst << 5 | ((mtx ? t * 8 + d : 16 + ct) * 32 + q * 6 + si) << 3}};
}
__device__ __forceinline__ unsigned in_word(unsigned w, unsigned ch) { return w + ((w < XS_BYTES << 5 ? ch * 32768 : (ch >> 1) * 262144 + (ch & 1) * 256) << 5); }
__device__ __forceinline__ unsigned records(char* myws, unsigned call, unsigned ro, unsigned need, bool& dead) {
    const unsigned long long t0 = __builtin_amdgcn_s_memrealtime();
    for (unsigned sp = 0; !dead;) {
        const u32x2 rec = __builtin_amdgcn_raw_buffer_load_b64(mk(myws, CTL_BYTES), ro, 0, AUX_SYS);
        if (((unsigned)__ballot(rec[1] == call) & need) == need) return rec[0];
        dead = spin_fail<4>(myws, call, t0, sp);
    }
    return 0;
}
__device__ __forceinline__ void store_in(char* myws, const unsigned (&D)[16], unsigned w, int lane) {
    const unsigned dst = w >> 13 << 8, st = dst < XS_BYTES ? 2048 : 16384, vo = (lane >> 4) * st + (lane & 15) * 16 + dst;
#pragma unroll
    for (int i = 0; i < 4; ++i)
        __builtin_amdgcn_raw_buffer_store_b128((u32x4){D[4 * i], D[4 * i + 1], D[4 * i + 2], D[4 * i + 3]}, mk(myws + OFF_XS, XS_BYTES + WS_BYTES), vo + i * 4 * st, 0, AUX_AGENT);
}
__device__ __forceinline__ void ring(const Params& p, char* myws, unsigned call, LAS char* lds, const Codec& cw, unsigned c, unsigned en0, unsigned en1, int wid, int lane) {
    const unsigned rank = p.rank & 7, par = call & 1;
    const bool cp = c >= 30;
    const unsigned si = c % 6, g = (rank ^ (cp ? 1 : si + 2)) & 7, quad = 1 + (si >> 1), ctl = (unsigned)(size_t)lds + LDS_RING;
    const unsigned pki = OFF_PKI + par * PKI_PAR, pko = OFF_PKO + par * PKO_PAR, reco = OFF_RECO + par * 12 * 1024 * 8;
    const rsrc_t rctl = mk(myws, CTL_BYTES);
    if (wid == 0) {
        __builtin_amdgcn_s_setprio(3);
        unsigned long long src0 = (unsigned long long)p.ws[g], src1 = (unsigned long long)p.ws[g ^ 1];
        asm volatile("" : "+s"(src0), "+s"(src1));
        unsigned av = 0, k0 = 0, d = 0, lim = 0;
        unsigned long long raw = 0;
        int full = 0;
        auto post = [&](unsigned issued) { asm volatile("ds_write_b64 %0, %1" :: "v"(ctl + R_FULL), "v"((u32x2){(unsigned)full, issued}) : "memory"); };
        for (unsigned k = 0;; k += 4) {
            if (av <= k) {
                k0 = k;
                for (Patience pt;;) {
                    const unsigned q = rfl(lds_ld2(ctl + R_AVAIL, ctl + R_DESC + ((k * 16 + lane * 4) & 511), d));
                    av = q < k + 16 ? q : k + 16;
                    if (av > k) break;
                    if (full < (int)k) { WAIT_V(0); full = k; post(k); } else if (pt.over(myws, call)) return;
                }
                lim = plane2_bytes(d); raw = __ballot((lane & 3) == 1 && d != OOB);
            }
            const unsigned l = (k - k0) * 4, w = rdl(d, l + 3);
            if ((w & 3) == K_END) {
                WAIT_V(6); full = max(full, (int)k - 2); post(k);
                __builtin_amdgcn_s_waitcnt(0x0F70);
                full = 0x7FFFFFFF; post(k);
                break;
            }
            const rsrc_t rs[2] = {mk((void*)(w & si & 1 ? src1 : src0), WS_NEED), mk((void*)(w & ~si & 1 ? src1 : src0), WS_NEED)};
            LAS char* const to = lds + (k & 31) * 4096;
            if ((raw >> l) & 0xFFFF)
#pragma unroll
                for (int i = 0; i < 4; ++i) __builtin_amdgcn_raw_ptr_buffer_load_lds(rs[i & 1], (LAS void*)(to + i * 4096 + 3072), 16, lane * 16, rdl(d, l + 4 * i + 1), 0, AUX_SYS);
#pragma unroll
            for (int i = 0; i < 4; ++i) {
                const unsigned o0 = rdl(d, l + 4 * i), v2 = lane * 16u < rdl(lim, l + 4 * i + 2) ? lane * 16 : OOB;
                SCHED;
                asm volatile("s_waitcnt vmcnt(%0)" :: "i"(3 * (DEPTH - 1)) : "memory");
                __builtin_amdgcn_raw_ptr_buffer_load_lds(rs[i & 1], (LAS void*)(to + i * 4096), 16, lane * 16, o0, 0, AUX_SYS);
                __builtin_amdgcn_raw_ptr_buffer_load_lds(rs[i & 1], (LAS void*)(to + i * 4096), 16, lane * 16, o0, 1024, AUX_SYS);
                __builtin_amdgcn_raw_ptr_buffer_load_lds(rs[i & 1], (LAS void*)(to + i * 4096), 16, v2, o0, 2048, AUX_SYS);
                SCHED;
                full = max(full, (int)k + i + 1 - DEPTH);
                post(k + i + 1);
            }
        }
    }
    if (wid == 1) {
        const Piece tb[2] = {describe(en0, si, rank, par), describe(en1, si, rank, par)};
        auto piece = [&](int f, unsigned pc) -> unsigned { const unsigned a = rdl(tb[0].f[f], pc & 63), b = rdl(tb[1].f[f], pc & 63); return pc & 64 ? b : a; };
        constexpr unsigned NG_IN = (128 - NPUSH) * 2;
        const unsigned big = (cp ? GD : 256) - TAILG / 2, ng_out = big + TAILG, l = lane & 7;
        unsigned kind = cp ? K_COPY : K_UNPACK, gr = cp ? c - 30 : c / 6, tk = 0, ro = OOB, ch = 0, w = 0;
        u32x2 rec = {0, 0};
        bool dead = false;
        auto ask = [&]() {
            if (kind == K_UNPACK && gr >= NG_IN) { kind = K_SUM; gr = (c / 6) * 2 + (si & 1); }
            if (kind != K_UNPACK && gr >= ng_out) { kind = K_END; return; }
            if (kind == K_UNPACK) { const unsigned h = (gr & 1) * 8 + l; ro = piece(0, gr >> 1) + h * 8; ch = piece(1, gr >> 1) + h; w = in_word(piece(2, gr >> 1), h); }
            else {
                const unsigned sl = (gr < big ? gr * 8 : big * 4 + gr * 4) + l, n = (cp ? sl : sl >> 1) & 1023;
                ch = (cp ? 1 : 2 * quad + (l & 1)) * 1024 + n; ro = reco + ch * 8; w = kind | n << 3;
            }
            rec = __builtin_amdgcn_raw_buffer_load_b64(rctl, lane < 8 ? ro : lane == 8 ? OFF_TMO - 4 : OOB, 0, AUX_SYS);
            tk = __builtin_amdgcn_raw_ptr_buffer_atomic_add_i32(1, rctl, lane == 0 ? OFF_TKT + (kind == K_UNPACK ? si : cp ? 6 : 6 + quad) * LSTR : OOB, 0, 0);
        };
        ask();
        for (unsigned k = 0;;) {
            const bool in = kind == K_UNPACK, end = kind == K_END;
            const unsigned cnt = in || gr < big ? 8 : 4, need = (1u << cnt) - 1;
            u32x4 d = {OOB, OOB, rec[0], K_END};
            if (!end) {
                const unsigned m = (unsigned)__ballot(rec[1] == call);
                dead |= (m >> 8) & 1;
                if ((m & need) != need) d[2] = records(myws, call, lane < 8 ? ro : OOB, need, dead);
                d[3] = dead ? w | 4 : w;
                if (!dead) { d[0] = (in ? pki : pko) + ch * 3072; if (is_raw(d[2])) d[1] = (in ? pki + PKI_MAIN : pko + PKO_MAIN) + ch * 1024; }
                gr = rfl(tk) - (call - 1) * (in ? NG_IN : ng_out) + (in ? NPW : cp ? 2 + CPY : 2 * NPW);
                ask();
            }
            for (Patience pt;;) {
                unsigned fr;
                const unsigned s = k + l;
                if (rfl(lds_ld2(ctl + R_ISSUED, ctl + R_FREE + (s & 31) * 4, fr)) + AHEAD >= k && !__ballot(fr * 32 + 32 <= s)) break;
                if (pt.over(myws, call)) return;
            }
            if (lane < 8) *(LAS u32x4*)(lds + LDS_RING + R_DESC + ((k + l) & 31) * 16) = d;
            k += cnt;
            lds_st(ctl + R_AVAIL, k);
            if (end) break;
        }
    }
    const rsrc_t rd = mk(myws, WS_NEED), rout = mk(p.out, IN_BYTES);
    unsigned owe = ~0u, owe_r0 = 0;
    auto announce = [&]() {
        if (owe != ~0u) {
            WAIT_V(0);
            const unsigned n = (owe >> 3) & 1023;
            if (cp) __builtin_amdgcn_raw_buffer_store_b64((u32x2){owe_r0, call}, rctl, lane == 0 ? reco + n * 8 : OOB, 0, AUX_SYS);
            else __builtin_amdgcn_raw_ptr_buffer_atomic_add_i32(1, rctl, lane == 0 ? rdy(n) : OOB, 0, 0);
            owe = ~0u;
        }
    };
    auto draw = [&](unsigned n) -> unsigned {
        unsigned t = 0;
        if (lane == 0) t = __hip_atomic_fetch_add((LAS unsigned*)(lds + LDS_RING + R_NEXT), n, __ATOMIC_RELAXED, __HIP_MEMORY_SCOPE_WORKGROUP);
        return t;
    };
    auto look = [&](unsigned k) -> unsigned {
        return *(volatile LAS unsigned*)(lds + LDS_RING + (lane < 8 ? R_DESC + ((k * 16 + lane * 4) & 511) : (lane & 1) * 4));
    };
    auto release = [&](unsigned s) { *(volatile LAS unsigned*)(lds + LDS_RING + R_FREE + (s & 31) * 4) = (s >> 5) + 1; };
    unsigned k = rfl(draw(1)), d = look(k);
    for (;;) {
        for (Patience pt; rdl(d, 8) <= (k | (rdl(d, 3) & 1)); d = look(k)) {
            announce();
            if (pt.over(myws, call)) return;
        }
        if (k >= rdl(d, 9)) break;
        const unsigned r0 = rdl(d, 2), r1 = rdl(d, 6), w = rdl(d, 3), kind = w & 3, n = (w >> 3) & 1023;
        const unsigned tk = draw(kind == K_SUM ? 2 : 1);
        LAS const char* in = lds + (k & 31) * 4096;
        auto next = [&]() { k = kind == K_SUM ? (rfl(tk) + 1) & ~1u : rfl(tk); d = look(k); announce(); };
        if (kind == K_COPY) {
            u32x4 P[4];
#pragma unroll
            for (int i = 0; i < 4; ++i) P[i] = *(LAS const u32x4*)(in + i * 1024 + lane * 16);
            FENCE; release(k); next();
            store_chunk(rd, pko + n * 3072, pko + PKO_MAIN + n * 1024, lane, P, r0);
            if (!(w & 4)) { owe = w; owe_r0 = r0; }
            continue;
        }
        if (kind & k & 1) { next(); continue; }
        unsigned D[16];
        decode(in, r0, cw, lane, D, [&]() { release(k); });
        if (kind == K_UNPACK) {
            next();
            store_in(myws, D, w, lane);
            if (!(w & 4)) owe = w;
            continue;
        }
        float v[32];
        add_chunk(v, D, true);
        decode(in + 4096, r1, cw, lane, D, [&]() { release(k + 1); });
        next();
        add_chunk(v, D, false);
        store_out(rout, (rank >> 1) ^ quad, n, v, lane, w & 4);
    }
    announce();
}

__device__ __forceinline__ void reduce_own(const Params& p, char* myws, unsigned call, const Codec& cw, LAS char* lds, int wid, int lane) {
    const unsigned par = call & 1;
    const rsrc_t rctl = mk(myws, CTL_BYTES), rout = mk(p.out, IN_BYTES), rs = mk(myws + OFF_PKO + par * PKO_PAR, PKO_PAR);
    for (unsigned it = 0;; ++it) {
        LAS unsigned* bx = (LAS unsigned*)(lds + LDS_RING) + (it & 1) * 64;
        if (wid == 0) {
            const unsigned g = rfl(__builtin_amdgcn_raw_ptr_buffer_atomic_add_i32(1, rctl, lane == 0 ? OFF_TKT + 11 * LSTR : OOB, 0, 0)) - (call - 1) * (128 + NGEMM - CPY);
            const unsigned s = lane >> 3, n = lane < 40 ? (s ? 7 + s : g >= GD) * 1024 + g * 8 + (lane & 7) : g >= GD ? GD * 8 - GATE : g * 8;
            const unsigned ro = lane <= 40 ? OFF_RECO + (par * 12 * 1024 + n) * 8 : lane == 41 ? OFF_TMO - 4 : OOB;
            u32x2 rec = {0, 0};
            const unsigned long long t0 = __builtin_amdgcn_s_memrealtime();
            bool dead = false;
            for (unsigned sp = 0; g < 128 && !dead;) {
                rec = __builtin_amdgcn_raw_buffer_load_b64(rctl, ro, 0, AUX_SYS);
                const unsigned long long m = __ballot(rec[1] == call);
                dead = (m >> 41) & 1;
                if ((m & 0x1FFFFFFFFFFull) == 0x1FFFFFFFFFFull) break;
                if (!dead) dead = spin_fail<32>(myws, call, t0, sp);
            }
            bx[lane] = lane < 40 ? rec[0] : lane == 40 ? g : dead;
            WAIT_L(0);
        }
        BAR;
        FENCE;
        const unsigned bv = bx[lane], g = rdl(bv, 40), r0 = rdl(bv, wid), n = g * 8 + wid;
        const bool dead = rdl(bv, 41);
        if (g >= 128) break;
        const unsigned c = g >= GD ? 1024 + n : n;
        load_chunk(mk((g >= GD ? p.ws[(p.rank ^ 1) & 7] : myws) + OFF_PKO + par * PKO_PAR, PKO_PAR), dead ? OOB : c * 3072,
                   is_raw(r0) && !dead ? PKO_MAIN + c * 1024 : OOB, r0, lds + wid * 4096, lane);
        u32x4 Q[4][4];
#pragma unroll
        for (int s = 0; s < 4; ++s)
#pragma unroll
            for (int i = 0; i < 4; ++i)
                Q[s][i] = __builtin_amdgcn_raw_buffer_load_b128(rs, lane * 16, i < 3 ? ((8 + s) * 1024 + n) * 3072 + i * 1024 : PKO_MAIN + ((8 + s) * 1024 + n) * 1024, AUX_SYS);
        WAIT_V(0);
        unsigned D[16];
        float v[32];
        decode(lds + wid * 4096, r0, cw, lane, D, []() {});
        add_chunk(v, D, true);
#pragma unroll
        for (int s = 0; s < 4; ++s) {
#pragma unroll
            for (int i = 0; i < 16; ++i) D[i] = Q[s][i >> 2][i & 3];
            add_chunk(v, D, false);
        }
        store_out(rout, (unsigned)p.rank >> 1, n, v, lane, dead);
    }
}

__global__ __launch_bounds__(512, 2) void gemm_rs_kernel(const Params p) {
    extern __shared__ __attribute__((aligned(16))) char shm[];
    LAS char* lds = (LAS char*)shm;
    const int tid = threadIdx.x, wid = rfl(tid >> 6), lane = tid & 63;
    const unsigned b = blockIdx.x, rank = p.rank & 7;
    char* const myws = p.ws[rank];
    const unsigned z = b & 7, slot = b >> 3, s2 = slot * 2 + (z & 1);
    const unsigned g0 = b >= NGEMM ? 1024 : z >= 6 ? (slot < 12 ? s2 : 84 + s2) : z >= 4 ? (slot < 14 ? 52 + s2 : s2 - 4) : z >= 2 ? (slot < 14 ? 140 : 52) + s2 : 1024;
    if (g0 - URGENT < 1024 - URGENT) __builtin_amdgcn_s_sleep(50);
    u32x4 R[4];
    pack_load(p, g0, wid, lane, R);
    if (tid < 64) *(LAS unsigned*)(lds + LDS_STAGE + 256 + tid * 4) = 0;
    if (tid < 64) *(LAS unsigned*)(lds + LDS_RING + tid * 4) = 0;
    *(LAS unsigned*)(lds + LDS_LUT + tid * 4) = lut_entry(tid);
    unsigned* callp = (unsigned*)(myws + OFF_CALL + b * 128);
    const unsigned call = rfl(__hip_atomic_load(callp, __ATOMIC_RELAXED, __HIP_MEMORY_SCOPE_AGENT)) + 1;
    const unsigned par = call & 1;
    const Codec cw = {lds + LDS_STAGE + 1024 + wid * CW_BYTES, lds + LDS_LUT};
    const unsigned en0 = p.order.e[lane], en1 = p.order.e[64 + lane];
    {
        unsigned G = g0, it = 0;
        unsigned long long jobs = 0x18346CA8A8A8A8A8ull;
        asm volatile("" :: "s"(call) : "memory");
        WAIT_L(0); BAR;
        if (tid == 128) __hip_atomic_store(callp, call, __ATOMIC_RELAXED, __HIP_MEMORY_SCOPE_AGENT);
        unsigned Gp = 0, rp = 0, np = 0;
        auto publish = [&]() {
            LAS u32x2* lrec = (LAS u32x2*)(lds + LDS_STAGE + (np++ & 1) * 64);
            if (lane == 0) lrec[wid] = (u32x2){rp, call};
            WAIT_L(0); BAR;
            if (wid < 2) {
                const PackJob j = pack_job(Gp, wid);
                const unsigned h = j.mtx ? j.hi * 4 + wid * 2 + j.kk : wid * 4 + j.hi * 2 + j.kk;
                if ((h >> 1) != (rank >> 1) && lane < 8) {
                    const u32x2 r = lrec[lane];
                    __builtin_amdgcn_raw_buffer_store_b64(r, mk(p.ws[h & 7], CTL_BYTES), lane * 8, OFF_RECI + ((par * 8 + rank) * 2048 + j.rec_idx) * 8, AUX_SYS);
                }
            }
        };
        while (G < 1024) {
            const unsigned Gn = g0 < ((jobs >> 8) & 255) ? G + ((unsigned)jobs & 255) : 1024;
            jobs >>= 8;
            unsigned D[16];
#pragma unroll
            for (int i = 0; i < 16; ++i) D[i] = R[i >> 2][i & 3];
            asm volatile("" : "+v"(D[0]), "+v"(D[4]), "+v"(D[8]), "+v"(D[12]));
            SCHED;
            pack_load(p, it ? Gn : 1024, wid, lane, R);
            SCHED;
            u32x4 P[4];
            const unsigned r0 = encode(D, pick_top(D), cw, lane, P, [&]() {
                __builtin_amdgcn_s_waitcnt(0x0F70);
                FENCE;
                if (it > 1) publish();
            });
            const PackJob j = pack_job(G, wid);
            if (G < 8 * NPUSH) {
#pragma unroll
                for (int o = 0; o < 2; ++o) {
                    const unsigned h = j.hi * 4 + o * 2 + j.kk, c = 8192 + rank * 128 + (j.x ^ o) * 16 + (G & 1) * 8 + wid;
                    store_chunk(mk(p.ws[h & 7] + OFF_PKI + par * PKI_PAR, (h >> 1) != (rank >> 1) ? PKI_PAR : 0), c * 3072, PKI_MAIN + c * 1024, lane, P, r0);
                }
            } else store_chunk(mk(myws + OFF_PKI + par * PKI_PAR, PKI_PAR), j.gc * 3072, PKI_MAIN + j.gc * 1024, lane, P, r0);
            Gp = G; rp = r0;
            if (it == 0 || Gn >= 1024) {
                SCHED; pack_load(p, Gn, wid, lane, R); SCHED;
                WAIT_V(4); publish();
                __builtin_amdgcn_s_waitcnt(0x0F70);
            }
            if (it == 3 && g0 < 12 * NPUSH) {
                const unsigned id = g0 * 8 + wid, si = id / (16 * NPUSH), jj = id % (16 * NPUSH), ch = 8192 + ((rank ^ (si + 2)) & 7) * 128 + jj;
                const Piece pc = describe(64 | jj >> 4, si, rank, par);
                bool dead = false;
                const unsigned q0 = rfl(records(myws, call, pc.f[0] + (jj & 15) * 8, 1, dead));
                load_chunk(mk(myws + OFF_PKI + par * PKI_PAR, PKI_PAR), ch * 3072, is_raw(q0) ? PKI_MAIN + ch * 1024 : OOB, q0, lds + wid * 4096, lane);
                __builtin_amdgcn_s_waitcnt(0x0F70);
                decode(lds + wid * 4096, q0, cw, lane, D, []() {});
                store_in(myws, D, in_word(pc.f[2], jj & 15), lane);
                WAIT_V(0);
                __builtin_amdgcn_raw_ptr_buffer_atomic_add_i32(1, mk(myws, CTL_BYTES), lane == 0 && !dead ? rdy((pc.f[2] >> 3) & 1023) : OOB, 0, 0);
            }
            G = Gn; ++it;
        }
    }
    const unsigned gi = rank >> 2, gj = (rank >> 1) & 1;
    unsigned role = b - NGEMM;
    unsigned tile = g0 - 108, kq = 0;
    const int wr = wid >> 2, wc = wid & 3, fr = lane & 15, fq = lane >> 4;
    const unsigned voffA = (wid * 8 + (lane >> 3)) * 2048 + (((lane & 7) ^ ((wid * 4 + (lane >> 4)) & 7)) * 16);
    const unsigned voffB = (wid * 4 + (lane >> 4)) * 16384 + (((lane & 15) ^ (((lane >> 4) << 2) | (wid & 2))) * 16);
    const unsigned ldsw = wid * 1024;
    int aoff[2], boff[2];
#pragma unroll
    for (int k = 0; k < 2; ++k) aoff[k] = (wr * 64 + fr) * 128 + (((4 * k + fq) ^ ((fr >> 1) & 7)) * 16);
#pragma unroll
    for (int n = 0; n < 2; ++n) { const int q = (lane & 15) >> 2, pp = lane & 3;
        boff[n] = (int)(unsigned)(size_t)lds + 65536 + (8 * fq + q) * 256 + (((4 * wc + 2 * n + (pp >> 1)) ^ ((q << 2) | ((fq & 1) << 1))) * 16) + 8 * (pp & 1); }
    const rsrc_t rctl = mk(myws, CTL_BYTES);
#define HT 16384
#define SA(bb, h) (((bb) * 2 + (h)) * HT)
#define SB(bb, h) ((4 + (bb) * 2 + (h)) * HT)
#define STAGE_A(bb, h, rs, so) do { _Pragma("unroll") for (int _i = 0; _i < 2; ++_i) \
        __builtin_amdgcn_raw_ptr_buffer_load_lds(rs, (LAS void*)(lds + SA(bb, h) + ldsw + _i * 8192), 16, voffA, (so) + (h) * 262144 + _i * 131072, 0, AUX_AGENT); } while (0)
#define STAGE_B(bb, h, rs, so) do { _Pragma("unroll") for (int _i = 0; _i < 2; ++_i) \
        __builtin_amdgcn_raw_ptr_buffer_load_lds(rs, (LAS void*)(lds + SB(bb, h) + ldsw + _i * 8192), 16, voffB, (so) + (h) * 256 + _i * 524288, 0, AUX_AGENT); } while (0)
#define LDA(dst, bb, h) do { _Pragma("unroll") for (int m = 0; m < 4; ++m) _Pragma("unroll") for (int k = 0; k < 2; ++k) \
        dst[m][k] = *(const LAS bf16x8*)(lds + SA(bb, h) + aoff[k] + m * 2048); } while (0)
#define LDB(T, bb, h) asm volatile( \
        "ds_read_b64_tr_b16 %0, %8 offset:%10\n\tds_read_b64_tr_b16 %1, %8 offset:%11\n\tds_read_b64_tr_b16 %2, %8 offset:%12\n\tds_read_b64_tr_b16 %3, %8 offset:%13\n\t" \
        "ds_read_b64_tr_b16 %4, %9 offset:%10\n\tds_read_b64_tr_b16 %5, %9 offset:%11\n\tds_read_b64_tr_b16 %6, %9 offset:%12\n\tds_read_b64_tr_b16 %7, %9 offset:%13" \
        : "=&v"(T[0]), "=&v"(T[1]), "=&v"(T[2]), "=&v"(T[3]), "=&v"(T[4]), "=&v"(T[5]), "=&v"(T[6]), "=&v"(T[7]) \
        : "v"(boff[0]), "v"(boff[1]), "i"(SB(bb, h) - 65536), "i"(SB(bb, h) - 65536 + 1024), "i"(SB(bb, h) - 65536 + 8192), "i"(SB(bb, h) - 65536 + 9216) : "memory")
#define LDB_USE(dst, T) do { asm volatile("s_waitcnt lgkmcnt(0)" : "+v"(T[0]), "+v"(T[1]), "+v"(T[2]), "+v"(T[3]), "+v"(T[4]), "+v"(T[5]), "+v"(T[6]), "+v"(T[7]) :: "memory"); \
        _Pragma("unroll") for (int _j = 0; _j < 4; ++_j) dst[_j >> 1][_j & 1] = __builtin_shufflevector(T[2 * _j], T[2 * _j + 1], 0, 1, 2, 3, 4, 5, 6, 7); } while (0)
#define MMA(ai, bj, At, Bt) do { __builtin_amdgcn_s_setprio(1); _Pragma("unroll") for (int m = 0; m < 4; ++m) _Pragma("unroll") for (int n = 0; n < 2; ++n) _Pragma("unroll") for (int k = 0; k < 2; ++k) \
        acc[ai][bj][m][n] = __builtin_amdgcn_mfma_f32_16x16x32_bf16(Bt[n][k], At[m][k], acc[ai][bj][m][n], 0, 0, 0); __builtin_amdgcn_s_setprio(0); } while (0)
    constexpr unsigned AK = 128, BK = 1048576;
    for (int un = tile < 32 ? -1 : 0; un < 2 && b < NGEMM; ++un) {
        const unsigned d = un ? 0 : 1 + slot % 7, s0 = un ? kq * 8 : 0, nst = un ? 8 : 32;
        const unsigned ut = un ? tile & 1 : z & 1, uct = un ? (tile >> 1) & 15 : (z >> 1) * 4 + slot / 7;
        const unsigned rt = ut * 8 + d;
        const unsigned aown = ((rank ^ d) * 1024 + gi * 512 + ut * 256) * 2048, bown = (gj * 4096 + uct * 256) * 2;
        const unsigned arem = rt * 256 * 2048, brem = uct * 512;
        const unsigned rdy_off = rdy((lane < 32 ? rt : 16 + uct) * 32 + (lane & 31));
        auto step_a = [&](unsigned s) -> unsigned { const unsigned r = s - 8, q = (r * 43) >> 8, si = r - q * 6;
            return s < 8 ? aown + s * 256 : arem + (si >> 1) * 8 * MiB + (si & 1) * 1024 + q * 256; };
        auto step_b = [&](unsigned s) -> unsigned { const unsigned r = s - 8, q = (r * 43) >> 8, si = r - q * 6;
            return s < 8 ? bown + s * 128 * 16384 : brem + ((si >> 1) * 512 + q * 128) * 16384 + (si & 1) * 8192; };
        LAS unsigned* lseq = (LAS unsigned*)(lds + LDS_STAGE + 256 + (un & 1) * 128);
        const unsigned want = un ? 0xFFu << s0 : 0xFFFFFFFFu;
        unsigned seen = want & 0xFFu, seqv = (lane < 8 && s0 == 0) ? lane + 1 : 0;
        auto get_step = [&](unsigned k) -> unsigned {
            unsigned e = rdl(seqv, k);
            if (e == 0) {
                if (wid == 0 && (unsigned)__builtin_popcount(seen) <= k) {
                    const unsigned long long t0 = __builtin_amdgcn_s_memrealtime();
                    unsigned fresh;
                    for (unsigned sp = 0;;) {
                        const unsigned v = __builtin_amdgcn_raw_buffer_load_b32(rctl, rdy_off, 0, AUX_AGENT);
                        const unsigned long long m = __ballot((int)(v - call * 16u) >= 0);
                        fresh = (((unsigned)m & (unsigned)(m >> 32)) << 8) & want & ~seen;
                        if (fresh) break;
                        if (spin_fail<48>(myws, call, t0, sp)) { fresh = want & ~seen; break; }
                        if (z >= 2 || un) __builtin_amdgcn_s_sleep(60);
                    }
                    if (lane < 32 && ((fresh >> lane) & 1))
                        __hip_atomic_store(lseq + __builtin_popcount(seen) + __builtin_popcount(fresh & ((1u << lane) - 1)), (unsigned)lane + 1, __ATOMIC_RELEASE, __HIP_MEMORY_SCOPE_WORKGROUP);
                    seen |= fresh;
                }
                for (Patience pt;;) {
                    seqv = __hip_atomic_load(lseq + (lane & 31), __ATOMIC_ACQUIRE, __HIP_MEMORY_SCOPE_WORKGROUP);
                    e = rdl(seqv, k);
                    if (e || pt.over(myws, call)) break;
                }
                asm volatile("" ::: "memory");
            }
            return (e - 1) & 31;
        };
        f32x4 acc[2][2][4][2];
#pragma unroll
        for (int a = 0; a < 2; ++a)
#pragma unroll
            for (int bb = 0; bb < 2; ++bb)
#pragma unroll
                for (int m = 0; m < 4; ++m)
#pragma unroll
                    for (int n = 0; n < 2; ++n) acc[a][bb][m][n] = (f32x4){0.f, 0.f, 0.f, 0.f};
        bf16x8 At[4][2], B0[2][2], B1[2][2]; s16x4 TB[8];
        unsigned sn = get_step(0);
        rsrc_t ra = mk(sn < 8 ? p.x : myws + OFF_XS, sn < 8 ? IN_BYTES : XS_BYTES);
        unsigned sa = step_a(sn);
        {
            const rsrc_t rb = mk(sn < 8 ? p.w : myws + OFF_WS, sn < 8 ? IN_BYTES : WS_BYTES);
            const unsigned sb = step_b(sn);
            STAGE_B(0, 0, rb, sb); STAGE_A(0, 0, ra, sa); STAGE_B(0, 1, rb, sb); STAGE_A(0, 1, ra, sa);
            if (wr == 1) BAR;
            WAIT_V(4); BAR;
            STAGE_B(1, 0, rb, sb + BK); STAGE_A(1, 0, ra, sa + AK); STAGE_B(1, 1, rb, sb + BK);
            WAIT_V(6); BAR;
            LDB(TB, 0, 0); LDB_USE(B0, TB);
        }
        for (unsigned s = 0; s < nst; ++s) {
            if (s + 1 < nst) sn = get_step(s + 1);
            const rsrc_t rna = mk(sn < 8 ? p.x : myws + OFF_XS, sn < 8 ? IN_BYTES : XS_BYTES);
            const rsrc_t rnb = mk(sn < 8 ? p.w : myws + OFF_WS, sn < 8 ? IN_BYTES : WS_BYTES);
            const unsigned na = step_a(sn), nb = step_b(sn);
            LDA(At, 0, 0); STAGE_A(1, 1, ra, sa + AK);
            WAIT_V(10); BAR; WAIT_L(0); MMA(0, 0, At, B0); BAR; SCHED;
            LDB(TB, 0, 1); STAGE_B(0, 0, rnb, nb);
            WAIT_V(10); BAR; LDB_USE(B1, TB); MMA(0, 1, At, B1); BAR;
            LDA(At, 0, 1); STAGE_A(0, 0, rna, na);
            WAIT_V(10); BAR; WAIT_L(0); MMA(1, 0, At, B0); BAR; SCHED;
            LDB(TB, 1, 0); STAGE_B(0, 1, rnb, nb);
            WAIT_V(10); BAR; SCHED; MMA(1, 1, At, B1); SCHED; LDB_USE(B0, TB); BAR;
            LDA(At, 1, 0); STAGE_A(0, 1, rna, na);
            WAIT_V(10); BAR; WAIT_L(0); MMA(0, 0, At, B0); BAR; SCHED;
            LDB(TB, 1, 1); STAGE_B(1, 0, rnb, nb + BK);
            WAIT_V(10); BAR; LDB_USE(B1, TB); MMA(0, 1, At, B1); BAR;
            LDA(At, 1, 1); STAGE_A(1, 0, rna, na + AK);
            WAIT_V(10); BAR; WAIT_L(0); MMA(1, 0, At, B0); BAR; SCHED;
            LDB(TB, 0, 0); STAGE_B(1, 1, rnb, nb + BK);
            WAIT_V(10); BAR; SCHED; MMA(1, 1, At, B1); SCHED; LDB_USE(B0, TB); BAR;
            ra = rna; sa = na;
        }
        if (wr == 0) BAR;
        {
            const unsigned dst = (rank ^ d) & 7;
            const unsigned c0 = (un ? 8 + kq : d) * 1024 + (uct >> 1) * 128 + (uct & 1) * 16 + ut * 8 + wid;
            unsigned tk = 0;
            if (un == 0 && wid == 0) tk = __builtin_amdgcn_raw_ptr_buffer_atomic_add_i32(1, rctl, lane == 0 ? OFF_TKT + 10 * LSTR : OOB, 0, 0);
            const rsrc_t rpk = mk(myws + OFF_PKO + par * PKO_PAR, PKO_PAR);
            unsigned top = 1, rp = 0;
            const unsigned tmo = __hip_atomic_load((unsigned*)(myws + OFF_TMO), __ATOMIC_RELAXED, __HIP_MEMORY_SCOPE_AGENT);
            auto publish = [&](unsigned c, unsigned r0, int n) {
                __builtin_amdgcn_raw_buffer_store_b64((u32x2){r0, call}, mk(p.ws[dst], CTL_BYTES), tmo != call && lane < n ? OFF_RECO + (par * 12 * 1024 + c - lane * 32) * 8 : OOB, 0, AUX_SYS);
            };
#pragma unroll
            for (int u = 0; u < 4; ++u) {
                unsigned D[16];
#pragma unroll
                for (int m = 0; m < 4; ++m)
#pragma unroll
                    for (int n = 0; n < 2; ++n) {
                        const f32x4 a = acc[u >> 1][u & 1][m][n];
                        D[m * 4 + n * 2] = cvt_pk_bf16(a[0], a[1]); D[m * 4 + n * 2 + 1] = cvt_pk_bf16(a[2], a[3]);
                    }
                u32x4 P[4];
                unsigned r0 = RAW << 16;
                const unsigned c = c0 + u * 32;
                if (un) {
#pragma unroll
                    for (int i = 0; i < 16; ++i) P[i >> 2][i & 3] = D[i];
                } else {
                    if (u == 0) top = pick_top(D);
                    r0 = encode(D, top, cw, lane, P, [&]() { if (u) { WAIT_V(0); publish(c - 32, rp, 1); } });
                }
                store_chunk(rpk, c * 3072, PKO_MAIN + c * 1024, lane, P, r0);
                rp = r0;
            }
            WAIT_V(0);
            publish(c0 + 96, rp, un ? 4 : 1);
            if (un == 0 && tid == 0) *(LAS unsigned*)(lds + LDS_STAGE + 128) = tk - (call - 1) * NGEMM + 32;
            WAIT_L(0);
        }
        BAR;
        if (un) continue;
        role = rfl(*(volatile LAS unsigned*)(lds + LDS_STAGE + 128));
        if (role < 32 + CPY) break;
        role += 256 - (32 + CPY);
        if (role >= 256 + 96) break;
        tile = (role - 256) / 3; kq = 1 + (role - 256) % 3;
    }
    if (role < 32 + CPY) ring(p, myws, call, lds, cw, role, en0, en1, wid, lane);
    else reduce_own(p, myws, call, cw, lds, wid, lane);
}

}

extern "C" void kernel_launch(int rank, int nranks, ncclComm_t comm, hipStream_t stream,
                              void* const* d_in, const int* in_sizes, int n_in,
                              void* d_out, int out_size,
                              const int* in_dtypes, int out_dtype,
                              void* const* d_out_all,
                              void* const* d_ws_all, size_t ws_bytes) {
    (void)comm; (void)in_dtypes; (void)out_dtype; (void)d_out_all;
    if (nranks != NR || (unsigned)rank >= (unsigned)NR || n_in < 2 || ws_bytes < WS_NEED || out_size != 1024 * 8192 ||
        in_sizes[0] != 8192 * 1024 || in_sizes[1] != 1024 * 8192) {
        fprintf(stderr, "kernel_launch: unexpected problem geometry\n");
        return;
    }
    HIP_CHECK(hipFuncSetAttribute((const void*)gemm_rs_kernel, hipFuncAttributeMaxDynamicSharedMemorySize, LDS_BYTES));
    Params p;
    p.x = (const char*)d_in[0]; p.w = (const char*)d_in[1]; p.out = (char*)d_out; p.rank = rank;
    constexpr Order order = make_order();
    p.order = order;
    for (int r = 0; r < NR; ++r) p.ws[r] = (char*)d_ws_all[r];
    hipLaunchKernelGGL(gemm_rs_kernel, dim3(256), dim3(512), LDS_BYTES, stream, p);
    HIP_CHECK(hipGetLastError());
}
```
